```python
import numpy as np

kernel_name = "gpu_gemm_rs_m8192_k8192_n8192_bf16_none"


ABSMAX_RTOL = 0.008

COMPUTE_DTYPE = "bfloat16"
EPILOGUE = "none"
IN_QUANT = None
OUT_QUANT = None
IN_DTYPES = {"x": "bfloat16", "w_mat": "bfloat16"}
WORLD_SIZE = 8

def _to_bf16(x):
    u = np.ascontiguousarray(x, dtype=np.float32).view(np.uint32)
    return ((u + 0x7FFF + ((u >> 16) & 1)) & 0xFFFF0000).view(np.float32)

def setup_inputs():

    x = _to_bf16(np.random.randn(8192, 8192).astype(np.float32))
    w_mat = _to_bf16((np.random.randn(8192, 8192) / np.sqrt(8192)).astype(np.float32))
    return {"x": x, "w_mat": w_mat}

def _epilogue(y):
    return y

def reference(x, w_mat):
    y = (x @ w_mat).astype(np.float32)
    return _epilogue(y)
```

```cpp
#include <hip/hip_runtime.h>
#include <rccl/rccl.h>
#include <cstdio>

namespace gr {
typedef unsigned u32;
typedef unsigned long long u64;
typedef unsigned short u16;
typedef u32 u32x4 __attribute__((ext_vector_type(4)));
typedef u16 u16x2 __attribute__((ext_vector_type(2)));
typedef short bf16x8 __attribute__((ext_vector_type(8)));
typedef short s16x4 __attribute__((ext_vector_type(4)));
typedef float f32x4 __attribute__((ext_vector_type(4)));
#define LAS __attribute__((address_space(3)))
#define GAS __attribute__((address_space(1)))
typedef __amdgpu_buffer_rsrc_t rsrc_t;

constexpr int NR = 8;
constexpr int NWG = 256, NTH = 512, NGEMM = 192, NCOMM = NWG - NGEMM;
constexpr int GEMM_WAVES = NGEMM * 8, COMM_WAVES = NCOMM * 8;
constexpr u32 SLOT = 8448;
constexpr u32 HDR = 8192;
constexpr u64 DEADLINE = 100000000ull;

constexpr u32 OFF_CALL = 0;
constexpr u32 OFF_TMO = 16384;
constexpr u32 OFF_RDYA = 20480;
constexpr u32 OFF_RDYB = 24576;
constexpr u32 OFF_FLG = 32768;
constexpr u32 F_IN = 0;
constexpr u32 F_D3 = F_IN + 2 * 16 * 64 * 4 * 8;
constexpr u32 F_P = F_D3 + 64 * 2 * 16 * 8;
constexpr u32 F_R = F_P + 64 * 16 * 8;
constexpr u32 F_OWN = F_R + 32 * 16 * 8;
constexpr u32 FLG_BYTES = F_OWN + 64 * 4 * 16 * 8;
constexpr u32 OFF_DBG = 512 * 1024;
constexpr u32 CTL_BYTES = 1024 * 1024;
static_assert(OFF_FLG + 2 * FLG_BYTES <= OFF_DBG, "control region");
constexpr u32 IN_BYTES = 4096 * SLOT;
constexpr u32 OUT_BYTES = 192 * 16 * SLOT;
constexpr u32 REL_BYTES = 32 * 16 * SLOT;
constexpr u32 OFF_IN = CTL_BYTES;
constexpr u32 OFF_OUT = OFF_IN + 2 * IN_BYTES;
constexpr u32 OFF_REL = OFF_OUT + 2 * OUT_BYTES;
constexpr u32 OFF_OWN = OFF_REL + 2 * REL_BYTES;
constexpr u32 OFF_A = OFF_OWN + 64 * 4 * 16 * SLOT;
constexpr u32 OFF_B = OFF_A + 16 * 64 * 32768;
constexpr u32 WS_NEED = OFF_B + 16 * 64 * 32768;
static_assert(WS_NEED <= 257u * 1024u * 1024u, "workspace");
static_assert(OFF_A % 256 == 0 && OFF_OUT % 128 == 0 && OFF_OWN % 128 == 0, "alignment");

constexpr u32 META_RAW = 1u << 13, META_ZERO = 1u << 14;
#define GR_IN_WPL 18
#define GR_O_WPL 16
constexpr int IN_WPL = GR_IN_WPL, O_WPL = GR_O_WPL;
#define GR_GAUX 16
constexpr int AUX_SYS = 17, AUX_DEV = GR_GAUX;
constexpr int LDS_STAGE = 131072, LDS_CTL = LDS_STAGE, LDS_BYTES = LDS_STAGE + 256;

struct Params {
    const void* x;
    const void* w;
    void* out;
    void* ws[NR];
    int rank;
    int pad;
};

__device__ __forceinline__ u32 rfl(u32 v) { return (u32)__builtin_amdgcn_readfirstlane((int)v); }
__device__ __forceinline__ u32 pk_max(u32 a, u32 b) { u32 r; asm("v_pk_max_u16 %0, %1, %2" : "=v"(r) : "v"(a), "v"(b)); return r; }
__device__ __forceinline__ u32 pk_min(u32 a, u32 b) { u32 r; asm("v_pk_min_u16 %0, %1, %2" : "=v"(r) : "v"(a), "v"(b)); return r; }
__device__ __forceinline__ u32 pk_min_s(u32 a, u32 s) { u32 r; asm("v_pk_min_u16 %0, %1, %2" : "=v"(r) : "v"(a), "s"(s)); return r; }
__device__ __forceinline__ u32 pk_add_s(u32 a, u32 s) { u32 r; asm("v_pk_add_u16 %0, %1, %2" : "=v"(r) : "v"(a), "s"(s)); return r; }
__device__ __forceinline__ u32 pk_sub_s(u32 a, u32 s) { u32 r; asm("v_pk_sub_u16 %0, %1, %2" : "=v"(r) : "v"(a), "s"(s)); return r; }
__device__ __forceinline__ u32 pk_subsat_s(u32 a, u32 s) { u32 r; asm("v_pk_sub_u16 %0, %1, %2 clamp" : "=v"(r) : "v"(a), "s"(s)); return r; }
__device__ __forceinline__ u32 pk_mad_s(u32 a, u32 s, u32 c) { u32 r; asm("v_pk_mad_u16 %0, %1, %2, %3" : "=v"(r) : "v"(a), "s"(s), "v"(c)); return r; }
__device__ __forceinline__ u32 cvt_pk_bf16(float lo, float hi) { u32 r; asm volatile("v_cvt_pk_bf16_f32 %0, %1, %2" : "=v"(r) : "v"(lo), "v"(hi)); return r; }
__device__ __forceinline__ rsrc_t mk_rsrc(void* p) { return __builtin_amdgcn_make_buffer_rsrc(p, (short)0, (int)WS_NEED, 0x00020000); }
__device__ __forceinline__ void wait_vm0() { asm volatile("s_waitcnt vmcnt(0)" ::: "memory"); }
__device__ __forceinline__ void wait_vm0_known() { __builtin_amdgcn_s_waitcnt(0x0F70); asm volatile("" ::: "memory"); }

struct Ctx {
    char* ws;
    rsrc_t own;
    u32 call, par;
    int rank, lane;
};
#define DT0() do {} while (0)
#define DT(i) do {} while (0)
#define DDEP(d) do {} while (0)

__device__ __forceinline__ u64 flag_load(const Ctx& c, u32 foff) {
    return __hip_atomic_load((GAS const u64*)(c.ws + OFF_FLG + c.par * FLG_BYTES + foff), __ATOMIC_RELAXED, __HIP_MEMORY_SCOPE_SYSTEM);
}
__device__ __forceinline__ bool flag_test(const Ctx& c, u64 v, u32& meta) {
    meta = rfl((u32)v);
    return (int)(rfl((u32)(v >> 32)) - c.call) >= 0;
}
__device__ __forceinline__ bool wait_flag(const Ctx& c, u32 foff, u32& meta) {
    GAS u32* tmo = (GAS u32*)(c.ws + OFF_TMO);
    if (flag_test(c, flag_load(c, foff), meta)) return true;
    const u64 t0 = __builtin_amdgcn_s_memrealtime();
    for (;;) {
        __builtin_amdgcn_s_sleep(2);
        if (flag_test(c, flag_load(c, foff), meta)) return true;
        if (rfl(__hip_atomic_load(tmo, __ATOMIC_RELAXED, __HIP_MEMORY_SCOPE_AGENT)) == c.call || __builtin_amdgcn_s_memrealtime() - t0 > DEADLINE) {
            __hip_atomic_store(tmo, c.call, __ATOMIC_RELAXED, __HIP_MEMORY_SCOPE_AGENT);
            meta = 0; return false;
        }
    }
}
__device__ __forceinline__ void post_flag(const Params& P, const Ctx& c, int dst, u32 foff, u32 meta) {
    if ((unsigned)dst >= (unsigned)NR) return;
    GAS u64* f = (GAS u64*)((char*)P.ws[dst] + OFF_FLG + c.par * FLG_BYTES + foff);
    if (c.lane == 0) __hip_atomic_store(f, ((u64)c.call << 32) | meta, __ATOMIC_RELAXED, __HIP_MEMORY_SCOPE_SYSTEM);
}

__device__ __forceinline__ void store_raw(const u32x4 (&d)[8], rsrc_t rs, u32 slot, int lane) {
#pragma unroll
    for (int v = 0; v < 8; ++v) __builtin_amdgcn_raw_buffer_store_b128(d[v], rs, (int)(slot + v * 1024 + lane * 16), 0, AUX_SYS);
}
__device__ __forceinline__ u32 pack_store(const u32x4 (&d)[8], rsrc_t rs, u32 slot, int lane) {
    u32 mx = 0, mn = 0xFFFFFFFFu, mz = 0xFFFFFFFFu;
#pragma unroll
    for (int f = 0; f < 8; ++f)
#pragma unroll
        for (int e = 0; e < 4; ++e) { const u32 a = d[f][e] & 0x7FFF7FFFu; mx = pk_max(mx, a); mn = pk_min(mn, pk_sub_s(a, 0x00010001u)); mz = pk_min(mz, a); }
    const u32 emax = ((mx & 0xFFFFu) > (mx >> 16) ? (mx & 0xFFFFu) : (mx >> 16)) >> 7;
    u32 E = 0;
#pragma unroll
    for (int b = 7; b >= 0; --b) { const u32 cand = E | (1u << b); if (__builtin_amdgcn_ballot_w64(emax >= cand)) E = cand; }
    const u32 lo = E > 15 ? E - 15 : 0, L = lo << 7;
    const u32 mnl = (mn & 0xFFFFu) < (mn >> 16) ? (mn & 0xFFFFu) : (mn >> 16);
    const u32 zero = __builtin_amdgcn_ballot_w64((mz & 0xFFFFu) == 0 || (mz >> 16) == 0) ? META_ZERO : 0u;
    u32 cnt = 0, hdr = 0;
    if (__builtin_amdgcn_ballot_w64(mnl < L)) {
#pragma unroll
        for (int s = 0; s < 64; ++s) {
            if (cnt <= 15) {
                const u32 dw = d[s >> 3][(s >> 1) & 3], val = (s & 1) ? (dw >> 16) : (dw & 0xFFFFu);
                u64 b = __builtin_amdgcn_ballot_w64(((val & 0x7FFFu) - 1u) < L);
                while (b && cnt <= 15) {
                    const int l = __builtin_ctzll(b); b &= b - 1;
                    const u32 ent = ((u32)(l * 64 + s) << 16) | (u32)__builtin_amdgcn_readlane((int)val, l);
                    if ((u32)lane == cnt) hdr = ent;
                    ++cnt;
                }
            }
        }
        if (cnt > 15) { store_raw(d, rs, slot, lane); return META_RAW; }
        if (lane < 16) __builtin_amdgcn_raw_buffer_store_b32(hdr, rs, (int)(slot + HDR + lane * 4), 0, AUX_SYS);
    }
    const u32 LL = rfl(L | (L << 16));
    u32x4 V[6];
#pragma unroll
    for (int f = 0; f < 8; ++f) {
        u32 h[4];
#pragma unroll
        for (int e = 0; e < 4; ++e) { const u32 v = d[f][e]; h[e] = pk_subsat_s(v & 0x7FFF7FFFu, LL) | ((v >> 4) & 0x08000800u); }
        V[f >> 1][(f & 1) * 2 + 0] = __builtin_amdgcn_perm(h[1], h[0], 0x06040200u);
        V[f >> 1][(f & 1) * 2 + 1] = __builtin_amdgcn_perm(h[3], h[2], 0x06040200u);
        V[4 + (f >> 2)][f & 3] = __builtin_amdgcn_perm(h[1], h[0], 0x07050301u) | (__builtin_amdgcn_perm(h[3], h[2], 0x07050301u) << 4);
    }
#pragma unroll
    for (int v = 0; v < 6; ++v) __builtin_amdgcn_raw_buffer_store_b128(V[v], rs, (int)(slot + v * 1024 + lane * 16), 0, AUX_SYS);
    return lo | (cnt << 8) | zero;
}
struct Pull { u32x4 V[6]; u32 h; };
__device__ __forceinline__ void pull_issue(Pull& p, rsrc_t rs, u32 slot, int lane) {
#pragma unroll
    for (int v = 0; v < 6; ++v) p.V[v] = __builtin_amdgcn_raw_buffer_load_b128(rs, (int)(slot + v * 1024 + lane * 16), 0, AUX_SYS);
    p.h = __builtin_amdgcn_raw_buffer_load_b32(rs, (int)(slot + HDR + (lane & 15) * 4), 0, AUX_SYS);
}
__device__ __forceinline__ void unpack(u32x4 (&d)[8], const Pull& p, rsrc_t rs, u32 slot, u32 meta, int lane) {
    if (meta & META_RAW) {
#pragma unroll
        for (int v = 0; v < 6; ++v) d[v] = p.V[v];
        d[6] = __builtin_amdgcn_raw_buffer_load_b128(rs, (int)(slot + 6 * 1024 + lane * 16), 0, AUX_SYS);
        d[7] = __builtin_amdgcn_raw_buffer_load_b128(rs, (int)(slot + 7 * 1024 + lane * 16), 0, AUX_SYS);
        wait_vm0_known();
        return;
    }
    const u32 cnt = (meta >> 8) & 31u;
    const u32 L = (meta & 0xFFu) << 7, LL = L | (L << 16);
#pragma unroll
    for (int f = 0; f < 8; ++f) {
        const u32 B0 = p.V[f >> 1][(f & 1) * 2], B1 = p.V[f >> 1][(f & 1) * 2 + 1], N = p.V[4 + (f >> 2)][f & 3];
        const u32 N0 = (N & 0x07070707u) | ((N << 4) & 0x80808080u), N1 = (N & 0x80808080u) | ((N >> 4) & 0x07070707u);
        d[f][0] = __builtin_amdgcn_perm(N0, B0, 0x05010400u); d[f][1] = __builtin_amdgcn_perm(N0, B0, 0x07030602u);
        d[f][2] = __builtin_amdgcn_perm(N1, B1, 0x05010400u); d[f][3] = __builtin_amdgcn_perm(N1, B1, 0x07030602u);
    }
    if (meta & META_ZERO) {
#pragma unroll
        for (int q = 0; q < 32; ++q) { const u32 h = d[q >> 2][q & 3]; d[q >> 2][q & 3] = pk_mad_s(pk_min_s(h & 0x7FFF7FFFu, 0x00010001u), LL, h); }
    } else {
#pragma unroll
        for (int q = 0; q < 32; ++q) d[q >> 2][q & 3] = pk_add_s(d[q >> 2][q & 3], LL);
    }
    for (u32 i = 0; i < cnt; ++i) {
        const u32 ent = (u32)__builtin_amdgcn_readlane((int)p.h, (int)i);
        const u32 l = ent >> 22, s = (ent >> 16) & 63u, raw = ent & 0xFFFFu;
        const u32 keep = (s & 1) ? 0x0000FFFFu : 0xFFFF0000u, val = (s & 1) ? (raw << 16) : raw;
        const bool me = (u32)lane == l;
#pragma unroll
        for (int q = 0; q < 32; ++q) { const u32 o = d[q >> 2][q & 3]; d[q >> 2][q & 3] = (me && (s >> 1) == (u32)q) ? ((o & keep) | val) : o; }
    }
}
__device__ __forceinline__ void fill_nan(u32x4 (&d)[8]) {
#pragma unroll
    for (int v = 0; v < 8; ++v) d[v] = (u32x4){0x7FC07FC0u, 0x7FC07FC0u, 0x7FC07FC0u, 0x7FC07FC0u};
}
__device__ __forceinline__ void copy_store(const Ctx& c, const Pull& p, rsrc_t rs, u32 slot, u32 dst, u32 meta) {
    const int lane = c.lane;
    if (meta & META_RAW) {
        const u32x4 a = __builtin_amdgcn_raw_buffer_load_b128(rs, (int)(slot + 6 * 1024 + lane * 16), 0, AUX_SYS);
        const u32x4 b = __builtin_amdgcn_raw_buffer_load_b128(rs, (int)(slot + 7 * 1024 + lane * 16), 0, AUX_SYS);
        wait_vm0_known();
        __builtin_amdgcn_raw_buffer_store_b128(a, c.own, (int)(dst + 6 * 1024 + lane * 16), 0, AUX_SYS);
        __builtin_amdgcn_raw_buffer_store_b128(b, c.own, (int)(dst + 7 * 1024 + lane * 16), 0, AUX_SYS);
    } else if ((meta >> 8) & 31u) {
        if (lane < 16) __builtin_amdgcn_raw_buffer_store_b32(p.h, c.own, (int)(dst + HDR + lane * 4), 0, AUX_SYS);
    }
#pragma unroll
    for (int v = 0; v < 6; ++v) __builtin_amdgcn_raw_buffer_store_b128(p.V[v], c.own, (int)(dst + v * 1024 + lane * 16), 0, AUX_SYS);
}

struct Step { u32 foff, slot, meta; int sel, p0, p1, p2, p3; };
template <class Gen, class Act, class Pub>
__device__ __forceinline__ void pull_list(Ctx& c, int nsteps, rsrc_t rs0, rsrc_t rs1, Gen gen, Act act, Pub pub) {
    if (nsteps <= 0) return;
    DT0();
    Step cur = gen(0), prev = cur;
    bool ok = wait_flag(c, cur.foff, cur.meta), have_prev = false;
    DT(1);
    Pull pa, pb;
    pull_issue(pa, cur.sel ? rs1 : rs0, cur.slot, c.lane);
    int i = 0;
    auto half = [&](const Pull& p, Pull& pn) {
        const bool more = i + 1 < nsteps;
        Step nxt = gen(more ? i + 1 : i);
        const u64 fv = flag_load(c, nxt.foff);
        wait_vm0_known();
        DT(0);
        if (have_prev) pub(prev);
        const bool nready = more && flag_test(c, fv, nxt.meta);
        if (nready) pull_issue(pn, nxt.sel ? rs1 : rs0, nxt.slot, c.lane);
        act(cur, p, cur.sel ? rs1 : rs0, ok);
        prev = cur; have_prev = true; ok = true;
        DT(2);
        if (more && !nready) {
            wait_vm0_known(); pub(prev); have_prev = false;
            ok = wait_flag(c, nxt.foff, nxt.meta);
            pull_issue(pn, nxt.sel ? rs1 : rs0, nxt.slot, c.lane);
            DT(1);
        }
        cur = nxt; ++i;
    };
    for (;;) {
        half(pa, pb); if (i >= nsteps) break;
        half(pb, pa); if (i >= nsteps) break;
    }
    wait_vm0_known();
    if (have_prev) pub(prev);
    DT(3);
}

__device__ __forceinline__ int cq_of(int q, int r) {
    return ((r >> 1) ^ ((q & 1) ? (r >> 2) : 0) ^ ((q & 2) ? r : 0)) & 1;
}
__device__ __forceinline__ int link_d(int link) { return (0x765431 >> (4 * link)) & 7; }
__device__ __forceinline__ int link_q(int link, int i) { return ((i ? 0x233231 : 0x110020) >> (4 * link)) & 3; }
__device__ __forceinline__ int link_of_d(int d) { return d == 1 ? 0 : d - 2; }
__device__ __forceinline__ int t_of(int d, int q, int kq) {
    if (d == 0) return kq * 4 + q;
    const int link = link_of_d(d);
    return 16 + (2 * kq + (link_q(link, 0) == q ? 0 : 1)) * 6 + link;
}
__device__ __forceinline__ int d3_pair(int d) { return (d & 1) * 2 + ((d >> 1) & 1); }
__device__ __forceinline__ int d3_of_pair(int p) { return 4 + (p >> 1) + (p & 1) * 2; }
__device__ __forceinline__ int rowtile_of(int rc, int o, int a) {
    const int d = rc ^ o;
    return d == 0 ? 12 + a : d == 2 ? a : 4 + 2 * d3_pair(d) + (a & 1);
}
__device__ __forceinline__ void block_of_rowtile(int rc, int i, int& o, int& a) {
    if (i >= 12) { o = rc; a = i - 12; }
    else if (i < 4) { o = rc ^ 2; a = i; }
    else { const int p = (i - 4) >> 1; o = rc ^ d3_of_pair(p); a = 2 * (p >> 1) + ((i - 4) & 1); }
}
__device__ __forceinline__ u32 raw_frag_off(int type, int sub, int f) {
    return type == 0 ? (u32)(sub * 8192 + f * 1024) : (u32)((f >> 2) * 16384 + sub * 4096 + (f & 3) * 1024);
}
__device__ __forceinline__ void store_operand(const Ctx& c, const u32x4 (&d)[8], int type, int tile, int t, int sub) {
    const u32 base = (type == 0 ? OFF_A : OFF_B) + (u32)(tile * 64 + t) * 32768u + (u32)c.lane * 16u;
#pragma unroll
    for (int f = 0; f < 8; ++f) __builtin_amdgcn_raw_buffer_store_b128(d[f], c.own, (int)(base + raw_frag_off(type, sub, f)), 0, AUX_SYS);
}
__device__ __forceinline__ void count_operand(const Ctx& c, int type, int tile, int t) {
    GAS u32* p = (GAS u32*)(c.ws + (type == 0 ? OFF_RDYA : OFF_RDYB)) + tile * 64 + t;
    if (c.lane == 0) __hip_atomic_fetch_add(p, 1u, __ATOMIC_RELAXED, __HIP_MEMORY_SCOPE_AGENT);
}

__device__ __forceinline__ void pack_job(const Params& P, const Ctx& c, LAS unsigned char* tile, int pj) {
    const int kq = pj >> 10, q = pj & 3, type = (pj >> 2) & 1, g = (pj & 1023) >> 3, kt = 4 * q + kq, lane = c.lane, s = c.rank;
    u32x4 d[8];
    if (type == 0) {
        GAS const char* src = (GAS const char*)P.x + (size_t)(64 * g + (lane & 15)) * 2048 + (64 * kt + 8 * (lane >> 4)) * 2;
#pragma unroll
        for (int f = 0; f < 8; ++f) d[f] = *(GAS const u32x4*)(src + (f >> 1) * 16 * 2048 + (f & 1) * 64);
    } else {
        GAS const char* src = (GAS const char*)P.w + (size_t)(64 * kt + (lane >> 3)) * 16384 + (64 * g) * 2 + (lane & 7) * 16;
#pragma unroll
        for (int v = 0; v < 8; ++v)
            __builtin_amdgcn_global_load_lds((GAS const u32*)(src + (size_t)v * 8 * 16384), (LAS u32*)(tile + v * 1024), 16, 0, 0);
        wait_vm0();
        const int lg = lane >> 4, lq = (lane >> 2) & 3, lp = lane & 3;
        LAS unsigned char* b = tile + (8 * lg + lq) * 128 + lp * 16;
#pragma unroll
        for (int f = 0; f < 8; ++f) {
            const int o = (f & 1) * 32 * 128 + (f >> 2) * 64 + ((f >> 1) & 1) * 8;
            const s16x4 lo4 = __builtin_amdgcn_ds_read_tr16_b64_v4i16((LAS s16x4*)(b + o));
            const s16x4 hi4 = __builtin_amdgcn_ds_read_tr16_b64_v4i16((LAS s16x4*)(b + o + 4 * 128));
            typedef u32 u32x2 __attribute__((ext_vector_type(2)));
            const u32x2 l2 = __builtin_bit_cast(u32x2, lo4), h2 = __builtin_bit_cast(u32x2, hi4);
            d[f] = (u32x4){l2[0], l2[1], h2[0], h2[1]};
        }
        asm volatile("s_waitcnt lgkmcnt(0)" ::: "memory");
    }
    int rc[2], tl[2], tt[2];
    const int cs = cq_of(q, s);
#pragma unroll
    for (int i = 0; i < 2; ++i) {
        int e1, gam;
        if (type == 0) { const int o = g >> 4, h = (g >> 3) & 1; gam = i; e1 = h == 0 ? (o & 1) : ((o ^ (o >> 2) ^ gam) & 1); }
        else { gam = g >> 6; e1 = i; }
        const int e2 = (cs ^ ((q & 2) ? e1 : 0) ^ ((q & 1) ? gam : 0)) & 1;
        rc[i] = e1 | (e2 << 1) | (gam << 2);
        tl[i] = type == 0 ? rowtile_of(rc[i], g >> 4, (g >> 2) & 3) : ((g >> 2) & 15);
        tt[i] = t_of(rc[i] ^ s, q, kq);
    }
    const int sub = g & 3;
    const u32 meta = pack_store(d, c.own, OFF_IN + c.par * IN_BYTES + (u32)(type * 2048 + kt * 128 + g) * SLOT, lane);
    if (rc[0] == s) store_operand(c, d, type, tl[0], tt[0], sub);
    if (rc[1] == s) store_operand(c, d, type, tl[1], tt[1], sub);
    wait_vm0();
#pragma unroll
    for (int i = 0; i < 2; ++i) {
        if (rc[i] == s) count_operand(c, type, tl[i], tt[i]);
        else post_flag(P, c, rc[i], F_IN + (u32)(((type * 16 + tl[i]) * 64 + tt[i]) * 4 + sub) * 8u, meta);
    }
}

__device__ __forceinline__ void in_list(const Params& P, Ctx& c, int link, int widx, int wpl) {
    const int s = c.rank ^ link_d(link);
    auto gen = [&](int i) {
        const int j = widx + wpl * i;
        int kc, type, tile, sub;
        if (j < 896) { kc = j / 112; const int u = j % 112; if (u < 48) { type = 0; tile = u >> 2; } else { type = 1; tile = (u - 48) >> 2; } sub = u & 3; }
        else { const int m = j - 896; kc = m >> 4; type = 0; tile = 12 + ((m >> 2) & 3); sub = m & 3; }
        const int t = 16 + kc * 6 + link, kt = 4 * link_q(link, kc & 1) + (kc >> 1);
        int g;
        if (type == 0) { int o, a; block_of_rowtile(c.rank, tile, o, a); g = o * 16 + a * 4 + sub; } else g = (c.rank >> 2) * 64 + tile * 4 + sub;
        Step st; st.foff = F_IN + (u32)(((type * 16 + tile) * 64 + t) * 4 + sub) * 8u;
        st.slot = OFF_IN + c.par * IN_BYTES + (u32)(type * 2048 + kt * 128 + g) * SLOT;
        st.sel = 0; st.meta = 0; st.p0 = type; st.p1 = tile; st.p2 = t; st.p3 = sub;
        return st;
    };
    auto act = [&](Step& st, const Pull& p, rsrc_t rs, bool ok) {
        u32x4 d[8];
        if (ok) unpack(d, p, rs, st.slot, st.meta, c.lane); else fill_nan(d);
        store_operand(c, d, st.p0, st.p1, st.p2, st.p3);
    };
    auto pub = [&](const Step& st) { count_operand(c, st.p0, st.p1, st.p2); };
    const rsrc_t rs = mk_rsrc(P.ws[s & 7]);
    pull_list(c, (1024 - widx + wpl - 1) / wpl, rs, rs, gen, act, pub);
}

__device__ __forceinline__ void acc_add(float (&acc)[64], const u32x4 (&d)[8]) {
#pragma unroll
    for (int q = 0; q < 32; ++q) { const u32 v = d[q >> 2][q & 3]; acc[2 * q] += __builtin_bit_cast(float, v << 16); acc[2 * q + 1] += __builtin_bit_cast(float, v & 0xFFFF0000u); }
}
__device__ __forceinline__ void write_out(const Params& P, const Ctx& c, const float (&acc)[64], int a, int cj, int jb) {
    const int w = jb >> 1, ai = jb & 1, lane = c.lane;
    const bool bad = rfl(__hip_atomic_load((GAS u32*)(c.ws + OFF_TMO), __ATOMIC_RELAXED, __HIP_MEMORY_SCOPE_AGENT)) == c.call;
    GAS char* o = (GAS char*)P.out + (size_t)(a * 256 + ai * 128 + (w >> 2) * 64 + (lane & 15)) * 16384 + (cj * 256 + (w & 3) * 64 + 8 * (lane >> 4)) * 2;
#pragma unroll
    for (int f = 0; f < 8; ++f) {
        u32x4 v;
#pragma unroll
        for (int e = 0; e < 4; ++e) v[e] = bad ? 0x7FC07FC0u : cvt_pk_bf16(acc[f * 8 + 2 * e], acc[f * 8 + 2 * e + 1]);
        *(GAS u32x4*)(o + (size_t)(f >> 1) * 16 * 16384 + (f & 1) * 64) = v;
    }
}
__device__ __forceinline__ void copy_list(const Params& P, Ctx& c, int grp, int widx, int wpl) {
    const int r = c.rank;
    auto gen = [&](int i) {
        const int j = widx + wpl * i, jb = j & 15;
        Step st; st.meta = 0;
        if (grp == 0) {
            const int pt = 2 * (j >> 4);
            st.foff = F_P + (u32)(pt * 16 + jb) * 8u; st.slot = OFF_OUT + c.par * OUT_BYTES + (u32)(pt * 16 + jb) * SLOT;
            st.p0 = (int)(OFF_OWN + (u32)((pt * 4 + 3) * 16 + jb) * SLOT); st.p1 = r; st.p2 = (int)(F_OWN + (u32)((pt * 4 + 3) * 16 + jb) * 8u);
        } else {
            const int second = j >= 256, u = (j >> 4) & 15, path = (grp - 1) ^ second, pt = 4 * u + 1 + 2 * path, rs = path * 16 + u;
            if (!second) {
                st.foff = F_R + (u32)(rs * 16 + jb) * 8u; st.slot = OFF_OUT + c.par * OUT_BYTES + (u32)(pt * 16 + jb) * SLOT;
                st.p0 = (int)(OFF_REL + c.par * REL_BYTES + (u32)(rs * 16 + jb) * SLOT); st.p1 = r ^ (path ? 1 : 3); st.p2 = (int)(F_P + (u32)(pt * 16 + jb) * 8u);
            } else {
                st.foff = F_P + (u32)(pt * 16 + jb) * 8u; st.slot = OFF_REL + c.par * REL_BYTES + (u32)(rs * 16 + jb) * SLOT;
                st.p0 = (int)(OFF_OWN + (u32)((pt * 4 + 3) * 16 + jb) * SLOT); st.p1 = r; st.p2 = (int)(F_OWN + (u32)((pt * 4 + 3) * 16 + jb) * 8u);
            }
        }
        st.p3 = 0; st.sel = 0;
        return st;
    };
    auto act = [&](Step& st, const Pull& p, rsrc_t rs, bool ok) {
        if (ok) copy_store(c, p, rs, st.slot, (u32)st.p0, st.meta);
        else { u32x4 d[8]; fill_nan(d); store_raw(d, c.own, (u32)st.p0, c.lane); st.meta = META_RAW; }
    };
    auto pub = [&](const Step& st) { post_flag(P, c, st.p1, (u32)st.p2, st.meta); };
    const rsrc_t rs = mk_rsrc(P.ws[(r ^ (grp == 0 ? 2 : grp == 1 ? 1 : 3)) & 7]);
    pull_list(c, (512 - widx + wpl - 1) / wpl, rs, rs, gen, act, pub);
}
__device__ __forceinline__ void reduce_list(const Params& P, Ctx& c, int h, int widx, int wpl) {
    const int r = c.rank;
    float acc[64];
    auto gen = [&](int i) {
        const int job = widx + wpl * (i >> 1), sl = (i ^ widx) & 1, jb = job & 15, ot2 = job >> 4;
        const int a = 2 * h + (ot2 & 1), j = ot2 >> 1, ot = a * 16 + j, p = 2 * h + sl, ti = 4 + 2 * p + (a & 1);
        Step st; st.meta = 0;
        st.foff = F_D3 + (u32)((ot * 2 + sl) * 16 + jb) * 8u; st.sel = sl;
        st.slot = OFF_OUT + c.par * OUT_BYTES + (u32)((ti * 16 + j) * 16 + jb) * SLOT;
        st.p0 = i & 1; st.p1 = a; st.p2 = (1 - (r >> 2)) * 16 + j; st.p3 = jb;
        return st;
    };
    auto act = [&](Step& st, const Pull& p, rsrc_t rs, bool ok) {
        u32x4 d[8];
        if (ok) unpack(d, p, rs, st.slot, st.meta, c.lane); else fill_nan(d);
        if (st.p0 == 0) {
#pragma unroll
            for (int q = 0; q < 64; ++q) acc[q] = 0.f;
        }
        acc_add(acc, d);
        if (st.p0 == 1) write_out(P, c, acc, st.p1, st.p2, st.p3);
    };
    auto pub = [&](const Step&) {};
    pull_list(c, 2 * ((512 - widx + wpl - 1) / wpl), mk_rsrc(P.ws[(r ^ d3_of_pair(2 * h)) & 7]), mk_rsrc(P.ws[(r ^ d3_of_pair(2 * h + 1)) & 7]), gen, act, pub);
}
__device__ __forceinline__ void own_job(const Params& P, const Ctx& c, int n) {
    const int pt = n >> 4, jb = n & 15;
    float acc[64];
#pragma unroll
    for (int i = 0; i < 64; ++i) acc[i] = 0.f;
    for (int part = 0; part < 4; ++part) {
        u32 meta; u32x4 d[8];
        if (wait_flag(c, F_OWN + (u32)((pt * 4 + part) * 16 + jb) * 8u, meta)) {
            Pull p; const u32 slot = OFF_OWN + (u32)((pt * 4 + part) * 16 + jb) * SLOT;
            pull_issue(p, c.own, slot, c.lane); unpack(d, p, c.own, slot, meta, c.lane);
        } else fill_nan(d);
        acc_add(acc, d);
    }
    write_out(P, c, acc, pt >> 4, (c.rank >> 2) * 16 + (pt & 15), jb);
}

struct GJob { int ti, tj, t0, nt; };

#define STAMP(i) do {} while (0)

__device__ __forceinline__ void gemm_role(const Params& P, const Ctx& c, LAS unsigned char* lds, int xcd, int idx) {
    const int tid = threadIdx.x, wid = (int)rfl(tid >> 6), lane = c.lane, wr = wid >> 2, wc = wid & 3;
    constexpr int HTB = 16384;
    const u32 voff = (u32)tid * 16u;
    const unsigned ldsw = (unsigned)wid * 1024u;
    const int aoff = wr * 8192 + lane * 16, boff = wc * 4096 + lane * 16;
    LAS volatile int* ctl = (LAS volatile int*)(lds + LDS_CTL);
    int avail = 0; bool bad = false;
#define G_SA(b, h) (((b) * 2 + (h)) * HTB)
#define G_SB(b, h) ((4 + (b) * 2 + (h)) * HTB)
#define G_STAGE(bufoff, gbase) do { _Pragma("unroll") for (int _i = 0; _i < 2; ++_i) \
        __builtin_amdgcn_global_load_lds((GAS const u32*)((GAS const char*)(gbase) + voff + _i * 8192), (LAS u32*)(lds + (bufoff) + ldsw + _i * 8192), 16, 0, AUX_DEV); } while (0)
#define G_LDA(dst, b, h) do { _Pragma("unroll") for (int m = 0; m < 4; ++m) _Pragma("unroll") for (int k = 0; k < 2; ++k) dst[m][k] = *(const LAS bf16x8*)(lds + G_SA(b, h) + aoff + m * 2048 + k * 1024); } while (0)
#define G_LDB(dst, b, h) do { _Pragma("unroll") for (int n = 0; n < 2; ++n) _Pragma("unroll") for (int k = 0; k < 2; ++k) dst[n][k] = *(const LAS bf16x8*)(lds + G_SB(b, h) + boff + n * 2048 + k * 1024); } while (0)
#define G_MMA(ai, bj, At, Bt) do { __builtin_amdgcn_s_setprio(1); _Pragma("unroll") for (int m = 0; m < 4; ++m) _Pragma("unroll") for (int n = 0; n < 2; ++n) _Pragma("unroll") for (int k = 0; k < 2; ++k) \
        acc[ai][bj][m][n] = __builtin_amdgcn_mfma_f32_16x16x32_bf16(Bt[n][k], At[m][k], acc[ai][bj][m][n], 0, 0, 0); __builtin_amdgcn_s_setprio(0); } while (0)
#define G_WAIT_V(n) asm volatile("s_waitcnt vmcnt(" #n ")" ::: "memory")
#define G_WAIT_L(n) asm volatile("s_waitcnt lgkmcnt(" #n ")" ::: "memory")
#define G_BAR __builtin_amdgcn_s_barrier()
#define G_SCHED __builtin_amdgcn_sched_barrier(0)
    auto ensure = [&](const GJob& j, int tneed) {
        if (avail >= tneed) return;
        if (wid == 0) {
            GAS const u32* ra = (GAS const u32*)(c.ws + OFF_RDYA) + j.ti * 64 + lane;
            GAS const u32* rb = (GAS const u32*)(c.ws + OFF_RDYB) + j.tj * 64 + lane;
            GAS u32* tmo = (GAS u32*)(c.ws + OFF_TMO);
            const u32 target = 4u * c.call;
            const u64 below = j.t0 ? ((1ull << j.t0) - 1ull) : 0ull;
            const u64 t0 = __builtin_amdgcn_s_memrealtime();
            int av, bd = 0;
            for (;;) {
                const u32 a = __hip_atomic_load(ra, __ATOMIC_RELAXED, __HIP_MEMORY_SCOPE_AGENT), b = __hip_atomic_load(rb, __ATOMIC_RELAXED, __HIP_MEMORY_SCOPE_AGENT);
                const u64 ok = __builtin_amdgcn_ballot_w64((int)(a - target) >= 0 && (int)(b - target) >= 0) | below;
                av = ok == ~0ull ? 64 : __builtin_ctzll(~ok);
                if (av >= tneed) break;
                if (rfl(__hip_atomic_load(tmo, __ATOMIC_RELAXED, __HIP_MEMORY_SCOPE_AGENT)) == c.call || __builtin_amdgcn_s_memrealtime() - t0 > DEADLINE) {
                    __hip_atomic_store(tmo, c.call, __ATOMIC_RELAXED, __HIP_MEMORY_SCOPE_AGENT); av = 64; bd = 1; break;
                }
                __builtin_amdgcn_s_sleep(4);
            }
            if (lane == 0) { ctl[0] = av; if (bd) ctl[1] = 1; }
            G_WAIT_L(0);
        }
        asm volatile("" ::: "memory"); G_BAR; asm volatile("" ::: "memory");
        avail = (int)rfl((u32)ctl[0]); bad = rfl((u32)ctl[1]) != 0;
    };
    GJob cur, nxt;
    cur.ti = 6 * (xcd & 1) + idx % 6; cur.tj = 4 * (xcd >> 1) + idx / 6; cur.t0 = 0; cur.nt = 64;
    { const int third = idx >> 3; nxt.ti = 12 + (idx & 3); nxt.tj = 2 * xcd + ((idx >> 2) & 1); nxt.t0 = 22 * third; nxt.nt = third == 2 ? 20 : 22; }
    f32x4 acc[2][2][4][2];
#pragma unroll
    for (int a = 0; a < 2; ++a)
#pragma unroll
        for (int b = 0; b < 2; ++b)
#pragma unroll
            for (int m = 0; m < 4; ++m)
#pragma unroll
                for (int n = 0; n < 2; ++n) acc[a][b][m][n] = (f32x4){0.f, 0.f, 0.f, 0.f};
    bf16x8 At[4][2], B0[2][2], B1[2][2];
    constexpr size_t kstep = 32768, hstep = 16384;
    const char* cA = c.ws + OFF_A + (size_t)(cur.ti * 64 + cur.t0) * kstep;
    const char* cB = c.ws + OFF_B + (size_t)(cur.tj * 64 + cur.t0) * kstep;
    ensure(cur, cur.t0 + 2);
    G_STAGE(G_SB(0, 0), cB); G_STAGE(G_SA(0, 0), cA); G_STAGE(G_SB(0, 1), cB + hstep); G_STAGE(G_SA(0, 1), cA + hstep);
    if (wr == 1) G_BAR;
    G_WAIT_V(4); G_BAR;
    G_STAGE(G_SB(1, 0), cB + kstep); G_STAGE(G_SA(1, 0), cA + kstep); G_STAGE(G_SB(1, 1), cB + hstep + kstep);
    G_WAIT_V(6); G_BAR;
    STAMP(2);
    for (int ui = 0; ui < 2; ++ui) {
        const bool has_next = ui == 0;
        const char* nA = has_next ? c.ws + OFF_A + (size_t)(nxt.ti * 64 + nxt.t0) * kstep : cA;
        const char* nB = has_next ? c.ws + OFF_B + (size_t)(nxt.tj * 64 + nxt.t0) * kstep : cB;
        const int nt = cur.nt;
        for (int t = 0; t < nt; t += 2) {
            const bool last = (t == nt - 2);
            const char* a1 = cA + (size_t)(t + 1) * kstep;
            const char* a2 = last ? nA : cA + (size_t)(t + 2) * kstep; const char* b2 = last ? nB : cB + (size_t)(t + 2) * kstep;
            const char* a3 = a2 + kstep; const char* b3 = b2 + kstep;
            if (!last) ensure(cur, cur.t0 + t + 4);
            else if (has_next) { avail = 0; ensure(nxt, nxt.t0 + 2); }
            G_LDB(B0, 0, 0); G_SCHED; G_LDA(At, 0, 0); G_STAGE(G_SA(1, 1), a1 + hstep);
            G_WAIT_L(8); G_BAR; G_WAIT_L(0); G_MMA(0, 0, At, B0); G_BAR; G_SCHED;
            G_LDB(B1, 0, 1); G_STAGE(G_SB(0, 0), b2);
            G_BAR; G_WAIT_L(0); G_MMA(0, 1, At, B1); G_BAR;
            G_LDA(At, 0, 1); G_STAGE(G_SA(0, 0), a2);
            G_BAR; G_WAIT_L(0); G_MMA(1, 0, At, B0); G_BAR; G_SCHED;
            G_STAGE(G_SB(0, 1), b2 + hstep);
            G_WAIT_V(6); G_BAR; G_MMA(1, 1, At, B1); G_BAR;
            G_LDB(B0, 1, 0); G_SCHED; G_LDA(At, 1, 0); G_STAGE(G_SA(0, 1), a2 + hstep);
            G_WAIT_L(8); G_BAR; G_WAIT_L(0); G_MMA(0, 0, At, B0); G_BAR; G_SCHED;
            G_LDB(B1, 1, 1); G_STAGE(G_SB(1, 0), b3);
            G_BAR; G_WAIT_L(0); G_MMA(0, 1, At, B1); G_BAR;
            G_LDA(At, 1, 1); G_STAGE(G_SA(1, 0), a3);
            G_BAR; G_WAIT_L(0); G_MMA(1, 0, At, B0); G_BAR; G_SCHED;
            G_STAGE(G_SB(1, 1), b3 + hstep);
            G_WAIT_V(6); G_BAR; G_MMA(1, 1, At, B1); G_BAR;
        }
        STAMP(3 + 2 * ui);
        {
            u32 meta[2], slot[2];
#pragma unroll
            for (int ai = 0; ai < 2; ++ai) {
                u32x4 d[8];
#pragma unroll
                for (int m = 0; m < 4; ++m)
#pragma unroll
                    for (int bj = 0; bj < 2; ++bj) {
                        const f32x4 v0 = acc[ai][bj][m][0], v1 = acc[ai][bj][m][1];
                        d[m * 2 + bj] = (u32x4){cvt_pk_bf16(v0[0], v0[1]), cvt_pk_bf16(v0[2], v0[3]), cvt_pk_bf16(v1[0], v1[1]), cvt_pk_bf16(v1[2], v1[3])};
                    }
                if (bad) fill_nan(d);
                const int jb = wid * 2 + ai;
                if (has_next) { slot[ai] = OFF_OUT + c.par * OUT_BYTES + (u32)((cur.ti * 16 + cur.tj) * 16 + jb) * SLOT; meta[ai] = pack_store(d, c.own, slot[ai], lane); }
                else { slot[ai] = OFF_OWN + (u32)((((cur.ti - 12) * 16 + cur.tj) * 4 + (cur.t0 != 0) + (cur.t0 == 44)) * 16 + jb) * SLOT; store_raw(d, c.own, slot[ai], lane); meta[ai] = META_RAW; }
            }
            wait_vm0();
#pragma unroll
            for (int ai = 0; ai < 2; ++ai) {
                const int jb = wid * 2 + ai, r = c.rank;
                int dst; u32 foff;
                if (!has_next) { dst = r; foff = F_OWN + (u32)((((cur.ti - 12) * 16 + cur.tj) * 4 + (cur.t0 != 0) + (cur.t0 == 44)) * 16 + jb) * 8u; }
                else if (cur.ti < 4) {
                    const int pt = cur.ti * 16 + cur.tj;
                    if ((pt & 1) == 0) { dst = r ^ 2; foff = F_P + (u32)(pt * 16 + jb) * 8u; }
                    else { const int path = (pt >> 1) & 1; dst = r ^ (path ? 3 : 1); foff = F_R + (u32)((path * 16 + (pt >> 2)) * 16 + jb) * 8u; }
                } else {
                    const int p = (cur.ti - 4) >> 1, a = 2 * (p >> 1) + ((cur.ti - 4) & 1);
                    dst = r ^ d3_of_pair(p); foff = F_D3 + (u32)(((a * 16 + cur.tj) * 2 + (p & 1)) * 16 + jb) * 8u;
                }
                post_flag(P, c, dst, foff, meta[ai]);
            }
        }
        STAMP(4 + 2 * ui);
        if (!has_next) break;
#pragma unroll
        for (int a = 0; a < 2; ++a)
#pragma unroll
            for (int b = 0; b < 2; ++b)
#pragma unroll
                for (int m = 0; m < 4; ++m)
#pragma unroll
                    for (int n = 0; n < 2; ++n) acc[a][b][m][n] = (f32x4){0.f, 0.f, 0.f, 0.f};
        cur = nxt; cA = nA; cB = nB;
    }
    G_WAIT_V(0);
    if (wr == 0) G_BAR;
    G_BAR;
#undef G_SA
#undef G_SB
#undef G_STAGE
#undef G_LDA
#undef G_LDB
#undef G_MMA
#undef G_WAIT_V
#undef G_WAIT_L
#undef G_BAR
#undef G_SCHED
}

__global__ __launch_bounds__(NTH, 2) void gemm_rs_kernel(const Params P) {
    extern __shared__ __attribute__((aligned(16))) unsigned char shm[];
    LAS unsigned char* lds = (LAS unsigned char*)shm;
    const int wg = blockIdx.x, xcd = wg & 7, idx = wg >> 3, tid = threadIdx.x, wid = (int)rfl(tid >> 6);
    Ctx c;
    c.rank = P.rank & 7; c.lane = tid & 63;
    c.ws = (char*)P.ws[c.rank]; c.own = mk_rsrc(c.ws);
    LAS volatile u32* ctl = (LAS volatile u32*)(lds + LDS_CTL);
    if (tid == 0) {
        ctl[2] = __hip_atomic_fetch_add((GAS u32*)(c.ws + OFF_CALL + wg * 64), 1u, __ATOMIC_RELAXED, __HIP_MEMORY_SCOPE_AGENT) + 1u;
        ctl[0] = 0; ctl[1] = 0;
    }
    __syncthreads();
    c.call = rfl(ctl[2]); c.par = c.call & 1u;
    STAMP(0);
    if (idx < NGEMM / 8) {
        const int gw = (xcd * (NGEMM / 8) + idx) * 8 + wid;
        for (int pj = gw; pj < 4096; pj += GEMM_WAVES) pack_job(P, c, lds + wid * 8192, pj);
        STAMP(1);
        __syncthreads();
        gemm_role(P, c, lds, xcd, idx);
        if (gw < 1024) own_job(P, c, gw);
        STAMP(7);
    } else {
        const int cw = wid * NCOMM + (idx - NGEMM / 8) * 8 + xcd;
        if (cw >= 6 * IN_WPL && cw >= 7 * O_WPL) return;
        if (cw < 6 * IN_WPL) in_list(P, c, cw % 6, cw / 6, IN_WPL);
        STAMP(3);
        if (cw < 7 * O_WPL) {
            const int grp = cw % 7, widx = cw / 7;
            if (grp < 3) copy_list(P, c, grp, widx, O_WPL);
            else reduce_list(P, c, (grp - 3) >> 1, widx * 2 + ((grp - 3) & 1), 2 * O_WPL);
        }
        STAMP(7);
    }
}
}

extern "C" void kernel_launch(int rank, int nranks, ncclComm_t comm, hipStream_t stream,
                              void* const* d_in, const int* in_sizes, int n_in,
                              void* d_out, int out_size,
                              const int* in_dtypes, int out_dtype,
                              void* const* d_out_all,
                              void* const* d_ws_all, size_t ws_bytes) {
    (void)comm; (void)in_sizes; (void)n_in; (void)out_size; (void)in_dtypes; (void)out_dtype; (void)d_out_all;
    if (nranks != gr::NR || rank < 0 || rank >= gr::NR || ws_bytes < gr::WS_NEED) { fprintf(stderr, "kernel_launch: unexpected configuration\n"); return; }
    static bool init[gr::NR] = {};
    if (!init[rank]) {
        if (hipFuncSetAttribute((const void*)gr::gemm_rs_kernel, hipFuncAttributeMaxDynamicSharedMemorySize, gr::LDS_BYTES) != hipSuccess) { fprintf(stderr, "hipFuncSetAttribute failed\n"); return; }
        init[rank] = true;
    }
    gr::Params P;
    P.x = d_in[0]; P.w = d_in[1]; P.out = d_out; P.rank = rank; P.pad = 0;
    for (int i = 0; i < gr::NR; ++i) P.ws[i] = d_ws_all[i];
    hipLaunchKernelGGL(gr::gemm_rs_kernel, dim3(gr::NWG), dim3(gr::NTH), gr::LDS_BYTES, stream, P);
    hipError_t e = hipGetLastError();
    if (e != hipSuccess) fprintf(stderr, "launch failed: %s\n", hipGetErrorString(e));
}
```

```cpp
#include <hip/hip_runtime.h>
#include <rccl/rccl.h>
#include <cstdio>

namespace gr {
typedef unsigned u32;
typedef unsigned long long u64;
typedef unsigned short u16;
typedef u32 u32x4 __attribute__((ext_vector_type(4)));
typedef u16 u16x2 __attribute__((ext_vector_type(2)));
typedef short bf16x8 __attribute__((ext_vector_type(8)));
typedef short s16x4 __attribute__((ext_vector_type(4)));
typedef float f32x4 __attribute__((ext_vector_type(4)));
#define LAS __attribute__((address_space(3)))
#define GAS __attribute__((address_space(1)))
typedef __amdgpu_buffer_rsrc_t rsrc_t;

constexpr int NR = 8;
constexpr int NWG = 256, NTH = 512, NGEMM = 192, NCOMM = NWG - NGEMM;
constexpr int GEMM_WAVES = NGEMM * 8, COMM_WAVES = NCOMM * 8;
constexpr u32 SLOT = 8448;
constexpr u32 HDR = 8192;
constexpr u64 DEADLINE = 100000000ull;

constexpr u32 OFF_CALL = 0;
constexpr u32 OFF_TMO = 16384;
constexpr u32 OFF_RDYA = 20480;
constexpr u32 OFF_RDYB = 24576;
constexpr u32 OFF_TKT = 28672;
constexpr u32 OFF_FLG = 32768;
constexpr u32 F_IN = 0;
constexpr u32 F_D3 = F_IN + 2 * 16 * 64 * 4 * 8;
constexpr u32 F_P = F_D3 + 64 * 2 * 16 * 8;
constexpr u32 F_R = F_P + 64 * 16 * 8;
constexpr u32 F_OWN = F_R + 32 * 16 * 8;
constexpr u32 FLG_BYTES = F_OWN + 64 * 4 * 16 * 8;
constexpr u32 OFF_DBG = 512 * 1024;
constexpr u32 CTL_BYTES = 1024 * 1024;
static_assert(OFF_FLG + 2 * FLG_BYTES <= OFF_DBG, "control region");
constexpr u32 IN_BYTES = 4096 * SLOT;
constexpr u32 OUT_BYTES = 192 * 16 * SLOT;
constexpr u32 REL_BYTES = 32 * 16 * SLOT;
constexpr u32 OFF_IN = CTL_BYTES;
constexpr u32 OFF_OUT = OFF_IN + 2 * IN_BYTES;
constexpr u32 OFF_REL = OFF_OUT + 2 * OUT_BYTES;
constexpr u32 OFF_OWN = OFF_REL + 2 * REL_BYTES;
constexpr u32 OFF_A = OFF_OWN + 64 * 4 * 16 * SLOT;
constexpr u32 OFF_B = OFF_A + 16 * 64 * 32768;
constexpr u32 WS_NEED = OFF_B + 16 * 64 * 32768;
static_assert(WS_NEED <= 257u * 1024u * 1024u, "workspace");
static_assert(OFF_A % 256 == 0 && OFF_OUT % 128 == 0 && OFF_OWN % 128 == 0, "alignment");

constexpr u32 META_RAW = 1u << 13, META_ZERO = 1u << 14;
#define GR_IN_WPL 16
#define GR_O_WPL 15
constexpr int IN_WPL = GR_IN_WPL, O_WPL = GR_O_WPL;
#define GR_GAUX 16
constexpr int AUX_SYS = 17, AUX_DEV = GR_GAUX;
constexpr int LDS_STAGE = 131072, LDS_CTL = LDS_STAGE, LDS_BYTES = LDS_STAGE + 256;

struct Params {
    const void* x;
    const void* w;
    void* out;
    void* ws[NR];
    int rank;
    int pad;
};

__device__ __forceinline__ u32 rfl(u32 v) { return (u32)__builtin_amdgcn_readfirstlane((int)v); }
__device__ __forceinline__ u32 pk_max(u32 a, u32 b) { u32 r; asm("v_pk_max_u16 %0, %1, %2" : "=v"(r) : "v"(a), "v"(b)); return r; }
__device__ __forceinline__ u32 pk_min(u32 a, u32 b) { u32 r; asm("v_pk_min_u16 %0, %1, %2" : "=v"(r) : "v"(a), "v"(b)); return r; }
__device__ __forceinline__ u32 pk_min_s(u32 a, u32 s) { u32 r; asm("v_pk_min_u16 %0, %1, %2" : "=v"(r) : "v"(a), "s"(s)); return r; }
__device__ __forceinline__ u32 pk_add_s(u32 a, u32 s) { u32 r; asm("v_pk_add_u16 %0, %1, %2" : "=v"(r) : "v"(a), "s"(s)); return r; }
__device__ __forceinline__ u32 pk_sub_s(u32 a, u32 s) { u32 r; asm("v_pk_sub_u16 %0, %1, %2" : "=v"(r) : "v"(a), "s"(s)); return r; }
__device__ __forceinline__ u32 pk_subsat_s(u32 a, u32 s) { u32 r; asm("v_pk_sub_u16 %0, %1, %2 clamp" : "=v"(r) : "v"(a), "s"(s)); return r; }
__device__ __forceinline__ u32 pk_mad_s(u32 a, u32 s, u32 c) { u32 r; asm("v_pk_mad_u16 %0, %1, %2, %3" : "=v"(r) : "v"(a), "s"(s), "v"(c)); return r; }
__device__ __forceinline__ u32 cvt_pk_bf16(float lo, float hi) { u32 r; asm volatile("v_cvt_pk_bf16_f32 %0, %1, %2" : "=v"(r) : "v"(lo), "v"(hi)); return r; }
__device__ __forceinline__ rsrc_t mk_rsrc(void* p) { return __builtin_amdgcn_make_buffer_rsrc(p, (short)0, (int)WS_NEED, 0x00020000); }
__device__ __forceinline__ void wait_vm0() { asm volatile("s_waitcnt vmcnt(0)" ::: "memory"); }
__device__ __forceinline__ void wait_vm0_known() { __builtin_amdgcn_s_waitcnt(0x0F70); asm volatile("" ::: "memory"); }

struct Ctx {
    char* ws;
    rsrc_t own;
    u32 call, par;
    int rank, lane;
};
#define DT0() do {} while (0)
#define DT(i) do {} while (0)
#define DDEP(d) do {} while (0)

__device__ __forceinline__ u64 flag_load(const Ctx& c, u32 foff) {
    return __hip_atomic_load((GAS const u64*)(c.ws + OFF_FLG + c.par * FLG_BYTES + foff), __ATOMIC_RELAXED, __HIP_MEMORY_SCOPE_SYSTEM);
}
__device__ __forceinline__ bool flag_test(const Ctx& c, u64 v, u32& meta) {
    meta = rfl((u32)v);
    return (int)(rfl((u32)(v >> 32)) - c.call) >= 0;
}
__device__ __forceinline__ bool wait_flag(const Ctx& c, u32 foff, u32& meta) {
    GAS u32* tmo = (GAS u32*)(c.ws + OFF_TMO);
    if (flag_test(c, flag_load(c, foff), meta)) return true;
    const u64 t0 = __builtin_amdgcn_s_memrealtime();
    for (;;) {
        __builtin_amdgcn_s_sleep(2);
        if (flag_test(c, flag_load(c, foff), meta)) return true;
        if (rfl(__hip_atomic_load(tmo, __ATOMIC_RELAXED, __HIP_MEMORY_SCOPE_AGENT)) == c.call || __builtin_amdgcn_s_memrealtime() - t0 > DEADLINE) {
            __hip_atomic_store(tmo, c.call, __ATOMIC_RELAXED, __HIP_MEMORY_SCOPE_AGENT);
            meta = 0; return false;
        }
    }
}
__device__ __forceinline__ void post_flag(const Params& P, const Ctx& c, int dst, u32 foff, u32 meta) {
    if ((unsigned)dst >= (unsigned)NR) return;
    GAS u64* f = (GAS u64*)((char*)P.ws[dst] + OFF_FLG + c.par * FLG_BYTES + foff);
    if (c.lane == 0) __hip_atomic_store(f, ((u64)c.call << 32) | meta, __ATOMIC_RELAXED, __HIP_MEMORY_SCOPE_SYSTEM);
}

__device__ __forceinline__ void store_raw(const u32x4 (&d)[8], rsrc_t rs, u32 slot, int lane) {
#pragma unroll
    for (int v = 0; v < 8; ++v) __builtin_amdgcn_raw_buffer_store_b128(d[v], rs, (int)(slot + v * 1024 + lane * 16), 0, AUX_SYS);
}
__device__ __forceinline__ u32 pack_store(const u32x4 (&d)[8], rsrc_t rs, u32 slot, int lane) {
    u32 mx = 0, mn = 0xFFFFFFFFu, mz = 0xFFFFFFFFu;
#pragma unroll
    for (int f = 0; f < 8; ++f)
#pragma unroll
        for (int e = 0; e < 4; ++e) { const u32 a = d[f][e] & 0x7FFF7FFFu; mx = pk_max(mx, a); mn = pk_min(mn, pk_sub_s(a, 0x00010001u)); mz = pk_min(mz, a); }
    const u32 emax = ((mx & 0xFFFFu) > (mx >> 16) ? (mx & 0xFFFFu) : (mx >> 16)) >> 7;
    u32 E = 0;
#pragma unroll
    for (int b = 7; b >= 0; --b) { const u32 cand = E | (1u << b); if (__builtin_amdgcn_ballot_w64(emax >= cand)) E = cand; }
    const u32 lo = E > 15 ? E - 15 : 0, L = lo << 7;
    const u32 mnl = (mn & 0xFFFFu) < (mn >> 16) ? (mn & 0xFFFFu) : (mn >> 16);
    const u32 zero = __builtin_amdgcn_ballot_w64((mz & 0xFFFFu) == 0 || (mz >> 16) == 0) ? META_ZERO : 0u;
    u32 cnt = 0, hdr = 0;
    if (__builtin_amdgcn_ballot_w64(mnl < L)) {
#pragma unroll
        for (int s = 0; s < 64; ++s) {
            if (cnt <= 15) {
                const u32 dw = d[s >> 3][(s >> 1) & 3], val = (s & 1) ? (dw >> 16) : (dw & 0xFFFFu);
                u64 b = __builtin_amdgcn_ballot_w64(((val & 0x7FFFu) - 1u) < L);
                while (b && cnt <= 15) {
                    const int l = __builtin_ctzll(b); b &= b - 1;
                    const u32 ent = ((u32)(l * 64 + s) << 16) | (u32)__builtin_amdgcn_readlane((int)val, l);
                    if ((u32)lane == cnt) hdr = ent;
                    ++cnt;
                }
            }
        }
        if (cnt > 15) { store_raw(d, rs, slot, lane); return META_RAW; }
        if (lane < 16) __builtin_amdgcn_raw_buffer_store_b32(hdr, rs, (int)(slot + HDR + lane * 4), 0, AUX_SYS);
    }
    const u32 LL = rfl(L | (L << 16));
    u32x4 V[6];
#pragma unroll
    for (int f = 0; f < 8; ++f) {
        u32 h[4];
#pragma unroll
        for (int e = 0; e < 4; ++e) { const u32 v = d[f][e]; h[e] = pk_subsat_s(v & 0x7FFF7FFFu, LL) | ((v >> 4) & 0x08000800u); }
        V[f >> 1][(f & 1) * 2 + 0] = __builtin_amdgcn_perm(h[1], h[0], 0x06040200u);
        V[f >> 1][(f & 1) * 2 + 1] = __builtin_amdgcn_perm(h[3], h[2], 0x06040200u);
        V[4 + (f >> 2)][f & 3] = __builtin_amdgcn_perm(h[1], h[0], 0x07050301u) | (__builtin_amdgcn_perm(h[3], h[2], 0x07050301u) << 4);
    }
#pragma unroll
    for (int v = 0; v < 6; ++v) __builtin_amdgcn_raw_buffer_store_b128(V[v], rs, (int)(slot + v * 1024 + lane * 16), 0, AUX_SYS);
    return lo | (cnt << 8) | zero;
}
struct Pull { u32x4 V[6]; u32 h; };
__device__ __forceinline__ void pull_issue(Pull& p, rsrc_t rs, u32 slot, int lane) {
#pragma unroll
    for (int v = 0; v < 6; ++v) p.V[v] = __builtin_amdgcn_raw_buffer_load_b128(rs, (int)(slot + v * 1024 + lane * 16), 0, AUX_SYS);
    p.h = __builtin_amdgcn_raw_buffer_load_b32(rs, (int)(slot + HDR + (lane & 15) * 4), 0, AUX_SYS);
}
__device__ __forceinline__ void unpack(u32x4 (&d)[8], const Pull& p, rsrc_t rs, u32 slot, u32 meta, int lane) {
    if (meta & META_RAW) {
#pragma unroll
        for (int v = 0; v < 6; ++v) d[v] = p.V[v];
        d[6] = __builtin_amdgcn_raw_buffer_load_b128(rs, (int)(slot + 6 * 1024 + lane * 16), 0, AUX_SYS);
        d[7] = __builtin_amdgcn_raw_buffer_load_b128(rs, (int)(slot + 7 * 1024 + lane * 16), 0, AUX_SYS);
        wait_vm0_known();
        return;
    }
    const u32 cnt = (meta >> 8) & 31u;
    const u32 L = (meta & 0xFFu) << 7, LL = L | (L << 16);
#pragma unroll
    for (int f = 0; f < 8; ++f) {
        const u32 B0 = p.V[f >> 1][(f & 1) * 2], B1 = p.V[f >> 1][(f & 1) * 2 + 1], N = p.V[4 + (f >> 2)][f & 3];
        const u32 N0 = (N & 0x07070707u) | ((N << 4) & 0x80808080u), N1 = (N & 0x80808080u) | ((N >> 4) & 0x07070707u);
        d[f][0] = __builtin_amdgcn_perm(N0, B0, 0x05010400u); d[f][1] = __builtin_amdgcn_perm(N0, B0, 0x07030602u);
        d[f][2] = __builtin_amdgcn_perm(N1, B1, 0x05010400u); d[f][3] = __builtin_amdgcn_perm(N1, B1, 0x07030602u);
    }
    if (meta & META_ZERO) {
#pragma unroll
        for (int q = 0; q < 32; ++q) { const u32 h = d[q >> 2][q & 3]; d[q >> 2][q & 3] = pk_mad_s(pk_min_s(h & 0x7FFF7FFFu, 0x00010001u), LL, h); }
    } else {
#pragma unroll
        for (int q = 0; q < 32; ++q) d[q >> 2][q & 3] = pk_add_s(d[q >> 2][q & 3], LL);
    }
    for (u32 i = 0; i < cnt; ++i) {
        const u32 ent = (u32)__builtin_amdgcn_readlane((int)p.h, (int)i);
        const u32 l = ent >> 22, s = (ent >> 16) & 63u, raw = ent & 0xFFFFu;
        const u32 keep = (s & 1) ? 0x0000FFFFu : 0xFFFF0000u, val = (s & 1) ? (raw << 16) : raw;
        const bool me = (u32)lane == l;
#pragma unroll
        for (int q = 0; q < 32; ++q) { const u32 o = d[q >> 2][q & 3]; d[q >> 2][q & 3] = (me && (s >> 1) == (u32)q) ? ((o & keep) | val) : o; }
    }
}
__device__ __forceinline__ void fill_nan(u32x4 (&d)[8]) {
#pragma unroll
    for (int v = 0; v < 8; ++v) d[v] = (u32x4){0x7FC07FC0u, 0x7FC07FC0u, 0x7FC07FC0u, 0x7FC07FC0u};
}
__device__ __forceinline__ void copy_store(const Ctx& c, const Pull& p, rsrc_t rs, u32 slot, u32 dst, u32 meta) {
    const int lane = c.lane;
    if (meta & META_RAW) {
        const u32x4 a = __builtin_amdgcn_raw_buffer_load_b128(rs, (int)(slot + 6 * 1024 + lane * 16), 0, AUX_SYS);
        const u32x4 b = __builtin_amdgcn_raw_buffer_load_b128(rs, (int)(slot + 7 * 1024 + lane * 16), 0, AUX_SYS);
        wait_vm0_known();
        __builtin_amdgcn_raw_buffer_store_b128(a, c.own, (int)(dst + 6 * 1024 + lane * 16), 0, AUX_SYS);
        __builtin_amdgcn_raw_buffer_store_b128(b, c.own, (int)(dst + 7 * 1024 + lane * 16), 0, AUX_SYS);
    } else if ((meta >> 8) & 31u) {
        if (lane < 16) __builtin_amdgcn_raw_buffer_store_b32(p.h, c.own, (int)(dst + HDR + lane * 4), 0, AUX_SYS);
    }
#pragma unroll
    for (int v = 0; v < 6; ++v) __builtin_amdgcn_raw_buffer_store_b128(p.V[v], c.own, (int)(dst + v * 1024 + lane * 16), 0, AUX_SYS);
}

__device__ __forceinline__ u32 ticket_issue(const Ctx& c, u32 off) {
    u32 r; u64 sv; const u32 one = 1u;
    asm volatile("s_mov_b64 %1, exec\n\ts_mov_b64 exec, 1\n\tglobal_atomic_add %0, %2, %3, %4 sc0\n\ts_mov_b64 exec, %1"
                 : "=&v"(r), "=&s"(sv) : "v"(off), "v"(one), "s"(c.ws) : "memory");
    return r;
}
__device__ __forceinline__ u32 ticket_get(u32 r) { asm volatile("" : "+v"(r) :: "memory"); return rfl(r); }

struct Step { u32 foff, slot, meta; int sel, p0, p1, p2, p3; };
template <int SPT, class Gen, class Act, class Pub>
__device__ __forceinline__ void pull_list(Ctx& c, int list, u32 first, u32 nwaves, u32 njobs, rsrc_t rs0, rsrc_t rs1, Gen gen, Act act, Pub pub) {
    DT0();
    const u32 tkoff = OFF_TKT + (u32)list * 128u, tkbase = nwaves - (c.call - 1u) * njobs;
    u32 jc = first, jn = 0, tk = 0;
    if (SPT == 1) tk = ticket_issue(c, tkoff);
    Step cur = gen(jc, 0), prev = cur;
    bool ok = wait_flag(c, cur.foff, cur.meta), have_prev = false, more = true;
    if (SPT == 1) { wait_vm0_known(); jn = ticket_get(tk) + tkbase; }
    DT(1);
    Pull pa, pb;
    pull_issue(pa, cur.sel ? rs1 : rs0, cur.slot, c.lane);
    auto half = [&](const Pull& p, Pull& pn, int sub) {
        const bool same = SPT == 2 && sub == 0;
        more = same || jn < njobs;
        Step nxt = gen(same || !more ? jc : jn, same ? 1 : 0);
        const bool draw = SPT == 1 ? more : same;
        if (draw) tk = ticket_issue(c, tkoff);
        const u64 fv = flag_load(c, nxt.foff);
        wait_vm0_known();
        DT(0);
        const u32 jd = draw ? ticket_get(tk) + tkbase : njobs;
        if (have_prev) pub(prev);
        const bool nready = more && flag_test(c, fv, nxt.meta);
        if (nready) pull_issue(pn, nxt.sel ? rs1 : rs0, nxt.slot, c.lane);
        DT(3);
        act(cur, p, cur.sel ? rs1 : rs0, ok);
        prev = cur; have_prev = true; ok = true;
        DT(2);
        if (more && !nready) {
            wait_vm0_known(); pub(prev); have_prev = false;
            ok = wait_flag(c, nxt.foff, nxt.meta);
            pull_issue(pn, nxt.sel ? rs1 : rs0, nxt.slot, c.lane);
            DT(1);
        }
        cur = nxt;
        if (SPT == 1) { jc = jn; jn = jd; }
        else if (same) jn = jd; else jc = jn;
    };
    for (;;) {
        half(pa, pb, 0); if (!more) break;
        half(pb, pa, 1); if (!more) break;
    }
    wait_vm0_known();
    if (have_prev) pub(prev);
}

__device__ __forceinline__ int cq_of(int q, int r) {
    return ((r >> 1) ^ ((q & 1) ? (r >> 2) : 0) ^ ((q & 2) ? r : 0)) & 1;
}
__device__ __forceinline__ int link_d(int link) { return (0x765431 >> (4 * link)) & 7; }
__device__ __forceinline__ int link_q(int link, int i) { return ((i ? 0x233231 : 0x110020) >> (4 * link)) & 3; }
__device__ __forceinline__ int link_of_d(int d) { return d == 1 ? 0 : d - 2; }
__device__ __forceinline__ int t_of(int d, int q, int kq) {
    if (d == 0) return kq * 4 + q;
    const int link = link_of_d(d);
    return 16 + (2 * kq + (link_q(link, 0) == q ? 0 : 1)) * 6 + link;
}
__device__ __forceinline__ int d3_pair(int d) { return (d & 1) * 2 + ((d >> 1) & 1); }
__device__ __forceinline__ int d3_of_pair(int p) { return 4 + (p >> 1) + (p & 1) * 2; }
__device__ __forceinline__ int rowtile_of(int rc, int o, int a) {
    const int d = rc ^ o;
    return d == 0 ? 12 + a : d == 2 ? a : 4 + 2 * d3_pair(d) + (a & 1);
}
__device__ __forceinline__ void block_of_rowtile(int rc, int i, int& o, int& a) {
    if (i >= 12) { o = rc; a = i - 12; }
    else if (i < 4) { o = rc ^ 2; a = i; }
    else { const int p = (i - 4) >> 1; o = rc ^ d3_of_pair(p); a = 2 * (p >> 1) + ((i - 4) & 1); }
}
__device__ __forceinline__ u32 raw_frag_off(int type, int sub, int f) {
    return type == 0 ? (u32)(sub * 8192 + f * 1024) : (u32)((f >> 2) * 16384 + sub * 4096 + (f & 3) * 1024);
}
__device__ __forceinline__ void store_operand(const Ctx& c, const u32x4 (&d)[8], int type, int tile, int t, int sub) {
    const u32 base = (type == 0 ? OFF_A : OFF_B) + (u32)(tile * 64 + t) * 32768u + (u32)c.lane * 16u;
#pragma unroll
    for (int f = 0; f < 8; ++f) __builtin_amdgcn_raw_buffer_store_b128(d[f], c.own, (int)(base + raw_frag_off(type, sub, f)), 0, AUX_SYS);
}
__device__ __forceinline__ void count_operand(const Ctx& c, int type, int tile, int t) {
    GAS u32* p = (GAS u32*)(c.ws + (type == 0 ? OFF_RDYA : OFF_RDYB)) + tile * 64 + t;
    if (c.lane == 0) __hip_atomic_fetch_add(p, 1u, __ATOMIC_RELAXED, __HIP_MEMORY_SCOPE_AGENT);
}

__device__ __forceinline__ void pack_job(const Params& P, const Ctx& c, LAS unsigned char* tile, int pj) {
    const int kq = pj >> 10, q = pj & 3, type = (pj >> 2) & 1, g = (pj & 1023) >> 3, kt = 4 * q + kq, lane = c.lane, s = c.rank;
    u32x4 d[8];
    if (type == 0) {
        GAS const char* src = (GAS const char*)P.x + (size_t)(64 * g + (lane & 15)) * 2048 + (64 * kt + 8 * (lane >> 4)) * 2;
#pragma unroll
        for (int f = 0; f < 8; ++f) d[f] = *(GAS const u32x4*)(src + (f >> 1) * 16 * 2048 + (f & 1) * 64);
    } else {
        GAS const char* src = (GAS const char*)P.w + (size_t)(64 * kt + (lane >> 3)) * 16384 + (64 * g) * 2 + (lane & 7) * 16;
#pragma unroll
        for (int v = 0; v < 8; ++v)
            __builtin_amdgcn_global_load_lds((GAS const u32*)(src + (size_t)v * 8 * 16384), (LAS u32*)(tile + v * 1024), 16, 0, 0);
        wait_vm0();
        const int lg = lane >> 4, lq = (lane >> 2) & 3, lp = lane & 3;
        LAS unsigned char* b = tile + (8 * lg + lq) * 128 + lp * 16;
#pragma unroll
        for (int f = 0; f < 8; ++f) {
            const int o = (f & 1) * 32 * 128 + (f >> 2) * 64 + ((f >> 1) & 1) * 8;
            const s16x4 lo4 = __builtin_amdgcn_ds_read_tr16_b64_v4i16((LAS s16x4*)(b + o));
            const s16x4 hi4 = __builtin_amdgcn_ds_read_tr16_b64_v4i16((LAS s16x4*)(b + o + 4 * 128));
            typedef u32 u32x2 __attribute__((ext_vector_type(2)));
            const u32x2 l2 = __builtin_bit_cast(u32x2, lo4), h2 = __builtin_bit_cast(u32x2, hi4);
            d[f] = (u32x4){l2[0], l2[1], h2[0], h2[1]};
        }
        asm volatile("s_waitcnt lgkmcnt(0)" ::: "memory");
    }
    int rc[2], tl[2], tt[2];
    const int cs = cq_of(q, s);
#pragma unroll
    for (int i = 0; i < 2; ++i) {
        int e1, gam;
        if (type == 0) { const int o = g >> 4, h = (g >> 3) & 1; gam = i; e1 = h == 0 ? (o & 1) : ((o ^ (o >> 2) ^ gam) & 1); }
        else { gam = g >> 6; e1 = i; }
        const int e2 = (cs ^ ((q & 2) ? e1 : 0) ^ ((q & 1) ? gam : 0)) & 1;
        rc[i] = e1 | (e2 << 1) | (gam << 2);
        tl[i] = type == 0 ? rowtile_of(rc[i], g >> 4, (g >> 2) & 3) : ((g >> 2) & 15);
        tt[i] = t_of(rc[i] ^ s, q, kq);
    }
    const int sub = g & 3;
    const u32 meta = pack_store(d, c.own, OFF_IN + c.par * IN_BYTES + (u32)(type * 2048 + kt * 128 + g) * SLOT, lane);
    if (rc[0] == s) store_operand(c, d, type, tl[0], tt[0], sub);
    if (rc[1] == s) store_operand(c, d, type, tl[1], tt[1], sub);
    wait_vm0();
#pragma unroll
    for (int i = 0; i < 2; ++i) {
        if (rc[i] == s) count_operand(c, type, tl[i], tt[i]);
        else post_flag(P, c, rc[i], F_IN + (u32)(((type * 16 + tl[i]) * 64 + tt[i]) * 4 + sub) * 8u, meta);
    }
}

__device__ __forceinline__ void in_list(const Params& P, Ctx& c, int link, int widx, int wpl) {
    const int s = c.rank ^ link_d(link);
    auto gen = [&](u32 job, int) {
        const int j = (int)job;
        int kc, type, tile, sub;
        if (j < 896) { kc = j / 112; const int u = j % 112; if (u < 48) { type = 0; tile = u >> 2; } else { type = 1; tile = (u - 48) >> 2; } sub = u & 3; }
        else { const int m = j - 896; kc = m >> 4; type = 0; tile = 12 + ((m >> 2) & 3); sub = m & 3; }
        const int t = 16 + kc * 6 + link, kt = 4 * link_q(link, kc & 1) + (kc >> 1);
        int g;
        if (type == 0) { int o, a; block_of_rowtile(c.rank, tile, o, a); g = o * 16 + a * 4 + sub; } else g = (c.rank >> 2) * 64 + tile * 4 + sub;
        Step st; st.foff = F_IN + (u32)(((type * 16 + tile) * 64 + t) * 4 + sub) * 8u;
        st.slot = OFF_IN + c.par * IN_BYTES + (u32)(type * 2048 + kt * 128 + g) * SLOT;
        st.sel = 0; st.meta = 0; st.p0 = type; st.p1 = tile; st.p2 = t; st.p3 = sub;
        return st;
    };
    auto act = [&](Step& st, const Pull& p, rsrc_t rs, bool ok) {
        u32x4 d[8];
        if (ok) unpack(d, p, rs, st.slot, st.meta, c.lane); else fill_nan(d);
        store_operand(c, d, st.p0, st.p1, st.p2, st.p3);
    };
    auto pub = [&](const Step& st) { count_operand(c, st.p0, st.p1, st.p2); };
    const rsrc_t rs = mk_rsrc(P.ws[s & 7]);
    pull_list<1>(c, link, (u32)widx, (u32)wpl, 1024u, rs, rs, gen, act, pub);
}

__device__ __forceinline__ void acc_add(float (&acc)[64], const u32x4 (&d)[8]) {
#pragma unroll
    for (int q = 0; q < 32; ++q) { const u32 v = d[q >> 2][q & 3]; acc[2 * q] += __builtin_bit_cast(float, v << 16); acc[2 * q + 1] += __builtin_bit_cast(float, v & 0xFFFF0000u); }
}
__device__ __forceinline__ void write_out(const Params& P, const Ctx& c, const float (&acc)[64], int a, int cj, int jb) {
    const int w = jb >> 1, ai = jb & 1, lane = c.lane;
    GAS char* o = (GAS char*)P.out + (size_t)(a * 256 + ai * 128 + (w >> 2) * 64 + (lane & 15)) * 16384 + (cj * 256 + (w & 3) * 64 + 8 * (lane >> 4)) * 2;
#pragma unroll
    for (int f = 0; f < 8; ++f) {
        u32x4 v;
#pragma unroll
        for (int e = 0; e < 4; ++e) v[e] = cvt_pk_bf16(acc[f * 8 + 2 * e], acc[f * 8 + 2 * e + 1]);
        *(GAS u32x4*)(o + (size_t)(f >> 1) * 16 * 16384 + (f & 1) * 64) = v;
    }
}
__device__ __forceinline__ void copy_list(const Params& P, Ctx& c, int grp, int widx, int wpl) {
    const int r = c.rank;
    auto gen = [&](u32 job, int) {
        const int j = (int)job, jb = j & 15;
        Step st; st.meta = 0;
        if (grp == 0) {
            const int u = j >> 4, pt = (u & 3) * 16 + (u >> 2) * 2;
            st.foff = F_P + (u32)(pt * 16 + jb) * 8u; st.slot = OFF_OUT + c.par * OUT_BYTES + (u32)(pt * 16 + jb) * SLOT;
            st.p0 = (int)(OFF_OWN + (u32)((pt * 4 + 3) * 16 + jb) * SLOT); st.p1 = r; st.p2 = (int)(F_OWN + (u32)((pt * 4 + 3) * 16 + jb) * 8u);
        } else {
            const int second = j >= 256, v = (j >> 4) & 15, u = (v & 3) * 4 + (v >> 2), path = (grp - 1) ^ second, pt = 4 * u + 1 + 2 * path, rs = path * 16 + u;
            if (!second) {
                st.foff = F_R + (u32)(rs * 16 + jb) * 8u; st.slot = OFF_OUT + c.par * OUT_BYTES + (u32)(pt * 16 + jb) * SLOT;
                st.p0 = (int)(OFF_REL + c.par * REL_BYTES + (u32)(rs * 16 + jb) * SLOT); st.p1 = r ^ (path ? 1 : 3); st.p2 = (int)(F_P + (u32)(pt * 16 + jb) * 8u);
            } else {
                st.foff = F_P + (u32)(pt * 16 + jb) * 8u; st.slot = OFF_REL + c.par * REL_BYTES + (u32)(rs * 16 + jb) * SLOT;
                st.p0 = (int)(OFF_OWN + (u32)((pt * 4 + 3) * 16 + jb) * SLOT); st.p1 = r; st.p2 = (int)(F_OWN + (u32)((pt * 4 + 3) * 16 + jb) * 8u);
            }
        }
        st.p3 = 0; st.sel = 0;
        return st;
    };
    auto act = [&](Step& st, const Pull& p, rsrc_t rs, bool ok) {
        if (ok) copy_store(c, p, rs, st.slot, (u32)st.p0, st.meta);
        else { u32x4 d[8]; fill_nan(d); store_raw(d, c.own, (u32)st.p0, c.lane); st.meta = META_RAW; }
    };
    auto pub = [&](const Step& st) { post_flag(P, c, st.p1, (u32)st.p2, st.meta); };
    const rsrc_t rs = mk_rsrc(P.ws[(r ^ (grp == 0 ? 2 : grp == 1 ? 1 : 3)) & 7]);
    pull_list<1>(c, 6 + grp, (u32)widx, (u32)wpl, 512u, rs, rs, gen, act, pub);
}
__device__ __forceinline__ void reduce_list(const Params& P, Ctx& c, int h, int widx, int wpl) {
    const int r = c.rank;
    float acc[64];
    auto gen = [&](u32 ujob, int sub) {
        const int job = (int)ujob, sl = (sub ^ job) & 1, jb = job & 15, ot2 = job >> 4;
        const int a = 2 * h + (ot2 & 1), j = ot2 >> 1, ot = a * 16 + j, p = 2 * h + sl, ti = 4 + 2 * p + (a & 1);
        Step st; st.meta = 0;
        st.foff = F_D3 + (u32)((ot * 2 + sl) * 16 + jb) * 8u; st.sel = sl;
        st.slot = OFF_OUT + c.par * OUT_BYTES + (u32)((ti * 16 + j) * 16 + jb) * SLOT;
        st.p0 = sub; st.p1 = a; st.p2 = (1 - (r >> 2)) * 16 + j; st.p3 = jb;
        return st;
    };
    auto act = [&](Step& st, const Pull& p, rsrc_t rs, bool ok) {
        u32x4 d[8];
        if (ok) unpack(d, p, rs, st.slot, st.meta, c.lane); else fill_nan(d);
        if (st.p0 == 0) {
#pragma unroll
            for (int q = 0; q < 64; ++q) acc[q] = 0.f;
        }
        acc_add(acc, d);
        if (st.p0 == 1) write_out(P, c, acc, st.p1, st.p2, st.p3);
    };
    auto pub = [&](const Step&) {};
    pull_list<2>(c, 9 + h, (u32)widx, (u32)wpl, 512u, mk_rsrc(P.ws[(r ^ d3_of_pair(2 * h)) & 7]), mk_rsrc(P.ws[(r ^ d3_of_pair(2 * h + 1)) & 7]), gen, act, pub);
}
__device__ __forceinline__ void own_job(const Params& P, const Ctx& c, int n) {
    const int pt = n >> 4, jb = n & 15;
    float acc[64];
#pragma unroll
    for (int i = 0; i < 64; ++i) acc[i] = 0.f;
    for (int part = 0; part < 4; ++part) {
        u32 meta; u32x4 d[8];
        if (wait_flag(c, F_OWN + (u32)((pt * 4 + part) * 16 + jb) * 8u, meta)) {
            Pull p; const u32 slot = OFF_OWN + (u32)((pt * 4 + part) * 16 + jb) * SLOT;
            pull_issue(p, c.own, slot, c.lane); unpack(d, p, c.own, slot, meta, c.lane);
        } else fill_nan(d);
        acc_add(acc, d);
    }
    write_out(P, c, acc, pt >> 4, (c.rank >> 2) * 16 + (pt & 15), jb);
}

struct GJob { int ti, tj, t0, nt; };

#define STAMP(i) do {} while (0)

__device__ __forceinline__ void gemm_role(const Params& P, const Ctx& c, LAS unsigned char* lds, int xcd, int idx) {
    const int tid = threadIdx.x, wid = (int)rfl(tid >> 6), lane = c.lane, wr = wid >> 2, wc = wid & 3;
    constexpr int HTB = 16384;
    const u32 voff = (u32)tid * 16u;
    const unsigned ldsw = (unsigned)wid * 1024u;
    const int aoff = wr * 8192 + lane * 16, boff = wc * 4096 + lane * 16;
    LAS volatile int* ctl = (LAS volatile int*)(lds + LDS_CTL);
    int avail = 0; bool bad = false;
#define G_SA(b, h) (((b) * 2 + (h)) * HTB)
#define G_SB(b, h) ((4 + (b) * 2 + (h)) * HTB)
#define G_STAGE(bufoff, gbase) do { _Pragma("unroll") for (int _i = 0; _i < 2; ++_i) \
        __builtin_amdgcn_global_load_lds((GAS const u32*)((GAS const char*)(gbase) + voff + _i * 8192), (LAS u32*)(lds + (bufoff) + ldsw + _i * 8192), 16, 0, AUX_DEV); } while (0)
#define G_LDA(dst, b, h) do { _Pragma("unroll") for (int m = 0; m < 4; ++m) _Pragma("unroll") for (int k = 0; k < 2; ++k) dst[m][k] = *(const LAS bf16x8*)(lds + G_SA(b, h) + aoff + m * 2048 + k * 1024); } while (0)
#define G_LDB(dst, b, h) do { _Pragma("unroll") for (int n = 0; n < 2; ++n) _Pragma("unroll") for (int k = 0; k < 2; ++k) dst[n][k] = *(const LAS bf16x8*)(lds + G_SB(b, h) + boff + n * 2048 + k * 1024); } while (0)
#define G_MMA(ai, bj, At, Bt) do { __builtin_amdgcn_s_setprio(1); _Pragma("unroll") for (int m = 0; m < 4; ++m) _Pragma("unroll") for (int n = 0; n < 2; ++n) _Pragma("unroll") for (int k = 0; k < 2; ++k) \
        acc[ai][bj][m][n] = __builtin_amdgcn_mfma_f32_16x16x32_bf16(Bt[n][k], At[m][k], acc[ai][bj][m][n], 0, 0, 0); __builtin_amdgcn_s_setprio(0); } while (0)
#define G_WAIT_V(n) asm volatile("s_waitcnt vmcnt(" #n ")" ::: "memory")
#define G_WAIT_L(n) asm volatile("s_waitcnt lgkmcnt(" #n ")" ::: "memory")
#define G_BAR __builtin_amdgcn_s_barrier()
#define G_SCHED __builtin_amdgcn_sched_barrier(0)
    auto ensure = [&](const GJob& j, int tneed) {
        if (avail >= tneed) return;
        if (wid == 0) {
            GAS const u32* ra = (GAS const u32*)(c.ws + OFF_RDYA) + j.ti * 64 + lane;
            GAS const u32* rb = (GAS const u32*)(c.ws + OFF_RDYB) + j.tj * 64 + lane;
            GAS u32* tmo = (GAS u32*)(c.ws + OFF_TMO);
            const u32 target = 4u * c.call;
            const u64 below = j.t0 ? ((1ull << j.t0) - 1ull) : 0ull;
            const u64 t0 = __builtin_amdgcn_s_memrealtime();
            int av, bd = 0;
            for (;;) {
                const u32 a = __hip_atomic_load(ra, __ATOMIC_RELAXED, __HIP_MEMORY_SCOPE_AGENT), b = __hip_atomic_load(rb, __ATOMIC_RELAXED, __HIP_MEMORY_SCOPE_AGENT);
                const u64 ok = __builtin_amdgcn_ballot_w64((int)(a - target) >= 0 && (int)(b - target) >= 0) | below;
                av = ok == ~0ull ? 64 : __builtin_ctzll(~ok);
                if (av >= tneed) break;
                if (rfl(__hip_atomic_load(tmo, __ATOMIC_RELAXED, __HIP_MEMORY_SCOPE_AGENT)) == c.call || __builtin_amdgcn_s_memrealtime() - t0 > DEADLINE) {
                    __hip_atomic_store(tmo, c.call, __ATOMIC_RELAXED, __HIP_MEMORY_SCOPE_AGENT); av = 64; bd = 1; break;
                }
                __builtin_amdgcn_s_sleep(4);
            }
            if (lane == 0) { ctl[0] = av; if (bd) ctl[1] = 1; }
            G_WAIT_L(0);
        }
        asm volatile("" ::: "memory"); G_BAR; asm volatile("" ::: "memory");
        avail = (int)rfl((u32)ctl[0]); bad = rfl((u32)ctl[1]) != 0;
    };
    GJob cur, nxt;
    cur.ti = 6 * (xcd & 1) + idx % 6; cur.tj = 4 * (xcd >> 1) + idx / 6; cur.t0 = 0; cur.nt = 64;
    { const int third = idx >> 3; nxt.ti = 12 + (idx & 3); nxt.tj = 2 * xcd + ((idx >> 2) & 1); nxt.t0 = 22 * third; nxt.nt = third == 2 ? 20 : 22; }
    f32x4 acc[2][2][4][2];
    bf16x8 At[4][2], B0[2][2], B1[2][2];
    constexpr size_t kstep = 32768, hstep = 16384;
    for (int ui = 0; ui < 2; ++ui) {
        const bool has_next = ui == 0;
#pragma unroll
        for (int a = 0; a < 2; ++a)
#pragma unroll
            for (int b = 0; b < 2; ++b)
#pragma unroll
                for (int m = 0; m < 4; ++m)
#pragma unroll
                    for (int n = 0; n < 2; ++n) acc[a][b][m][n] = (f32x4){0.f, 0.f, 0.f, 0.f};
        const char* cA = c.ws + OFF_A + (size_t)(cur.ti * 64 + cur.t0) * kstep;
        const char* cB = c.ws + OFF_B + (size_t)(cur.tj * 64 + cur.t0) * kstep;
        avail = 0;
        ensure(cur, cur.t0 + 2);
        G_STAGE(G_SB(0, 0), cB); G_STAGE(G_SA(0, 0), cA); G_STAGE(G_SB(0, 1), cB + hstep); G_STAGE(G_SA(0, 1), cA + hstep);
        if (wr == 1) G_BAR;
        G_WAIT_V(4); G_BAR;
        G_STAGE(G_SB(1, 0), cB + kstep); G_STAGE(G_SA(1, 0), cA + kstep); G_STAGE(G_SB(1, 1), cB + hstep + kstep);
        G_WAIT_V(6); G_BAR;
        if (ui == 0) STAMP(2);
        const int nt = cur.nt;
        for (int t = 0; t < nt; t += 2) {
            const bool last = (t == nt - 2);
            const char* a1 = cA + (size_t)(t + 1) * kstep;
            const char* a2 = cA + (size_t)(last ? t : t + 2) * kstep; const char* b2 = cB + (size_t)(last ? t : t + 2) * kstep;
            const char* a3 = a2 + kstep; const char* b3 = b2 + kstep;
            if (!last) ensure(cur, cur.t0 + t + 4);
            G_LDB(B0, 0, 0); G_SCHED; G_LDA(At, 0, 0); G_STAGE(G_SA(1, 1), a1 + hstep);
            G_WAIT_L(8); G_BAR; G_WAIT_L(0); G_MMA(0, 0, At, B0); G_BAR; G_SCHED;
            G_LDB(B1, 0, 1); G_STAGE(G_SB(0, 0), b2);
            G_BAR; G_WAIT_L(0); G_MMA(0, 1, At, B1); G_BAR;
            G_LDA(At, 0, 1); G_STAGE(G_SA(0, 0), a2);
            G_BAR; G_WAIT_L(0); G_MMA(1, 0, At, B0); G_BAR; G_SCHED;
            G_STAGE(G_SB(0, 1), b2 + hstep);
            G_WAIT_V(6); G_BAR; G_MMA(1, 1, At, B1); G_BAR;
            G_LDB(B0, 1, 0); G_SCHED; G_LDA(At, 1, 0); G_STAGE(G_SA(0, 1), a2 + hstep);
            G_WAIT_L(8); G_BAR; G_WAIT_L(0); G_MMA(0, 0, At, B0); G_BAR; G_SCHED;
            G_LDB(B1, 1, 1); G_STAGE(G_SB(1, 0), b3);
            G_BAR; G_WAIT_L(0); G_MMA(0, 1, At, B1); G_BAR;
            G_LDA(At, 1, 1); G_STAGE(G_SA(1, 0), a3);
            G_BAR; G_WAIT_L(0); G_MMA(1, 0, At, B0); G_BAR; G_SCHED;
            G_STAGE(G_SB(1, 1), b3 + hstep);
            G_WAIT_V(6); G_BAR; G_MMA(1, 1, At, B1); G_BAR;
        }
        if (wr == 0) G_BAR;
        STAMP(3 + 2 * ui);
        {
            u32 meta[2], slot[2];
#pragma unroll
            for (int ai = 0; ai < 2; ++ai) {
                u32x4 d[8];
#pragma unroll
                for (int m = 0; m < 4; ++m)
#pragma unroll
                    for (int bj = 0; bj < 2; ++bj) {
                        const f32x4 v0 = acc[ai][bj][m][0], v1 = acc[ai][bj][m][1];
                        d[m * 2 + bj] = (u32x4){cvt_pk_bf16(v0[0], v0[1]), cvt_pk_bf16(v0[2], v0[3]), cvt_pk_bf16(v1[0], v1[1]), cvt_pk_bf16(v1[2], v1[3])};
                    }
                if (bad) fill_nan(d);
                const int jb = wid * 2 + ai;
                if (has_next) { slot[ai] = OFF_OUT + c.par * OUT_BYTES + (u32)((cur.ti * 16 + cur.tj) * 16 + jb) * SLOT; meta[ai] = pack_store(d, c.own, slot[ai], lane); }
                else { slot[ai] = OFF_OWN + (u32)((((cur.ti - 12) * 16 + cur.tj) * 4 + (cur.t0 != 0) + (cur.t0 == 44)) * 16 + jb) * SLOT; store_raw(d, c.own, slot[ai], lane); meta[ai] = META_RAW; }
            }
            wait_vm0();
#pragma unroll
            for (int ai = 0; ai < 2; ++ai) {
                const int jb = wid * 2 + ai, r = c.rank;
                int dst; u32 foff;
                if (!has_next) { dst = r; foff = F_OWN + (u32)((((cur.ti - 12) * 16 + cur.tj) * 4 + (cur.t0 != 0) + (cur.t0 == 44)) * 16 + jb) * 8u; }
                else if (cur.ti < 4) {
                    const int pt = cur.ti * 16 + cur.tj;
                    if ((pt & 1) == 0) { dst = r ^ 2; foff = F_P + (u32)(pt * 16 + jb) * 8u; }
                    else { const int path = (pt >> 1) & 1; dst = r ^ (path ? 3 : 1); foff = F_R + (u32)((path * 16 + (pt >> 2)) * 16 + jb) * 8u; }
                } else {
                    const int p = (cur.ti - 4) >> 1, a = 2 * (p >> 1) + ((cur.ti - 4) & 1);
                    dst = r ^ d3_of_pair(p); foff = F_D3 + (u32)(((a * 16 + cur.tj) * 2 + (p & 1)) * 16 + jb) * 8u;
                }
                post_flag(P, c, dst, foff, meta[ai]);
            }
        }
        STAMP(4 + 2 * ui);
        cur = nxt;
    }
#undef G_SA
#undef G_SB
#undef G_STAGE
#undef G_LDA
#undef G_LDB
#undef G_MMA
#undef G_WAIT_V
#undef G_WAIT_L
#undef G_BAR
#undef G_SCHED
}

__global__ __launch_bounds__(NTH, 2) void gemm_rs_kernel(const Params P) {
    extern __shared__ __attribute__((aligned(16))) unsigned char shm[];
    LAS unsigned char* lds = (LAS unsigned char*)shm;
    const int wg = blockIdx.x, xcd = wg & 7, idx = wg >> 3, tid = threadIdx.x, wid = (int)rfl(tid >> 6);
    Ctx c;
    c.rank = P.rank & 7; c.lane = tid & 63;
    c.ws = (char*)P.ws[c.rank]; c.own = mk_rsrc(c.ws);
    LAS volatile u32* ctl = (LAS volatile u32*)(lds + LDS_CTL);
    if (tid == 0) {
        ctl[2] = __hip_atomic_fetch_add((GAS u32*)(c.ws + OFF_CALL + wg * 64), 1u, __ATOMIC_RELAXED, __HIP_MEMORY_SCOPE_AGENT) + 1u;
        ctl[0] = 0; ctl[1] = 0;
    }
    __syncthreads();
    c.call = rfl(ctl[2]); c.par = c.call & 1u;
    STAMP(0);
    if (idx < NGEMM / 8) {
        const int gw = (xcd * (NGEMM / 8) + idx) * 8 + wid;
        for (int pj = gw; pj < 4096; pj += GEMM_WAVES) pack_job(P, c, lds + wid * 8192, pj);
        STAMP(1);
        __syncthreads();
        gemm_role(P, c, lds, xcd, idx);
        if (gw < 1024) own_job(P, c, gw);
        STAMP(7);
    } else {
        const int cw = wid * NCOMM + (idx - NGEMM / 8) * 8 + xcd;
        if (cw >= 6 * IN_WPL && cw >= 7 * O_WPL) return;
        if (cw < 6 * IN_WPL) in_list(P, c, cw % 6, cw / 6, IN_WPL);
        STAMP(3);
        if (cw < 7 * O_WPL) {
            const int grp = cw % 7, widx = cw / 7;
            if (grp < 3) copy_list(P, c, grp, widx, O_WPL);
            else reduce_list(P, c, (grp - 3) >> 1, widx * 2 + ((grp - 3) & 1), 2 * O_WPL);
        }
        STAMP(7);
    }
}
}

extern "C" void kernel_launch(int rank, int nranks, ncclComm_t comm, hipStream_t stream,
                              void* const* d_in, const int* in_sizes, int n_in,
                              void* d_out, int out_size,
                              const int* in_dtypes, int out_dtype,
                              void* const* d_out_all,
                              void* const* d_ws_all, size_t ws_bytes) {
    (void)comm; (void)in_sizes; (void)n_in; (void)out_size; (void)in_dtypes; (void)out_dtype; (void)d_out_all;
    if (nranks != gr::NR || rank < 0 || rank >= gr::NR || ws_bytes < gr::WS_NEED) { fprintf(stderr, "kernel_launch: unexpected configuration\n"); return; }
    static bool init[gr::NR] = {};
    if (!init[rank]) {
        if (hipFuncSetAttribute((const void*)gr::gemm_rs_kernel, hipFuncAttributeMaxDynamicSharedMemorySize, gr::LDS_BYTES) != hipSuccess) { fprintf(stderr, "hipFuncSetAttribute failed\n"); return; }
        init[rank] = true;
    }
    gr::Params P;
    P.x = d_in[0]; P.w = d_in[1]; P.out = d_out; P.rank = rank; P.pad = 0;
    for (int i = 0; i < gr::NR; ++i) P.ws[i] = d_ws_all[i];
    hipLaunchKernelGGL(gr::gemm_rs_kernel, dim3(gr::NWG), dim3(gr::NTH), gr::LDS_BYTES, stream, P);
    hipError_t e = hipGetLastError();
    if (e != hipSuccess) fprintf(stderr, "launch failed: %s\n", hipGetErrorString(e));
}
```

```cpp
#include <hip/hip_runtime.h>
#include <rccl/rccl.h>
#include <cstdio>

namespace gr {
typedef unsigned u32;
typedef unsigned long long u64;
typedef unsigned short u16;
typedef u32 u32x4 __attribute__((ext_vector_type(4)));
typedef u16 u16x2 __attribute__((ext_vector_type(2)));
typedef short bf16x8 __attribute__((ext_vector_type(8)));
typedef short s16x4 __attribute__((ext_vector_type(4)));
typedef float f32x4 __attribute__((ext_vector_type(4)));
#define LAS __attribute__((address_space(3)))
#define GAS __attribute__((address_space(1)))
typedef __amdgpu_buffer_rsrc_t rsrc_t;

constexpr int NR = 8;
constexpr int NWG = 256, NTH = 512, NGEMM = 192, NCOMM = NWG - NGEMM;
constexpr int GEMM_WAVES = NGEMM * 8, COMM_WAVES = NCOMM * 8;
constexpr u32 SLOT = 8448;
constexpr u32 HDR = 8192;
constexpr u64 DEADLINE = 100000000ull;

constexpr u32 OFF_CALL = 0;
constexpr u32 OFF_TMO = 16384;
constexpr u32 OFF_RDYA = 20480;
constexpr u32 OFF_RDYB = 24576;
constexpr u32 OFF_TKT = 28672;
constexpr u32 OFF_FLG = 32768;
constexpr u32 F_IN = 0;
constexpr u32 F_D3 = F_IN + 2 * 16 * 64 * 4 * 8;
constexpr u32 F_P = F_D3 + 64 * 2 * 16 * 8;
constexpr u32 F_R = F_P + 64 * 16 * 8;
constexpr u32 F_OWN = F_R + 32 * 16 * 8;
constexpr u32 FLG_BYTES = F_OWN + 64 * 4 * 16 * 8;
constexpr u32 OFF_DBG = 512 * 1024;
constexpr u32 CTL_BYTES = 1024 * 1024;
static_assert(OFF_FLG + 2 * FLG_BYTES <= OFF_DBG, "control region");
constexpr u32 IN_BYTES = 4096 * SLOT;
constexpr u32 OUT_BYTES = 192 * 16 * SLOT;
constexpr u32 REL_BYTES = 32 * 16 * SLOT;
constexpr u32 OFF_IN = CTL_BYTES;
constexpr u32 OFF_OUT = OFF_IN + 2 * IN_BYTES;
constexpr u32 OFF_REL = OFF_OUT + 2 * OUT_BYTES;
constexpr u32 OFF_OWN = OFF_REL + 2 * REL_BYTES;
constexpr u32 OFF_A = OFF_OWN + 64 * 4 * 16 * SLOT;
constexpr u32 OFF_B = OFF_A + 16 * 64 * 32768;
constexpr u32 WS_NEED = OFF_B + 16 * 64 * 32768;
static_assert(WS_NEED <= 257u * 1024u * 1024u, "workspace");
static_assert(OFF_A % 256 == 0 && OFF_OUT % 128 == 0 && OFF_OWN % 128 == 0, "alignment");

constexpr u32 META_RAW = 1u << 13, META_ZERO = 1u << 14;
#define GR_IN_WPL 16
#define GR_C_WPL 15
#define GR_X_WPL 15
constexpr int IN_WPL = GR_IN_WPL, C_WPL = GR_C_WPL, X_WPL = GR_X_WPL;
constexpr int O_WAVES = 3 * C_WPL + 4 * X_WPL;
#define GR_GAUX 16
constexpr int AUX_SYS = 17, AUX_DEV = GR_GAUX;
constexpr int LDS_STAGE = 131072, LDS_CTL = LDS_STAGE, LDS_BYTES = LDS_STAGE + 256;

struct Params {
    const void* x;
    const void* w;
    void* out;
    void* ws[NR];
    int rank;
    int pad;
};

__device__ __forceinline__ u32 rfl(u32 v) { return (u32)__builtin_amdgcn_readfirstlane((int)v); }
__device__ __forceinline__ u32 pk_max(u32 a, u32 b) { u32 r; asm("v_pk_max_u16 %0, %1, %2" : "=v"(r) : "v"(a), "v"(b)); return r; }
__device__ __forceinline__ u32 pk_min(u32 a, u32 b) { u32 r; asm("v_pk_min_u16 %0, %1, %2" : "=v"(r) : "v"(a), "v"(b)); return r; }
__device__ __forceinline__ u32 pk_min_s(u32 a, u32 s) { u32 r; asm("v_pk_min_u16 %0, %1, %2" : "=v"(r) : "v"(a), "s"(s)); return r; }
__device__ __forceinline__ u32 pk_add_s(u32 a, u32 s) { u32 r; asm("v_pk_add_u16 %0, %1, %2" : "=v"(r) : "v"(a), "s"(s)); return r; }
__device__ __forceinline__ u32 pk_sub_s(u32 a, u32 s) { u32 r; asm("v_pk_sub_u16 %0, %1, %2" : "=v"(r) : "v"(a), "s"(s)); return r; }
__device__ __forceinline__ u32 pk_subsat_s(u32 a, u32 s) { u32 r; asm("v_pk_sub_u16 %0, %1, %2 clamp" : "=v"(r) : "v"(a), "s"(s)); return r; }
__device__ __forceinline__ u32 pk_mad_s(u32 a, u32 s, u32 c) { u32 r; asm("v_pk_mad_u16 %0, %1, %2, %3" : "=v"(r) : "v"(a), "s"(s), "v"(c)); return r; }
__device__ __forceinline__ u32 cvt_pk_bf16(float lo, float hi) { u32 r; asm volatile("v_cvt_pk_bf16_f32 %0, %1, %2" : "=v"(r) : "v"(lo), "v"(hi)); return r; }
__device__ __forceinline__ rsrc_t mk_rsrc(void* p) { return __builtin_amdgcn_make_buffer_rsrc(p, (short)0, (int)WS_NEED, 0x00020000); }
__device__ __forceinline__ void wait_vm0() { asm volatile("s_waitcnt vmcnt(0)" ::: "memory"); }
__device__ __forceinline__ void wait_vm0_known() { __builtin_amdgcn_s_waitcnt(0x0F70); asm volatile("" ::: "memory"); }

struct Ctx {
    char* ws;
    rsrc_t own;
    u32 call, par;
    int rank, lane;
};
#define JLOG(list, job, sub, k) do {} while (0)
#define DT0() do {} while (0)
#define DT(i) do {} while (0)
#define DDEP(d) do {} while (0)

__device__ __forceinline__ u64 flag_load(const Ctx& c, u32 foff) {
    return __hip_atomic_load((GAS const u64*)(c.ws + OFF_FLG + c.par * FLG_BYTES + foff), __ATOMIC_RELAXED, __HIP_MEMORY_SCOPE_SYSTEM);
}
__device__ __forceinline__ bool flag_test(const Ctx& c, u64 v, u32& meta) {
    meta = rfl((u32)v);
    return (int)(rfl((u32)(v >> 32)) - c.call) >= 0;
}
__device__ __forceinline__ bool wait_flag(const Ctx& c, u32 foff, u32& meta) {
    GAS u32* tmo = (GAS u32*)(c.ws + OFF_TMO);
    if (flag_test(c, flag_load(c, foff), meta)) return true;
    const u64 t0 = __builtin_amdgcn_s_memrealtime();
    for (;;) {
        __builtin_amdgcn_s_sleep(2);
        if (flag_test(c, flag_load(c, foff), meta)) return true;
        if (rfl(__hip_atomic_load(tmo, __ATOMIC_RELAXED, __HIP_MEMORY_SCOPE_AGENT)) == c.call || __builtin_amdgcn_s_memrealtime() - t0 > DEADLINE) {
            __hip_atomic_store(tmo, c.call, __ATOMIC_RELAXED, __HIP_MEMORY_SCOPE_AGENT);
            meta = 0; return false;
        }
    }
}
__device__ __forceinline__ void post_flag(const Params& P, const Ctx& c, int dst, u32 foff, u32 meta) {
    if ((unsigned)dst >= (unsigned)NR) return;
    GAS u64* f = (GAS u64*)((char*)P.ws[dst] + OFF_FLG + c.par * FLG_BYTES + foff);
    if (c.lane == 0) __hip_atomic_store(f, ((u64)c.call << 32) | meta, __ATOMIC_RELAXED, __HIP_MEMORY_SCOPE_SYSTEM);
}

__device__ __forceinline__ void store_raw(const u32x4 (&d)[8], rsrc_t rs, u32 slot, int lane) {
#pragma unroll
    for (int v = 0; v < 8; ++v) __builtin_amdgcn_raw_buffer_store_b128(d[v], rs, (int)(slot + v * 1024 + lane * 16), 0, AUX_SYS);
}
__device__ __forceinline__ u32 pack_store(const u32x4 (&d)[8], rsrc_t rs, u32 slot, int lane) {
    u32 mx = 0, mn = 0xFFFFFFFFu, mz = 0xFFFFFFFFu;
#pragma unroll
    for (int f = 0; f < 8; ++f)
#pragma unroll
        for (int e = 0; e < 4; ++e) { const u32 a = d[f][e] & 0x7FFF7FFFu; mx = pk_max(mx, a); mn = pk_min(mn, pk_sub_s(a, 0x00010001u)); mz = pk_min(mz, a); }
    const u32 emax = ((mx & 0xFFFFu) > (mx >> 16) ? (mx & 0xFFFFu) : (mx >> 16)) >> 7;
    u32 E = 0;
#pragma unroll
    for (int b = 7; b >= 0; --b) { const u32 cand = E | (1u << b); if (__builtin_amdgcn_ballot_w64(emax >= cand)) E = cand; }
    const u32 lo = E > 15 ? E - 15 : 0, L = lo << 7;
    const u32 mnl = (mn & 0xFFFFu) < (mn >> 16) ? (mn & 0xFFFFu) : (mn >> 16);
    const u32 zero = __builtin_amdgcn_ballot_w64((mz & 0xFFFFu) == 0 || (mz >> 16) == 0) ? META_ZERO : 0u;
    u32 cnt = 0, hdr = 0;
    if (__builtin_amdgcn_ballot_w64(mnl < L)) {
#pragma unroll
        for (int s = 0; s < 64; ++s) {
            if (cnt <= 15) {
                const u32 dw = d[s >> 3][(s >> 1) & 3], val = (s & 1) ? (dw >> 16) : (dw & 0xFFFFu);
                u64 b = __builtin_amdgcn_ballot_w64(((val & 0x7FFFu) - 1u) < L);
                while (b && cnt <= 15) {
                    const int l = __builtin_ctzll(b); b &= b - 1;
                    const u32 ent = ((u32)(l * 64 + s) << 16) | (u32)__builtin_amdgcn_readlane((int)val, l);
                    if ((u32)lane == cnt) hdr = ent;
                    ++cnt;
                }
            }
        }
        if (cnt > 15) { store_raw(d, rs, slot, lane); return META_RAW; }
        if (lane < 16) __builtin_amdgcn_raw_buffer_store_b32(hdr, rs, (int)(slot + HDR + lane * 4), 0, AUX_SYS);
    }
    const u32 LL = rfl(L | (L << 16));
    u32x4 V[6];
#pragma unroll
    for (int f = 0; f < 8; ++f) {
        u32 h[4];
#pragma unroll
        for (int e = 0; e < 4; ++e) { const u32 v = d[f][e]; h[e] = pk_subsat_s(v & 0x7FFF7FFFu, LL) | ((v >> 4) & 0x08000800u); }
        V[f >> 1][(f & 1) * 2 + 0] = __builtin_amdgcn_perm(h[1], h[0], 0x06040200u);
        V[f >> 1][(f & 1) * 2 + 1] = __builtin_amdgcn_perm(h[3], h[2], 0x06040200u);
        V[4 + (f >> 2)][f & 3] = __builtin_amdgcn_perm(h[1], h[0], 0x07050301u) | (__builtin_amdgcn_perm(h[3], h[2], 0x07050301u) << 4);
    }
#pragma unroll
    for (int v = 0; v < 6; ++v) __builtin_amdgcn_raw_buffer_store_b128(V[v], rs, (int)(slot + v * 1024 + lane * 16), 0, AUX_SYS);
    return lo | (cnt << 8) | zero;
}
struct Pull { u32x4 V[6]; u32 h; };
__device__ __forceinline__ void pull_issue(Pull& p, rsrc_t rs, u32 slot, u32 meta, int lane) {
#pragma unroll
    for (int v = 0; v < 6; ++v) p.V[v] = __builtin_amdgcn_raw_buffer_load_b128(rs, (int)(slot + v * 1024 + lane * 16), 0, AUX_SYS);
    const u32 hoff = (meta & 0x1F00u) ? slot + HDR : 0x80000000u;
    p.h = __builtin_amdgcn_raw_buffer_load_b32(rs, (int)(hoff + (lane & 15) * 4), 0, AUX_SYS);
}
__device__ __forceinline__ void unpack(u32x4 (&d)[8], const Pull& p, rsrc_t rs, u32 slot, u32 meta, int lane) {
    if (meta & META_RAW) {
#pragma unroll
        for (int v = 0; v < 6; ++v) d[v] = p.V[v];
        d[6] = __builtin_amdgcn_raw_buffer_load_b128(rs, (int)(slot + 6 * 1024 + lane * 16), 0, AUX_SYS);
        d[7] = __builtin_amdgcn_raw_buffer_load_b128(rs, (int)(slot + 7 * 1024 + lane * 16), 0, AUX_SYS);
        wait_vm0_known();
        return;
    }
    const u32 cnt = (meta >> 8) & 31u;
    const u32 L = (meta & 0xFFu) << 7, LL = L | (L << 16);
#pragma unroll
    for (int f = 0; f < 8; ++f) {
        const u32 B0 = p.V[f >> 1][(f & 1) * 2], B1 = p.V[f >> 1][(f & 1) * 2 + 1], N = p.V[4 + (f >> 2)][f & 3];
        const u32 N0 = (N & 0x07070707u) | ((N << 4) & 0x80808080u), N1 = (N & 0x80808080u) | ((N >> 4) & 0x07070707u);
        d[f][0] = __builtin_amdgcn_perm(N0, B0, 0x05010400u); d[f][1] = __builtin_amdgcn_perm(N0, B0, 0x07030602u);
        d[f][2] = __builtin_amdgcn_perm(N1, B1, 0x05010400u); d[f][3] = __builtin_amdgcn_perm(N1, B1, 0x07030602u);
    }
    if (meta & META_ZERO) {
#pragma unroll
        for (int q = 0; q < 32; ++q) { const u32 h = d[q >> 2][q & 3]; d[q >> 2][q & 3] = pk_mad_s(pk_min_s(h & 0x7FFF7FFFu, 0x00010001u), LL, h); }
    } else {
#pragma unroll
        for (int q = 0; q < 32; ++q) d[q >> 2][q & 3] = pk_add_s(d[q >> 2][q & 3], LL);
    }
    for (u32 i = 0; i < cnt; ++i) {
        const u32 ent = (u32)__builtin_amdgcn_readlane((int)p.h, (int)i);
        const u32 l = ent >> 22, s = (ent >> 16) & 63u, raw = ent & 0xFFFFu;
        const u32 keep = (s & 1) ? 0x0000FFFFu : 0xFFFF0000u, val = (s & 1) ? (raw << 16) : raw;
        const bool me = (u32)lane == l;
#pragma unroll
        for (int q = 0; q < 32; ++q) { const u32 o = d[q >> 2][q & 3]; d[q >> 2][q & 3] = (me && (s >> 1) == (u32)q) ? ((o & keep) | val) : o; }
    }
}
__device__ __forceinline__ void fill_nan(u32x4 (&d)[8]) {
#pragma unroll
    for (int v = 0; v < 8; ++v) d[v] = (u32x4){0x7FC07FC0u, 0x7FC07FC0u, 0x7FC07FC0u, 0x7FC07FC0u};
}
__device__ __forceinline__ void copy_store(const Ctx& c, const Pull& p, rsrc_t rs, u32 slot, u32 dst, u32 meta) {
    const int lane = c.lane;
    if (meta & META_RAW) {
        const u32x4 a = __builtin_amdgcn_raw_buffer_load_b128(rs, (int)(slot + 6 * 1024 + lane * 16), 0, AUX_SYS);
        const u32x4 b = __builtin_amdgcn_raw_buffer_load_b128(rs, (int)(slot + 7 * 1024 + lane * 16), 0, AUX_SYS);
        wait_vm0_known();
        __builtin_amdgcn_raw_buffer_store_b128(a, c.own, (int)(dst + 6 * 1024 + lane * 16), 0, AUX_SYS);
        __builtin_amdgcn_raw_buffer_store_b128(b, c.own, (int)(dst + 7 * 1024 + lane * 16), 0, AUX_SYS);
    } else if ((meta >> 8) & 31u) {
        if (lane < 16) __builtin_amdgcn_raw_buffer_store_b32(p.h, c.own, (int)(dst + HDR + lane * 4), 0, AUX_SYS);
    }
#pragma unroll
    for (int v = 0; v < 6; ++v) __builtin_amdgcn_raw_buffer_store_b128(p.V[v], c.own, (int)(dst + v * 1024 + lane * 16), 0, AUX_SYS);
}

__device__ __forceinline__ u32 ticket_issue(const Ctx& c, u32 off) {
    u32 r; u64 sv; const u32 one = 1u;
    asm volatile("s_mov_b64 %1, exec\n\ts_mov_b64 exec, 1\n\tglobal_atomic_add %0, %2, %3, %4 sc0\n\ts_mov_b64 exec, %1"
                 : "=&v"(r), "=&s"(sv) : "v"(off), "v"(one), "s"(c.ws) : "memory");
    return r;
}
__device__ __forceinline__ u32 ticket_get(u32 r) { asm volatile("" : "+v"(r) :: "memory"); return rfl(r); }

struct Step { u32 foff, slot, meta; int sel, p0, p1, p2, p3; };
template <int SPT, class Gen, class Act, class Pub>
__device__ __forceinline__ void pull_list(Ctx& c, int list, u32 first, u32 nwaves, u32 njobs, rsrc_t rs0, rsrc_t rs1, Gen gen, Act act, Pub pub) {
    DT0();
    const u32 tkoff = OFF_TKT + (u32)list * 128u, tkbase = nwaves - (c.call - 1u) * njobs;
    u32 jc = first, jn = 0, tk = 0;
    if (SPT == 1) tk = ticket_issue(c, tkoff);
    Step cur = gen(jc, 0), prev = cur;
    bool ok = wait_flag(c, cur.foff, cur.meta), have_prev = false, more = true;
    if (SPT == 1) { wait_vm0_known(); jn = ticket_get(tk) + tkbase; }
    DT(1);
    Pull pa, pb;
    JLOG(list, jc, 0, 0);
    pull_issue(pa, cur.sel ? rs1 : rs0, cur.slot, cur.meta, c.lane);
    auto half = [&](const Pull& p, Pull& pn, int sub) {
        const bool same = SPT == 2 && sub == 0;
        more = same || jn < njobs;
        Step nxt = gen(same || !more ? jc : jn, same ? 1 : 0);
        const bool draw = SPT == 1 ? more : same;
        if (draw) tk = ticket_issue(c, tkoff);
        const u64 fv = flag_load(c, nxt.foff);
        wait_vm0_known();
        DT(0);
        JLOG(list, jc, sub, 1);
        const u32 jd = draw ? ticket_get(tk) + tkbase : njobs;
        if (have_prev) pub(prev);
        const bool nready = more && flag_test(c, fv, nxt.meta);
        if (nready) { JLOG(list, same ? jc : jn, same ? 1 : 0, 0); pull_issue(pn, nxt.sel ? rs1 : rs0, nxt.slot, nxt.meta, c.lane); }
        DT(3);
        act(cur, p, cur.sel ? rs1 : rs0, ok);
        prev = cur; have_prev = true; ok = true;
        DT(2);
        if (more && !nready) {
            wait_vm0_known(); pub(prev); have_prev = false;
            ok = wait_flag(c, nxt.foff, nxt.meta);
            JLOG(list, same ? jc : jn, same ? 1 : 0, 0);
            pull_issue(pn, nxt.sel ? rs1 : rs0, nxt.slot, nxt.meta, c.lane);
            DT(1);
        }
        cur = nxt;
        if (SPT == 1) { jc = jn; jn = jd; }
        else if (same) jn = jd; else jc = jn;
    };
    for (;;) {
        half(pa, pb, 0); if (!more) break;
        half(pb, pa, 1); if (!more) break;
    }
    wait_vm0_known();
    if (have_prev) pub(prev);
}

__device__ __forceinline__ int cq_of(int q, int r) {
    return ((r >> 1) ^ ((q & 1) ? (r >> 2) : 0) ^ ((q & 2) ? r : 0)) & 1;
}
__device__ __forceinline__ int link_d(int link) { return (0x765431 >> (4 * link)) & 7; }
__device__ __forceinline__ int link_q(int link, int i) { return ((i ? 0x233231 : 0x110020) >> (4 * link)) & 3; }
__device__ __forceinline__ int link_of_d(int d) { return d == 1 ? 0 : d - 2; }
__device__ __forceinline__ int t_of(int d, int q, int kq) {
    if (d == 0) return kq * 4 + q;
    const int link = link_of_d(d);
    return 16 + (2 * kq + (link_q(link, 0) == q ? 0 : 1)) * 6 + link;
}
__device__ __forceinline__ int d3_pair(int d) { return (d & 1) * 2 + ((d >> 1) & 1); }
__device__ __forceinline__ int d3_of_pair(int p) { return 4 + (p >> 1) + (p & 1) * 2; }
__device__ __forceinline__ int rowtile_of(int rc, int o, int a) {
    const int d = rc ^ o;
    return d == 0 ? 12 + a : d == 2 ? a : 4 + 2 * d3_pair(d) + (a & 1);
}
__device__ __forceinline__ void block_of_rowtile(int rc, int i, int& o, int& a) {
    if (i >= 12) { o = rc; a = i - 12; }
    else if (i < 4) { o = rc ^ 2; a = i; }
    else { const int p = (i - 4) >> 1; o = rc ^ d3_of_pair(p); a = 2 * (p >> 1) + ((i - 4) & 1); }
}
__device__ __forceinline__ u32 raw_frag_off(int type, int sub, int f) {
    return type == 0 ? (u32)(sub * 8192 + f * 1024) : (u32)((f >> 2) * 16384 + sub * 4096 + (f & 3) * 1024);
}
__device__ __forceinline__ void store_operand(const Ctx& c, const u32x4 (&d)[8], int type, int tile, int t, int sub) {
    const u32 base = (type == 0 ? OFF_A : OFF_B) + (u32)(tile * 64 + t) * 32768u + (u32)c.lane * 16u;
#pragma unroll
    for (int f = 0; f < 8; ++f) __builtin_amdgcn_raw_buffer_store_b128(d[f], c.own, (int)(base + raw_frag_off(type, sub, f)), 0, AUX_SYS);
}
__device__ __forceinline__ void count_operand(const Ctx& c, int type, int tile, int t) {
    GAS u32* p = (GAS u32*)(c.ws + (type == 0 ? OFF_RDYA : OFF_RDYB)) + tile * 64 + t;
    if (c.lane == 0) __hip_atomic_fetch_add(p, 1u, __ATOMIC_RELAXED, __HIP_MEMORY_SCOPE_AGENT);
}

__device__ __forceinline__ void pack_job(const Params& P, const Ctx& c, LAS unsigned char* tile, int pj) {
    const int kq = pj >> 10, q = pj & 3, type = (pj >> 2) & 1, g = (pj & 1023) >> 3, kt = 4 * q + kq, lane = c.lane, s = c.rank;
    u32x4 d[8];
    if (type == 0) {
        GAS const char* src = (GAS const char*)P.x + (size_t)(64 * g + (lane & 15)) * 2048 + (64 * kt + 8 * (lane >> 4)) * 2;
#pragma unroll
        for (int f = 0; f < 8; ++f) d[f] = *(GAS const u32x4*)(src + (f >> 1) * 16 * 2048 + (f & 1) * 64);
    } else {
        GAS const char* src = (GAS const char*)P.w + (size_t)(64 * kt + (lane >> 3)) * 16384 + (64 * g) * 2 + (lane & 7) * 16;
#pragma unroll
        for (int v = 0; v < 8; ++v)
            __builtin_amdgcn_global_load_lds((GAS const u32*)(src + (size_t)v * 8 * 16384), (LAS u32*)(tile + v * 1024), 16, 0, 0);
        wait_vm0();
        const int lg = lane >> 4, lq = (lane >> 2) & 3, lp = lane & 3;
        LAS unsigned char* b = tile + (8 * lg + lq) * 128 + lp * 16;
#pragma unroll
        for (int f = 0; f < 8; ++f) {
            const int o = (f & 1) * 32 * 128 + (f >> 2) * 64 + ((f >> 1) & 1) * 8;
            const s16x4 lo4 = __builtin_amdgcn_ds_read_tr16_b64_v4i16((LAS s16x4*)(b + o));
            const s16x4 hi4 = __builtin_amdgcn_ds_read_tr16_b64_v4i16((LAS s16x4*)(b + o + 4 * 128));
            typedef u32 u32x2 __attribute__((ext_vector_type(2)));
            const u32x2 l2 = __builtin_bit_cast(u32x2, lo4), h2 = __builtin_bit_cast(u32x2, hi4);
            d[f] = (u32x4){l2[0], l2[1], h2[0], h2[1]};
        }
        asm volatile("s_waitcnt lgkmcnt(0)" ::: "memory");
    }
    int rc[2], tl[2], tt[2];
    const int cs = cq_of(q, s);
#pragma unroll
    for (int i = 0; i < 2; ++i) {
        int e1, gam;
        if (type == 0) { const int o = g >> 4, h = (g >> 3) & 1; gam = i; e1 = h == 0 ? (o & 1) : ((o ^ (o >> 2) ^ gam) & 1); }
        else { gam = g >> 6; e1 = i; }
        const int e2 = (cs ^ ((q & 2) ? e1 : 0) ^ ((q & 1) ? gam : 0)) & 1;
        rc[i] = e1 | (e2 << 1) | (gam << 2);
        tl[i] = type == 0 ? rowtile_of(rc[i], g >> 4, (g >> 2) & 3) : ((g >> 2) & 15);
        tt[i] = t_of(rc[i] ^ s, q, kq);
    }
    const int sub = g & 3;
    const u32 meta = pack_store(d, c.own, OFF_IN + c.par * IN_BYTES + (u32)(type * 2048 + kt * 128 + g) * SLOT, lane);
    if (rc[0] == s) store_operand(c, d, type, tl[0], tt[0], sub);
    if (rc[1] == s) store_operand(c, d, type, tl[1], tt[1], sub);
    wait_vm0();
#pragma unroll
    for (int i = 0; i < 2; ++i) {
        if (rc[i] == s) count_operand(c, type, tl[i], tt[i]);
        else post_flag(P, c, rc[i], F_IN + (u32)(((type * 16 + tl[i]) * 64 + tt[i]) * 4 + sub) * 8u, meta);
    }
}

__device__ __forceinline__ void in_list(const Params& P, Ctx& c, int link, int widx, int wpl) {
    const int s = c.rank ^ link_d(link);
    auto gen = [&](u32 job, int) {
        const int j = (int)job;
        int kc, type, tile, sub;
        if (j < 896) { kc = j / 112; const int u = j % 112; if (u < 48) { type = 0; tile = u >> 2; } else { type = 1; tile = (u - 48) >> 2; } sub = u & 3; }
        else { const int m = j - 896; kc = m >> 4; type = 0; tile = 12 + ((m >> 2) & 3); sub = m & 3; }
        const int t = 16 + kc * 6 + link, kt = 4 * link_q(link, kc & 1) + (kc >> 1);
        int g;
        if (type == 0) { int o, a; block_of_rowtile(c.rank, tile, o, a); g = o * 16 + a * 4 + sub; } else g = (c.rank >> 2) * 64 + tile * 4 + sub;
        Step st; st.foff = F_IN + (u32)(((type * 16 + tile) * 64 + t) * 4 + sub) * 8u;
        st.slot = OFF_IN + c.par * IN_BYTES + (u32)(type * 2048 + kt * 128 + g) * SLOT;
        st.sel = 0; st.meta = 0; st.p0 = type; st.p1 = tile; st.p2 = t; st.p3 = sub;
        return st;
    };
    auto act = [&](Step& st, const Pull& p, rsrc_t rs, bool ok) {
        u32x4 d[8];
        if (ok) unpack(d, p, rs, st.slot, st.meta, c.lane); else fill_nan(d);
        store_operand(c, d, st.p0, st.p1, st.p2, st.p3);
    };
    auto pub = [&](const Step& st) { count_operand(c, st.p0, st.p1, st.p2); };
    const rsrc_t rs = mk_rsrc(P.ws[s & 7]);
    pull_list<1>(c, link, (u32)widx, (u32)wpl, 1024u, rs, rs, gen, act, pub);
}

__device__ __forceinline__ void acc_add(float (&acc)[64], const u32x4 (&d)[8]) {
#pragma unroll
    for (int q = 0; q < 32; ++q) { const u32 v = d[q >> 2][q & 3]; acc[2 * q] += __builtin_bit_cast(float, v << 16); acc[2 * q + 1] += __builtin_bit_cast(float, v & 0xFFFF0000u); }
}
__device__ __forceinline__ void write_out(const Params& P, const Ctx& c, const float (&acc)[64], int a, int cj, int jb) {
    const int w = jb >> 1, ai = jb & 1, lane = c.lane;
    GAS char* o = (GAS char*)P.out + (size_t)(a * 256 + ai * 128 + (w >> 2) * 64 + (lane & 15)) * 16384 + (cj * 256 + (w & 3) * 64 + 8 * (lane >> 4)) * 2;
#pragma unroll
    for (int f = 0; f < 8; ++f) {
        u32x4 v;
#pragma unroll
        for (int e = 0; e < 4; ++e) v[e] = cvt_pk_bf16(acc[f * 8 + 2 * e], acc[f * 8 + 2 * e + 1]);
        *(GAS u32x4*)(o + (size_t)(f >> 1) * 16 * 16384 + (f & 1) * 64) = v;
    }
}
__device__ __forceinline__ void copy_list(const Params& P, Ctx& c, int grp, int widx, int wpl) {
    const int r = c.rank;
    auto gen = [&](u32 job, int) {
        const int j = (int)job, jb = j & 15;
        Step st; st.meta = 0;
        if (grp == 0) {
            const int u = j >> 4, pt = (u & 3) * 16 + (u >> 2) * 2;
            st.foff = F_P + (u32)(pt * 16 + jb) * 8u; st.slot = OFF_OUT + c.par * OUT_BYTES + (u32)(pt * 16 + jb) * SLOT;
            st.p0 = (int)(OFF_OWN + (u32)((pt * 4 + 3) * 16 + jb) * SLOT); st.p1 = r; st.p2 = (int)(F_OWN + (u32)((pt * 4 + 3) * 16 + jb) * 8u);
        } else {
            const int second = j >= 256, v = (j >> 4) & 15, u = (v & 3) * 4 + (v >> 2), path = (grp - 1) ^ second, pt = 4 * u + 1 + 2 * path, rs = path * 16 + u;
            if (!second) {
                st.foff = F_R + (u32)(rs * 16 + jb) * 8u; st.slot = OFF_OUT + c.par * OUT_BYTES + (u32)(pt * 16 + jb) * SLOT;
                st.p0 = (int)(OFF_REL + c.par * REL_BYTES + (u32)(rs * 16 + jb) * SLOT); st.p1 = r ^ (path ? 1 : 3); st.p2 = (int)(F_P + (u32)(pt * 16 + jb) * 8u);
            } else {
                st.foff = F_P + (u32)(pt * 16 + jb) * 8u; st.slot = OFF_REL + c.par * REL_BYTES + (u32)(rs * 16 + jb) * SLOT;
                st.p0 = (int)(OFF_OWN + (u32)((pt * 4 + 3) * 16 + jb) * SLOT); st.p1 = r; st.p2 = (int)(F_OWN + (u32)((pt * 4 + 3) * 16 + jb) * 8u);
            }
        }
        st.p3 = 0; st.sel = 0;
        return st;
    };
    auto act = [&](Step& st, const Pull& p, rsrc_t rs, bool ok) {
        if (ok) copy_store(c, p, rs, st.slot, (u32)st.p0, st.meta);
        else { u32x4 d[8]; fill_nan(d); store_raw(d, c.own, (u32)st.p0, c.lane); st.meta = META_RAW; }
    };
    auto pub = [&](const Step& st) { post_flag(P, c, st.p1, (u32)st.p2, st.meta); };
    const rsrc_t rs = mk_rsrc(P.ws[(r ^ (grp == 0 ? 2 : grp == 1 ? 1 : 3)) & 7]);
    pull_list<1>(c, 6 + grp, (u32)widx, (u32)wpl, 512u, rs, rs, gen, act, pub);
}
__device__ __forceinline__ void reduce_list(const Params& P, Ctx& c, int h, int widx, int wpl) {
    const int r = c.rank;
    float acc[64];
    auto gen = [&](u32 ujob, int sub) {
        const int job = (int)ujob, sl = (sub ^ job) & 1, jb = job & 15, ot2 = job >> 4;
        const int a = 2 * h + (ot2 & 1), j = ot2 >> 1, ot = a * 16 + j, p = 2 * h + sl, ti = 4 + 2 * p + (a & 1);
        Step st; st.meta = 0;
        st.foff = F_D3 + (u32)((ot * 2 + sl) * 16 + jb) * 8u; st.sel = sl;
        st.slot = OFF_OUT + c.par * OUT_BYTES + (u32)((ti * 16 + j) * 16 + jb) * SLOT;
        st.p0 = sub; st.p1 = a; st.p2 = (1 - (r >> 2)) * 16 + j; st.p3 = jb;
        return st;
    };
    auto act = [&](Step& st, const Pull& p, rsrc_t rs, bool ok) {
        u32x4 d[8];
        if (ok) unpack(d, p, rs, st.slot, st.meta, c.lane); else fill_nan(d);
        if (st.p0 == 0) {
#pragma unroll
            for (int q = 0; q < 64; ++q) acc[q] = 0.f;
        }
        acc_add(acc, d);
        if (st.p0 == 1) write_out(P, c, acc, st.p1, st.p2, st.p3);
    };
    auto pub = [&](const Step&) {};
    pull_list<2>(c, 9 + h, (u32)widx, (u32)wpl, 512u, mk_rsrc(P.ws[(r ^ d3_of_pair(2 * h)) & 7]), mk_rsrc(P.ws[(r ^ d3_of_pair(2 * h + 1)) & 7]), gen, act, pub);
}
__device__ __forceinline__ void own_job(const Params& P, const Ctx& c, int n) {
    const int pt = n >> 4, jb = n & 15;
    float acc[64];
#pragma unroll
    for (int i = 0; i < 64; ++i) acc[i] = 0.f;
    for (int part = 0; part < 4; ++part) {
        u32 meta; u32x4 d[8];
        if (wait_flag(c, F_OWN + (u32)((pt * 4 + part) * 16 + jb) * 8u, meta)) {
            Pull p; const u32 slot = OFF_OWN + (u32)((pt * 4 + part) * 16 + jb) * SLOT;
            pull_issue(p, c.own, slot, meta, c.lane); unpack(d, p, c.own, slot, meta, c.lane);
        } else fill_nan(d);
        acc_add(acc, d);
    }
    write_out(P, c, acc, pt >> 4, (c.rank >> 2) * 16 + (pt & 15), jb);
}

struct GJob { int ti, tj, t0, nt; };

#define STAMP(i) do {} while (0)

__device__ __forceinline__ void gemm_role(const Params& P, const Ctx& c, LAS unsigned char* lds, int xcd, int idx) {
    const int tid = threadIdx.x, wid = (int)rfl(tid >> 6), lane = c.lane, wr = wid >> 2, wc = wid & 3;
    constexpr int HTB = 16384;
    const u32 voff = (u32)tid * 16u;
    const unsigned ldsw = (unsigned)wid * 1024u;
    const int aoff = wr * 8192 + lane * 16, boff = wc * 4096 + lane * 16;
    LAS volatile int* ctl = (LAS volatile int*)(lds + LDS_CTL);
    int avail = 0; bool bad = false;
#define G_SA(b, h) (((b) * 2 + (h)) * HTB)
#define G_SB(b, h) ((4 + (b) * 2 + (h)) * HTB)
#define G_STAGE(bufoff, gbase) do { _Pragma("unroll") for (int _i = 0; _i < 2; ++_i) \
        __builtin_amdgcn_global_load_lds((GAS const u32*)((GAS const char*)(gbase) + voff + _i * 8192), (LAS u32*)(lds + (bufoff) + ldsw + _i * 8192), 16, 0, AUX_DEV); } while (0)
#define G_LDA(dst, b, h) do { _Pragma("unroll") for (int m = 0; m < 4; ++m) _Pragma("unroll") for (int k = 0; k < 2; ++k) dst[m][k] = *(const LAS bf16x8*)(lds + G_SA(b, h) + aoff + m * 2048 + k * 1024); } while (0)
#define G_LDB(dst, b, h) do { _Pragma("unroll") for (int n = 0; n < 2; ++n) _Pragma("unroll") for (int k = 0; k < 2; ++k) dst[n][k] = *(const LAS bf16x8*)(lds + G_SB(b, h) + boff + n * 2048 + k * 1024); } while (0)
#define G_MMA(ai, bj, At, Bt) do { __builtin_amdgcn_s_setprio(1); _Pragma("unroll") for (int m = 0; m < 4; ++m) _Pragma("unroll") for (int n = 0; n < 2; ++n) _Pragma("unroll") for (int k = 0; k < 2; ++k) \
        acc[ai][bj][m][n] = __builtin_amdgcn_mfma_f32_16x16x32_bf16(Bt[n][k], At[m][k], acc[ai][bj][m][n], 0, 0, 0); __builtin_amdgcn_s_setprio(0); } while (0)
#define G_WAIT_V(n) asm volatile("s_waitcnt vmcnt(" #n ")" ::: "memory")
#define G_WAIT_L(n) asm volatile("s_waitcnt lgkmcnt(" #n ")" ::: "memory")
#define G_BAR __builtin_amdgcn_s_barrier()
#define G_SCHED __builtin_amdgcn_sched_barrier(0)
    auto ensure = [&](const GJob& j, int tneed) {
        if (avail >= tneed) return;
        if (wid == 0) {
            GAS const u32* ra = (GAS const u32*)(c.ws + OFF_RDYA) + j.ti * 64 + lane;
            GAS const u32* rb = (GAS const u32*)(c.ws + OFF_RDYB) + j.tj * 64 + lane;
            GAS u32* tmo = (GAS u32*)(c.ws + OFF_TMO);
            const u32 target = 4u * c.call;
            const u64 below = j.t0 ? ((1ull << j.t0) - 1ull) : 0ull;
            const u64 t0 = __builtin_amdgcn_s_memrealtime();
            int av, bd = 0;
            for (;;) {
                const u32 a = __hip_atomic_load(ra, __ATOMIC_RELAXED, __HIP_MEMORY_SCOPE_AGENT), b = __hip_atomic_load(rb, __ATOMIC_RELAXED, __HIP_MEMORY_SCOPE_AGENT);
                const u64 ok = __builtin_amdgcn_ballot_w64((int)(a - target) >= 0 && (int)(b - target) >= 0) | below;
                av = ok == ~0ull ? 64 : __builtin_ctzll(~ok);
                if (av >= tneed) break;
                if (rfl(__hip_atomic_load(tmo, __ATOMIC_RELAXED, __HIP_MEMORY_SCOPE_AGENT)) == c.call || __builtin_amdgcn_s_memrealtime() - t0 > DEADLINE) {
                    __hip_atomic_store(tmo, c.call, __ATOMIC_RELAXED, __HIP_MEMORY_SCOPE_AGENT); av = 64; bd = 1; break;
                }
                __builtin_amdgcn_s_sleep(4);
            }
            if (lane == 0) { ctl[0] = av; if (bd) ctl[1] = 1; }
            G_WAIT_L(0);
        }
        asm volatile("" ::: "memory"); G_BAR; asm volatile("" ::: "memory");
        avail = (int)rfl((u32)ctl[0]); bad = rfl((u32)ctl[1]) != 0;
    };
    GJob cur, nxt;
    cur.ti = 6 * (xcd & 1) + idx % 6; cur.tj = 4 * (xcd >> 1) + idx / 6; cur.t0 = 0; cur.nt = 64;
    { const int col = xcd * 6 + (idx >> 2), third = col >> 4; nxt.ti = 12 + (idx & 3); nxt.tj = col & 15; nxt.t0 = 22 * third; nxt.nt = third == 2 ? 20 : 22; }
    f32x4 acc[2][2][4][2];
    bf16x8 At[4][2], B0[2][2], B1[2][2];
    constexpr size_t kstep = 32768, hstep = 16384;
    for (int ui = 0; ui < 2; ++ui) {
        const bool has_next = ui == 0;
#pragma unroll
        for (int a = 0; a < 2; ++a)
#pragma unroll
            for (int b = 0; b < 2; ++b)
#pragma unroll
                for (int m = 0; m < 4; ++m)
#pragma unroll
                    for (int n = 0; n < 2; ++n) acc[a][b][m][n] = (f32x4){0.f, 0.f, 0.f, 0.f};
        const char* cA = c.ws + OFF_A + (size_t)(cur.ti * 64 + cur.t0) * kstep;
        const char* cB = c.ws + OFF_B + (size_t)(cur.tj * 64 + cur.t0) * kstep;
        avail = 0;
        ensure(cur, cur.t0 + 2);
        G_STAGE(G_SB(0, 0), cB); G_STAGE(G_SA(0, 0), cA); G_STAGE(G_SB(0, 1), cB + hstep); G_STAGE(G_SA(0, 1), cA + hstep);
        if (wr == 1) G_BAR;
        G_WAIT_V(4); G_BAR;
        G_STAGE(G_SB(1, 0), cB + kstep); G_STAGE(G_SA(1, 0), cA + kstep); G_STAGE(G_SB(1, 1), cB + hstep + kstep);
        G_WAIT_V(6); G_BAR;
        if (ui == 0) STAMP(2);
        const int nt = cur.nt;
        for (int t = 0; t < nt; t += 2) {
            const bool last = (t == nt - 2);
            const char* a1 = cA + (size_t)(t + 1) * kstep;
            const char* a2 = cA + (size_t)(last ? t : t + 2) * kstep; const char* b2 = cB + (size_t)(last ? t : t + 2) * kstep;
            const char* a3 = a2 + kstep; const char* b3 = b2 + kstep;
            if (!last) ensure(cur, cur.t0 + t + 4);
            G_LDB(B0, 0, 0); G_SCHED; G_LDA(At, 0, 0); G_STAGE(G_SA(1, 1), a1 + hstep);
            G_WAIT_L(8); G_BAR; G_WAIT_L(0); G_MMA(0, 0, At, B0); G_BAR; G_SCHED;
            G_LDB(B1, 0, 1); G_STAGE(G_SB(0, 0), b2);
            G_BAR; G_WAIT_L(0); G_MMA(0, 1, At, B1); G_BAR;
            G_LDA(At, 0, 1); G_STAGE(G_SA(0, 0), a2);
            G_BAR; G_WAIT_L(0); G_MMA(1, 0, At, B0); G_BAR; G_SCHED;
            G_STAGE(G_SB(0, 1), b2 + hstep);
            G_WAIT_V(6); G_BAR; G_MMA(1, 1, At, B1); G_BAR;
            G_LDB(B0, 1, 0); G_SCHED; G_LDA(At, 1, 0); G_STAGE(G_SA(0, 1), a2 + hstep);
            G_WAIT_L(8); G_BAR; G_WAIT_L(0); G_MMA(0, 0, At, B0); G_BAR; G_SCHED;
            G_LDB(B1, 1, 1); G_STAGE(G_SB(1, 0), b3);
            G_BAR; G_WAIT_L(0); G_MMA(0, 1, At, B1); G_BAR;
            G_LDA(At, 1, 1); G_STAGE(G_SA(1, 0), a3);
            G_BAR; G_WAIT_L(0); G_MMA(1, 0, At, B0); G_BAR; G_SCHED;
            G_STAGE(G_SB(1, 1), b3 + hstep);
            G_WAIT_V(6); G_BAR; G_MMA(1, 1, At, B1); G_BAR;
        }
        if (wr == 0) G_BAR;
        STAMP(3 + 2 * ui);
        {
            u32 meta[2], slot[2];
#pragma unroll
            for (int ai = 0; ai < 2; ++ai) {
                u32x4 d[8];
#pragma unroll
                for (int m = 0; m < 4; ++m)
#pragma unroll
                    for (int bj = 0; bj < 2; ++bj) {
                        const f32x4 v0 = acc[ai][bj][m][0], v1 = acc[ai][bj][m][1];
                        d[m * 2 + bj] = (u32x4){cvt_pk_bf16(v0[0], v0[1]), cvt_pk_bf16(v0[2], v0[3]), cvt_pk_bf16(v1[0], v1[1]), cvt_pk_bf16(v1[2], v1[3])};
                    }
                if (bad) fill_nan(d);
                const int jb = wid * 2 + ai;
                if (has_next) { slot[ai] = OFF_OUT + c.par * OUT_BYTES + (u32)((cur.ti * 16 + cur.tj) * 16 + jb) * SLOT; meta[ai] = pack_store(d, c.own, slot[ai], lane); }
                else { slot[ai] = OFF_OWN + (u32)((((cur.ti - 12) * 16 + cur.tj) * 4 + (cur.t0 != 0) + (cur.t0 == 44)) * 16 + jb) * SLOT; store_raw(d, c.own, slot[ai], lane); meta[ai] = META_RAW; }
            }
            wait_vm0();
#pragma unroll
            for (int ai = 0; ai < 2; ++ai) {
                const int jb = wid * 2 + ai, r = c.rank;
                int dst; u32 foff;
                if (!has_next) { dst = r; foff = F_OWN + (u32)((((cur.ti - 12) * 16 + cur.tj) * 4 + (cur.t0 != 0) + (cur.t0 == 44)) * 16 + jb) * 8u; }
                else if (cur.ti < 4) {
                    const int pt = cur.ti * 16 + cur.tj;
                    if ((pt & 1) == 0) { dst = r ^ 2; foff = F_P + (u32)(pt * 16 + jb) * 8u; }
                    else { const int path = (pt >> 1) & 1; dst = r ^ (path ? 3 : 1); foff = F_R + (u32)((path * 16 + (pt >> 2)) * 16 + jb) * 8u; }
                } else {
                    const int p = (cur.ti - 4) >> 1, a = 2 * (p >> 1) + ((cur.ti - 4) & 1);
                    dst = r ^ d3_of_pair(p); foff = F_D3 + (u32)(((a * 16 + cur.tj) * 2 + (p & 1)) * 16 + jb) * 8u;
                }
                post_flag(P, c, dst, foff, meta[ai]);
            }
        }
        STAMP(4 + 2 * ui);
        cur = nxt;
    }
#undef G_SA
#undef G_SB
#undef G_STAGE
#undef G_LDA
#undef G_LDB
#undef G_MMA
#undef G_WAIT_V
#undef G_WAIT_L
#undef G_BAR
#undef G_SCHED
}

__global__ __launch_bounds__(NTH, 2) void gemm_rs_kernel(const Params P) {
    extern __shared__ __attribute__((aligned(16))) unsigned char shm[];
    LAS unsigned char* lds = (LAS unsigned char*)shm;
    const int wg = blockIdx.x, xcd = wg & 7, idx = wg >> 3, tid = threadIdx.x, wid = (int)rfl(tid >> 6);
    Ctx c;
    c.rank = P.rank & 7; c.lane = tid & 63;
    c.ws = (char*)P.ws[c.rank]; c.own = mk_rsrc(c.ws);
    LAS volatile u32* ctl = (LAS volatile u32*)(lds + LDS_CTL);
    if (tid == 0) {
        ctl[2] = __hip_atomic_fetch_add((GAS u32*)(c.ws + OFF_CALL + wg * 64), 1u, __ATOMIC_RELAXED, __HIP_MEMORY_SCOPE_AGENT) + 1u;
        ctl[0] = 0; ctl[1] = 0;
    }
    __syncthreads();
    c.call = rfl(ctl[2]); c.par = c.call & 1u;
    STAMP(0);
    if (idx < NGEMM / 8) {
        const int gw = (xcd * (NGEMM / 8) + idx) * 8 + wid;
        for (int pj = gw; pj < 4096; pj += GEMM_WAVES) pack_job(P, c, lds + wid * 8192, pj);
        STAMP(1);
        __syncthreads();
        gemm_role(P, c, lds, xcd, idx);
        if (gw < 1024) own_job(P, c, gw);
        STAMP(7);
    } else {
        const int cw = wid * NCOMM + (idx - NGEMM / 8) * 8 + xcd;
        if (cw >= 6 * IN_WPL && cw >= O_WAVES) return;
#define GR_INMASK 63
        if (cw < 6 * IN_WPL && ((GR_INMASK >> (cw % 6)) & 1)) in_list(P, c, cw % 6, cw / 6, IN_WPL);
        STAMP(3);
        if (cw < 3 * C_WPL) copy_list(P, c, cw % 3, cw / 3, C_WPL);
        else if (cw < O_WAVES) reduce_list(P, c, (cw - 3 * C_WPL) & 1, (cw - 3 * C_WPL) >> 1, 2 * X_WPL);
        STAMP(7);
    }
}
}

extern "C" void kernel_launch(int rank, int nranks, ncclComm_t comm, hipStream_t stream,
                              void* const* d_in, const int* in_sizes, int n_in,
                              void* d_out, int out_size,
                              const int* in_dtypes, int out_dtype,
                              void* const* d_out_all,
                              void* const* d_ws_all, size_t ws_bytes) {
    (void)comm; (void)in_sizes; (void)n_in; (void)out_size; (void)in_dtypes; (void)out_dtype; (void)d_out_all;
    if (nranks != gr::NR || rank < 0 || rank >= gr::NR || ws_bytes < gr::WS_NEED) { fprintf(stderr, "kernel_launch: unexpected configuration\n"); return; }
    static bool init[gr::NR] = {};
    if (!init[rank]) {
        if (hipFuncSetAttribute((const void*)gr::gemm_rs_kernel, hipFuncAttributeMaxDynamicSharedMemorySize, gr::LDS_BYTES) != hipSuccess) { fprintf(stderr, "hipFuncSetAttribute failed\n"); return; }
        init[rank] = true;
    }
    gr::Params P;
    P.x = d_in[0]; P.w = d_in[1]; P.out = d_out; P.rank = rank; P.pad = 0;
    for (int i = 0; i < gr::NR; ++i) P.ws[i] = d_ws_all[i];
    hipLaunchKernelGGL(gr::gemm_rs_kernel, dim3(gr::NWG), dim3(gr::NTH), gr::LDS_BYTES, stream, P);
    hipError_t e = hipGetLastError();
    if (e != hipSuccess) fprintf(stderr, "launch failed: %s\n", hipGetErrorString(e));
}
```

```cpp
#include <hip/hip_runtime.h>
#include <rccl/rccl.h>
#include <cstdio>

namespace gr {
typedef unsigned u32;
typedef unsigned long long u64;
typedef unsigned short u16;
typedef u32 u32x4 __attribute__((ext_vector_type(4)));
typedef u32 u32x2 __attribute__((ext_vector_type(2)));
typedef u16 u16x2 __attribute__((ext_vector_type(2)));
typedef short bf16x8 __attribute__((ext_vector_type(8)));
typedef short s16x4 __attribute__((ext_vector_type(4)));
typedef float f32x4 __attribute__((ext_vector_type(4)));
#define LAS __attribute__((address_space(3)))
#define GAS __attribute__((address_space(1)))
typedef __amdgpu_buffer_rsrc_t rsrc_t;

constexpr int NR = 8;
constexpr int NWG = 256, NTH = 512, NGEMM = 192, NCOMM = NWG - NGEMM;
constexpr int GEMM_WAVES = NGEMM * 8, COMM_WAVES = NCOMM * 8;
constexpr u32 SLOT = 8448;
constexpr u32 PAY_Q2 = 5120, PAY_X = 5632;
constexpr u32 XMAX = 128;
constexpr int KP_LANE = 80, KP_BYTES = 64 * KP_LANE;
constexpr u64 DEADLINE = 100000000ull;

constexpr u32 OFF_CALL = 0;
constexpr u32 OFF_TMO = 16384;
constexpr u32 OFF_RDYA = 20480;
constexpr u32 OFF_RDYB = 24576;
constexpr u32 OFF_TKT = 28672;
constexpr u32 OFF_FLG = 32768;
constexpr u32 F_IN = 0;
constexpr u32 F_D3 = F_IN + 2 * 16 * 64 * 4 * 8;
constexpr u32 F_P = F_D3 + 64 * 2 * 16 * 8;
constexpr u32 F_R = F_P + 64 * 16 * 8;
constexpr u32 F_OWN = F_R + 32 * 16 * 8;
constexpr u32 FLG_BYTES = F_OWN + 64 * 4 * 16 * 8;
constexpr u32 OFF_DBG = 512 * 1024;
constexpr u32 CTL_BYTES = 1024 * 1024;
static_assert(OFF_FLG + 2 * FLG_BYTES <= OFF_DBG, "control region");
constexpr u32 IN_BYTES = 4096 * SLOT;
constexpr u32 OUT_BYTES = 192 * 16 * SLOT;
constexpr u32 REL_BYTES = 32 * 16 * SLOT;
constexpr u32 OFF_IN = CTL_BYTES;
constexpr u32 OFF_OUT = OFF_IN + 2 * IN_BYTES;
constexpr u32 OFF_REL = OFF_OUT + 2 * OUT_BYTES;
constexpr u32 OFF_OWN = OFF_REL + 2 * REL_BYTES;
constexpr u32 OFF_A = OFF_OWN + 64 * 4 * 16 * SLOT;
constexpr u32 OFF_B = OFF_A + 16 * 64 * 32768;
constexpr u32 WS_NEED = OFF_B + 16 * 64 * 32768;
static_assert(WS_NEED <= 257u * 1024u * 1024u, "workspace");
static_assert(OFF_A % 256 == 0 && OFF_OUT % 128 == 0 && OFF_OWN % 128 == 0, "alignment");

constexpr u32 META_RAW = 1u << 16;
#define GR_IN_WPL 16
#define GR_C_WPL 15
#define GR_X_WPL 15
constexpr int IN_WPL = GR_IN_WPL, C_WPL = GR_C_WPL, X_WPL = GR_X_WPL;
constexpr int O_WAVES = 3 * C_WPL + 4 * X_WPL;
#define GR_GAUX 16
constexpr int AUX_SYS = 17, AUX_DEV = GR_GAUX;
constexpr int LDS_STAGE = 131072, LDS_CTL = LDS_STAGE, LDS_BYTES = LDS_STAGE + 256;

struct Params {
    const void* x;
    const void* w;
    void* out;
    void* ws[NR];
    int rank;
    int pad;
};

__device__ __forceinline__ u32 rfl(u32 v) { return (u32)__builtin_amdgcn_readfirstlane((int)v); }
__device__ __forceinline__ u32 pk_add_s(u32 a, u32 s) { u32 r; asm("v_pk_add_u16 %0, %1, %2" : "=v"(r) : "v"(a), "s"(s)); return r; }
__device__ __forceinline__ u32 pk_sub_s(u32 a, u32 s) { u32 r; asm("v_pk_sub_u16 %0, %1, %2" : "=v"(r) : "v"(a), "s"(s)); return r; }
__device__ __forceinline__ u32 bfi(u32 m, u32 a, u32 b) { u32 r; asm("v_bfi_b32 %0, %1, %2, %3" : "=v"(r) : "s"(m), "v"(a), "v"(b)); return r; }
__device__ __forceinline__ u32 rotl(u32 x, int r) { return __builtin_amdgcn_alignbit(x, x, (u32)(32 - r)); }
__device__ __forceinline__ u32 rotr(u32 x, int r) { return __builtin_amdgcn_alignbit(x, x, (u32)r); }
__device__ __forceinline__ u32 ffbl(u32 x) { u32 r; asm("v_ffbl_b32 %0, %1" : "=v"(r) : "v"(x)); return r; }
__device__ __forceinline__ u32 wave_scan(u32 x) {
    x += (u32)__builtin_amdgcn_update_dpp(0, (int)x, 0x111, 0xF, 0xF, false);
    x += (u32)__builtin_amdgcn_update_dpp(0, (int)x, 0x112, 0xF, 0xF, false);
    x += (u32)__builtin_amdgcn_update_dpp(0, (int)x, 0x114, 0xF, 0xF, false);
    x += (u32)__builtin_amdgcn_update_dpp(0, (int)x, 0x118, 0xF, 0xF, false);
    x += (u32)__builtin_amdgcn_update_dpp(0, (int)x, 0x142, 0xA, 0xF, false);
    x += (u32)__builtin_amdgcn_update_dpp(0, (int)x, 0x143, 0xC, 0xF, false);
    return x;
}
__device__ __forceinline__ u32 cvt_pk_bf16(float lo, float hi) { u32 r; asm volatile("v_cvt_pk_bf16_f32 %0, %1, %2" : "=v"(r) : "v"(lo), "v"(hi)); return r; }
__device__ __forceinline__ rsrc_t mk_rsrc(void* p) { return __builtin_amdgcn_make_buffer_rsrc(p, (short)0, (int)WS_NEED, 0x00020000); }
__device__ __forceinline__ void wait_vm0() { asm volatile("s_waitcnt vmcnt(0)" ::: "memory"); }
__device__ __forceinline__ void wait_vm0_known() { __builtin_amdgcn_s_waitcnt(0x0F70); asm volatile("" ::: "memory"); }

struct Ctx {
    char* ws;
    rsrc_t own;
    u32 call, par;
    int rank, lane;
    LAS unsigned char* kp;
};
#define JLOG(list, job, sub, k) do {} while (0)
#define DT0() do {} while (0)
#define DT(i) do {} while (0)
#define DDEP(d) do {} while (0)

__device__ __forceinline__ u64 flag_load(const Ctx& c, u32 foff) {
    return __hip_atomic_load((GAS const u64*)(c.ws + OFF_FLG + c.par * FLG_BYTES + foff), __ATOMIC_RELAXED, __HIP_MEMORY_SCOPE_SYSTEM);
}
__device__ __forceinline__ bool flag_test(const Ctx& c, u64 v, u32& meta) {
    meta = rfl((u32)v);
    return (int)(rfl((u32)(v >> 32)) - c.call) >= 0;
}
__device__ __forceinline__ bool wait_flag(const Ctx& c, u32 foff, u32& meta) {
    GAS u32* tmo = (GAS u32*)(c.ws + OFF_TMO);
    if (flag_test(c, flag_load(c, foff), meta)) return true;
    const u64 t0 = __builtin_amdgcn_s_memrealtime();
    for (;;) {
        __builtin_amdgcn_s_sleep(2);
        if (flag_test(c, flag_load(c, foff), meta)) return true;
        if (rfl(__hip_atomic_load(tmo, __ATOMIC_RELAXED, __HIP_MEMORY_SCOPE_AGENT)) == c.call || __builtin_amdgcn_s_memrealtime() - t0 > DEADLINE) {
            __hip_atomic_store(tmo, c.call, __ATOMIC_RELAXED, __HIP_MEMORY_SCOPE_AGENT);
            meta = 0; return false;
        }
    }
}
__device__ __forceinline__ void post_flag(const Params& P, const Ctx& c, int dst, u32 foff, u32 meta) {
    if ((unsigned)dst >= (unsigned)NR) return;
    GAS u64* f = (GAS u64*)((char*)P.ws[dst] + OFF_FLG + c.par * FLG_BYTES + foff);
    if (c.lane == 0) __hip_atomic_store(f, ((u64)c.call << 32) | meta, __ATOMIC_RELAXED, __HIP_MEMORY_SCOPE_SYSTEM);
}

__device__ __forceinline__ void store_raw(const u32x4 (&d)[8], rsrc_t rs, u32 slot, int lane) {
#pragma unroll
    for (int v = 0; v < 8; ++v) __builtin_amdgcn_raw_buffer_store_b128(d[v], rs, (int)(slot + v * 1024 + lane * 16), 0, AUX_SYS);
}
__device__ __forceinline__ u32 kz_of(u32 L) { return ((0x8000u - L) & 0x7FFFu) >> 10; }
__device__ __forceinline__ u32 pack_store(const u32x4 (&d)[8], rsrc_t rs, u32 slot, int lane, LAS unsigned char* stage) {
    float m0 = 0.f, m1 = 0.f;
#pragma unroll
    for (int f = 0; f < 8; ++f) {
        asm("v_max3_f32 %0, |%1|, |%2|, %0" : "+v"(m0) : "v"(d[f][0]), "v"(d[f][1]));
        asm("v_max3_f32 %0, |%1|, |%2|, %0" : "+v"(m1) : "v"(d[f][2]), "v"(d[f][3]));
    }
    const u32 b0 = __builtin_bit_cast(u32, m0), b1 = __builtin_bit_cast(u32, m1);
    const u32 sum = (u32)__builtin_amdgcn_readlane((int)wave_scan(((b0 > b1 ? b0 : b1) >> 16) & 0x7FFFu), 63);
    const u32 top = ((sum >> 6) + 32u) >> 7, lo = top > 7u ? top - 7u : 0u, L = lo << 7, LL = L | (L << 16);
    u32 lb[16], hb[16], M[2] = {0u, 0u};
#pragma unroll
    for (int j = 0; j < 16; ++j) {
        const u32 w0 = pk_sub_s(d[j >> 1][(j & 1) * 2], LL), w1 = pk_sub_s(d[j >> 1][(j & 1) * 2 + 1], LL);
        lb[j] = __builtin_amdgcn_perm(w1, w0, 0x06040200u); hb[j] = __builtin_amdgcn_perm(w1, w0, 0x07050301u);
        M[j >> 3] = bfi(0x80808080u, (hb[j] & 0x7C7C7C7Cu) + 0x7C7C7C7Cu, M[j >> 3] >> 1);
    }
    const u32 c0 = (u32)__builtin_popcount(M[0]), c1 = (u32)__builtin_popcount(M[1]), incl = wave_scan(c0 + c1);
    const u32 cnt = (u32)__builtin_amdgcn_readlane((int)incl, 63);
    if (cnt > XMAX) { store_raw(d, rs, slot, lane); return META_RAW; }
    u32 Q[6];
#pragma unroll
    for (int g = 0; g < 2; ++g) {
        const u32* h = hb + 8 * g;
        Q[3 * g + 0] = bfi(0x60606060u, rotl(h[6], 6), bfi(0x1C1C1C1Cu, rotl(h[1], 3), h[0]));
        Q[3 * g + 1] = bfi(0x40404040u, rotl(h[7], 5), bfi(0x20202020u, rotl(h[6], 4), bfi(0x1C1C1C1Cu, rotl(h[3], 3), h[2])));
        Q[3 * g + 2] = bfi(0x60606060u, rotl(h[7], 6), bfi(0x1C1C1C1Cu, rotl(h[5], 3), h[4]));
    }
#pragma unroll
    for (int v = 0; v < 4; ++v)
        __builtin_amdgcn_raw_buffer_store_b128((u32x4){lb[4 * v], lb[4 * v + 1], lb[4 * v + 2], lb[4 * v + 3]}, rs, (int)(slot + v * 1024 + lane * 16), 0, AUX_SYS);
    __builtin_amdgcn_raw_buffer_store_b128((u32x4){Q[0], Q[1], Q[2], Q[3]}, rs, (int)(slot + 4096 + lane * 16), 0, AUX_SYS);
    __builtin_amdgcn_raw_buffer_store_b64((u32x2){Q[4], Q[5]}, rs, (int)(slot + PAY_Q2 + lane * 8), 0, AUX_SYS);
    if (cnt) {
        LAS unsigned char* mine = stage + lane * KP_LANE;
        LAS u16* list = (LAS u16*)(stage + KP_BYTES);
#pragma unroll
        for (int i = 0; i < 4; ++i) *(LAS u32x4*)(mine + 16 * i) = (u32x4){hb[4 * i], hb[4 * i + 1], hb[4 * i + 2], hb[4 * i + 3]};
        const u32 kz = kz_of(L);
        u32 off[2] = {incl - c0 - c1, incl - c1};
        bool bad = false;
        do {
#pragma unroll
            for (int g = 0; g < 2; ++g) {
                const bool has = M[g] != 0u;
                const u32 p = ffbl(M[g]), pos = (u32)(g * 32) + ((p & 7u) << 2) + ((p >> 3) & 3u);
                M[g] &= M[g] - 1u;
                const u32 k5 = ((u32)mine[pos] >> 2) & 31u;
                const bool z = k5 == kz;
                bad |= has && !z && ((k5 + 7u) & 31u) >= 15u;
                list[has ? off[g] : XMAX] = (u16)((((u32)lane * 64u + pos) << 4) | (z ? 8u : k5 & 15u));
                off[g] += has ? 1u : 0u;
            }
        } while (__builtin_amdgcn_ballot_w64((M[0] | M[1]) != 0u));
        if (__builtin_amdgcn_ballot_w64(bad)) { wait_vm0_known(); store_raw(d, rs, slot, lane); return META_RAW; }
        const u32 e = ((LAS const u32*)list)[lane];
        if (2u * (u32)lane < cnt) __builtin_amdgcn_raw_buffer_store_b32(e, rs, (int)(slot + PAY_X + lane * 4), 0, AUX_SYS);
    }
    return lo | (cnt << 8);
}
struct Pull { u32x4 V[5]; u32x2 q; u32 h; };
__device__ __forceinline__ void pull_issue(Pull& p, rsrc_t rs, u32 slot, u32 meta, int lane) {
#pragma unroll
    for (int v = 0; v < 5; ++v) p.V[v] = __builtin_amdgcn_raw_buffer_load_b128(rs, (int)(slot + v * 1024 + lane * 16), 0, AUX_SYS);
    p.q = __builtin_amdgcn_raw_buffer_load_b64(rs, (int)(slot + PAY_Q2 + lane * 8), 0, AUX_SYS);
    const u32 hoff = 2u * (u32)lane < ((meta >> 8) & 0xFFu) ? slot + PAY_X + (u32)lane * 4u : 0x80000000u;
    p.h = __builtin_amdgcn_raw_buffer_load_b32(rs, (int)hoff, 0, AUX_SYS);
}
__device__ __forceinline__ void kp_clear(LAS unsigned char* kp, int lane) {
#pragma unroll
    for (int i = 0; i < 5; ++i) *(LAS u32x4*)(kp + lane * KP_LANE + 16 * i) = (u32x4){0u, 0u, 0u, 0u};
}
__device__ __forceinline__ void unpack(u32x4 (&d)[8], const Pull& p, rsrc_t rs, u32 slot, u32 meta, int lane, LAS unsigned char* kp) {
    if (meta & META_RAW) {
#pragma unroll
        for (int v = 0; v < 5; ++v) d[v] = p.V[v];
#pragma unroll
        for (int v = 5; v < 8; ++v) d[v] = __builtin_amdgcn_raw_buffer_load_b128(rs, (int)(slot + v * 1024 + lane * 16), 0, AUX_SYS);
        wait_vm0_known();
        return;
    }
    const u32 cnt = (meta >> 8) & 0xFFu;
    const u32 L = (meta & 0xFFu) << 7, LL = L | (L << 16), kz = kz_of(L);
    u32 at[2];
#pragma unroll
    for (int i = 0; i < 2; ++i) {
        const u32 e = i ? p.h >> 16 : p.h & 0xFFFFu, pos = e >> 4, k4 = e & 15u;
        at[i] = 2u * (u32)lane + (u32)i < cnt ? pos + (pos >> 6) * 16u : (u32)KP_BYTES;
        kp[at[i]] = (unsigned char)((k4 == 8u ? kz : k4 | ((k4 & 8u) << 1)) << 2);
    }
    u32x4 K[4];
#pragma unroll
    for (int i = 0; i < 4; ++i) K[i] = *(LAS const u32x4*)(kp + lane * KP_LANE + 16 * i);
    kp[at[0]] = 0; kp[at[1]] = 0;
    const u32 Q[6] = {p.V[4][0], p.V[4][1], p.V[4][2], p.V[4][3], p.q[0], p.q[1]};
#pragma unroll
    for (int g = 0; g < 2; ++g) {
        const u32 q0 = Q[3 * g], q1 = Q[3 * g + 1], q2 = Q[3 * g + 2];
        const u32x4 k0 = K[2 * g], k1 = K[2 * g + 1];
        u32 h[8];
        h[0] = (q0 & 0x83838383u) | k0[0]; h[1] = (rotr(q0, 3) & 0x83838383u) | k0[1];
        h[2] = (q1 & 0x83838383u) | k0[2]; h[3] = (rotr(q1, 3) & 0x83838383u) | k0[3];
        h[4] = (q2 & 0x83838383u) | k1[0]; h[5] = (rotr(q2, 3) & 0x83838383u) | k1[1];
        h[6] = (rotr(q0, 6) & 0x81818181u) | ((rotr(q1, 4) & 0x02020202u) | k1[2]);
        h[7] = (rotr(q2, 6) & 0x81818181u) | ((rotr(q1, 5) & 0x02020202u) | k1[3]);
#pragma unroll
        for (int j = 0; j < 8; ++j) {
            const u32 B = p.V[2 * g + (j >> 2)][j & 3];
            d[4 * g + (j >> 1)][(j & 1) * 2] = pk_add_s(__builtin_amdgcn_perm(h[j], B, 0x05010400u), LL);
            d[4 * g + (j >> 1)][(j & 1) * 2 + 1] = pk_add_s(__builtin_amdgcn_perm(h[j], B, 0x07030602u), LL);
        }
    }
}
__device__ __forceinline__ void fill_nan(u32x4 (&d)[8]) {
#pragma unroll
    for (int v = 0; v < 8; ++v) d[v] = (u32x4){0x7FC07FC0u, 0x7FC07FC0u, 0x7FC07FC0u, 0x7FC07FC0u};
}
__device__ __forceinline__ void copy_store(const Ctx& c, const Pull& p, rsrc_t rs, u32 slot, u32 dst, u32 meta) {
    const int lane = c.lane;
    if (meta & META_RAW) {
        u32x4 a[3];
#pragma unroll
        for (int v = 0; v < 3; ++v) a[v] = __builtin_amdgcn_raw_buffer_load_b128(rs, (int)(slot + (5 + v) * 1024 + lane * 16), 0, AUX_SYS);
        wait_vm0_known();
#pragma unroll
        for (int v = 0; v < 3; ++v) __builtin_amdgcn_raw_buffer_store_b128(a[v], c.own, (int)(dst + (5 + v) * 1024 + lane * 16), 0, AUX_SYS);
    } else {
        __builtin_amdgcn_raw_buffer_store_b64(p.q, c.own, (int)(dst + PAY_Q2 + lane * 8), 0, AUX_SYS);
        if (2u * (u32)lane < ((meta >> 8) & 0xFFu)) __builtin_amdgcn_raw_buffer_store_b32(p.h, c.own, (int)(dst + PAY_X + lane * 4), 0, AUX_SYS);
    }
#pragma unroll
    for (int v = 0; v < 5; ++v) __builtin_amdgcn_raw_buffer_store_b128(p.V[v], c.own, (int)(dst + v * 1024 + lane * 16), 0, AUX_SYS);
}

__device__ __forceinline__ u32 ticket_issue(const Ctx& c, u32 off) {
    u32 r; u64 sv; const u32 one = 1u;
    asm volatile("s_mov_b64 %1, exec\n\ts_mov_b64 exec, 1\n\tglobal_atomic_add %0, %2, %3, %4 sc0\n\ts_mov_b64 exec, %1"
                 : "=&v"(r), "=&s"(sv) : "v"(off), "v"(one), "s"(c.ws) : "memory");
    return r;
}
__device__ __forceinline__ u32 ticket_get(u32 r) { asm volatile("" : "+v"(r) :: "memory"); return rfl(r); }

struct Step { u32 foff, slot, meta; int sel, p0, p1, p2, p3; };
template <int SPT, class Gen, class Act, class Pub>
__device__ __forceinline__ void pull_list(Ctx& c, int list, u32 first, u32 nwaves, u32 njobs, rsrc_t rs0, rsrc_t rs1, Gen gen, Act act, Pub pub) {
    DT0();
    const u32 tkoff = OFF_TKT + (u32)list * 128u, tkbase = nwaves - (c.call - 1u) * njobs;
    u32 jc = first, jn = 0, tk = 0;
    if (SPT == 1) tk = ticket_issue(c, tkoff);
    Step cur = gen(jc, 0), prev = cur;
    bool ok = wait_flag(c, cur.foff, cur.meta), have_prev = false, more = true;
    if (SPT == 1) { wait_vm0_known(); jn = ticket_get(tk) + tkbase; }
    DT(1);
    Pull pa, pb;
    JLOG(list, jc, 0, 0);
    pull_issue(pa, cur.sel ? rs1 : rs0, cur.slot, cur.meta, c.lane);
    auto half = [&](const Pull& p, Pull& pn, int sub) {
        const bool same = SPT == 2 && sub == 0;
        more = same || jn < njobs;
        Step nxt = gen(same || !more ? jc : jn, same ? 1 : 0);
        const bool draw = SPT == 1 ? more : same;
        if (draw) tk = ticket_issue(c, tkoff);
        const u64 fv = flag_load(c, nxt.foff);
        wait_vm0_known();
        DT(0);
        JLOG(list, jc, sub, 1);
        const u32 jd = draw ? ticket_get(tk) + tkbase : njobs;
        if (have_prev) pub(prev);
        const bool nready = more && flag_test(c, fv, nxt.meta);
        if (nready) { JLOG(list, same ? jc : jn, same ? 1 : 0, 0); pull_issue(pn, nxt.sel ? rs1 : rs0, nxt.slot, nxt.meta, c.lane); }
        DT(3);
        act(cur, p, cur.sel ? rs1 : rs0, ok);
        prev = cur; have_prev = true; ok = true;
        DT(2);
        if (more && !nready) {
            wait_vm0_known(); pub(prev); have_prev = false;
            ok = wait_flag(c, nxt.foff, nxt.meta);
            JLOG(list, same ? jc : jn, same ? 1 : 0, 0);
            pull_issue(pn, nxt.sel ? rs1 : rs0, nxt.slot, nxt.meta, c.lane);
            DT(1);
        }
        cur = nxt;
        if (SPT == 1) { jc = jn; jn = jd; }
        else if (same) jn = jd; else jc = jn;
    };
    for (;;) {
        half(pa, pb, 0); if (!more) break;
        half(pb, pa, 1); if (!more) break;
    }
    wait_vm0_known();
    if (have_prev) pub(prev);
}

__device__ __forceinline__ int cq_of(int q, int r) {
    return ((r >> 1) ^ ((q & 1) ? (r >> 2) : 0) ^ ((q & 2) ? r : 0)) & 1;
}
__device__ __forceinline__ int link_d(int link) { return (0x765431 >> (4 * link)) & 7; }
__device__ __forceinline__ int link_q(int link, int i) { return ((i ? 0x233231 : 0x110020) >> (4 * link)) & 3; }
__device__ __forceinline__ int link_of_d(int d) { return d == 1 ? 0 : d - 2; }
__device__ __forceinline__ int t_of(int d, int q, int kq) {
    if (d == 0) return kq * 4 + q;
    const int link = link_of_d(d);
    return 16 + (2 * kq + (link_q(link, 0) == q ? 0 : 1)) * 6 + link;
}
__device__ __forceinline__ int d3_pair(int d) { return (d & 1) * 2 + ((d >> 1) & 1); }
__device__ __forceinline__ int d3_of_pair(int p) { return 4 + (p >> 1) + (p & 1) * 2; }
__device__ __forceinline__ int rowtile_of(int rc, int o, int a) {
    const int d = rc ^ o;
    return d == 0 ? 12 + a : d == 2 ? a : 4 + 2 * d3_pair(d) + (a & 1);
}
__device__ __forceinline__ void block_of_rowtile(int rc, int i, int& o, int& a) {
    if (i >= 12) { o = rc; a = i - 12; }
    else if (i < 4) { o = rc ^ 2; a = i; }
    else { const int p = (i - 4) >> 1; o = rc ^ d3_of_pair(p); a = 2 * (p >> 1) + ((i - 4) & 1); }
}
__device__ __forceinline__ u32 raw_frag_off(int type, int sub, int f) {
    return type == 0 ? (u32)(sub * 8192 + f * 1024) : (u32)((f >> 2) * 16384 + sub * 4096 + (f & 3) * 1024);
}
__device__ __forceinline__ void store_operand(const Ctx& c, const u32x4 (&d)[8], int type, int tile, int t, int sub) {
    const u32 base = (type == 0 ? OFF_A : OFF_B) + (u32)(tile * 64 + t) * 32768u + (u32)c.lane * 16u;
#pragma unroll
    for (int f = 0; f < 8; ++f) __builtin_amdgcn_raw_buffer_store_b128(d[f], c.own, (int)(base + raw_frag_off(type, sub, f)), 0, AUX_SYS);
}
__device__ __forceinline__ void count_operand(const Ctx& c, int type, int tile, int t) {
    GAS u32* p = (GAS u32*)(c.ws + (type == 0 ? OFF_RDYA : OFF_RDYB)) + tile * 64 + t;
    if (c.lane == 0) __hip_atomic_fetch_add(p, 1u, __ATOMIC_RELAXED, __HIP_MEMORY_SCOPE_AGENT);
}

__device__ __forceinline__ void pack_job(const Params& P, const Ctx& c, LAS unsigned char* tile, int pj) {
    const int kq = pj >> 10, q = pj & 3, type = (pj >> 2) & 1, g = (pj & 1023) >> 3, kt = 4 * q + kq, lane = c.lane, s = c.rank;
    u32x4 d[8];
    if (type == 0) {
        GAS const char* src = (GAS const char*)P.x + (size_t)(64 * g + (lane & 15)) * 2048 + (64 * kt + 8 * (lane >> 4)) * 2;
#pragma unroll
        for (int f = 0; f < 8; ++f) d[f] = *(GAS const u32x4*)(src + (f >> 1) * 16 * 2048 + (f & 1) * 64);
    } else {
        GAS const char* src = (GAS const char*)P.w + (size_t)(64 * kt + (lane >> 3)) * 16384 + (64 * g) * 2 + (lane & 7) * 16;
#pragma unroll
        for (int v = 0; v < 8; ++v)
            __builtin_amdgcn_global_load_lds((GAS const u32*)(src + (size_t)v * 8 * 16384), (LAS u32*)(tile + v * 1024), 16, 0, 0);
        wait_vm0();
        const int lg = lane >> 4, lq = (lane >> 2) & 3, lp = lane & 3;
        LAS unsigned char* b = tile + (8 * lg + lq) * 128 + lp * 16;
#pragma unroll
        for (int f = 0; f < 8; ++f) {
            const int o = (f & 1) * 32 * 128 + (f >> 2) * 64 + ((f >> 1) & 1) * 8;
            const s16x4 lo4 = __builtin_amdgcn_ds_read_tr16_b64_v4i16((LAS s16x4*)(b + o));
            const s16x4 hi4 = __builtin_amdgcn_ds_read_tr16_b64_v4i16((LAS s16x4*)(b + o + 4 * 128));
            const u32x2 l2 = __builtin_bit_cast(u32x2, lo4), h2 = __builtin_bit_cast(u32x2, hi4);
            d[f] = (u32x4){l2[0], l2[1], h2[0], h2[1]};
        }
        asm volatile("s_waitcnt lgkmcnt(0)" ::: "memory");
    }
    int rc[2], tl[2], tt[2];
    const int cs = cq_of(q, s);
#pragma unroll
    for (int i = 0; i < 2; ++i) {
        int e1, gam;
        if (type == 0) { const int o = g >> 4, h = (g >> 3) & 1; gam = i; e1 = h == 0 ? (o & 1) : ((o ^ (o >> 2) ^ gam) & 1); }
        else { gam = g >> 6; e1 = i; }
        const int e2 = (cs ^ ((q & 2) ? e1 : 0) ^ ((q & 1) ? gam : 0)) & 1;
        rc[i] = e1 | (e2 << 1) | (gam << 2);
        tl[i] = type == 0 ? rowtile_of(rc[i], g >> 4, (g >> 2) & 3) : ((g >> 2) & 15);
        tt[i] = t_of(rc[i] ^ s, q, kq);
    }
    const int sub = g & 3;
    const u32 meta = pack_store(d, c.own, OFF_IN + c.par * IN_BYTES + (u32)(type * 2048 + kt * 128 + g) * SLOT, lane, tile);
    if (rc[0] == s) store_operand(c, d, type, tl[0], tt[0], sub);
    if (rc[1] == s) store_operand(c, d, type, tl[1], tt[1], sub);
    wait_vm0();
#pragma unroll
    for (int i = 0; i < 2; ++i) {
        if (rc[i] == s) count_operand(c, type, tl[i], tt[i]);
        else post_flag(P, c, rc[i], F_IN + (u32)(((type * 16 + tl[i]) * 64 + tt[i]) * 4 + sub) * 8u, meta);
    }
}

__device__ __forceinline__ void in_list(const Params& P, Ctx& c, int link, int widx, int wpl) {
    const int s = c.rank ^ link_d(link);
    auto gen = [&](u32 job, int) {
        const int j = (int)job;
        int kc, type, tile, sub;
        if (j < 896) { kc = j / 112; const int u = j % 112; if (u < 48) { type = 0; tile = u >> 2; } else { type = 1; tile = (u - 48) >> 2; } sub = u & 3; }
        else { const int m = j - 896; kc = m >> 4; type = 0; tile = 12 + ((m >> 2) & 3); sub = m & 3; }
        const int t = 16 + kc * 6 + link, kt = 4 * link_q(link, kc & 1) + (kc >> 1);
        int g;
        if (type == 0) { int o, a; block_of_rowtile(c.rank, tile, o, a); g = o * 16 + a * 4 + sub; } else g = (c.rank >> 2) * 64 + tile * 4 + sub;
        Step st; st.foff = F_IN + (u32)(((type * 16 + tile) * 64 + t) * 4 + sub) * 8u;
        st.slot = OFF_IN + c.par * IN_BYTES + (u32)(type * 2048 + kt * 128 + g) * SLOT;
        st.sel = 0; st.meta = 0; st.p0 = type; st.p1 = tile; st.p2 = t; st.p3 = sub;
        return st;
    };
    auto act = [&](Step& st, const Pull& p, rsrc_t rs, bool ok) {
        u32x4 d[8];
        if (ok) unpack(d, p, rs, st.slot, st.meta, c.lane, c.kp); else fill_nan(d);
        store_operand(c, d, st.p0, st.p1, st.p2, st.p3);
    };
    auto pub = [&](const Step& st) { count_operand(c, st.p0, st.p1, st.p2); };
    const rsrc_t rs = mk_rsrc(P.ws[s & 7]);
    pull_list<1>(c, link, (u32)widx, (u32)wpl, 1024u, rs, rs, gen, act, pub);
}

__device__ __forceinline__ void acc_add(float (&acc)[64], const u32x4 (&d)[8]) {
#pragma unroll
    for (int q = 0; q < 32; ++q) { const u32 v = d[q >> 2][q & 3]; acc[2 * q] += __builtin_bit_cast(float, v << 16); acc[2 * q + 1] += __builtin_bit_cast(float, v & 0xFFFF0000u); }
}
__device__ __forceinline__ void write_out(const Params& P, const Ctx& c, const float (&acc)[64], int a, int cj, int jb) {
    const int w = jb >> 1, ai = jb & 1, lane = c.lane;
    GAS char* o = (GAS char*)P.out + (size_t)(a * 256 + ai * 128 + (w >> 2) * 64 + (lane & 15)) * 16384 + (cj * 256 + (w & 3) * 64 + 8 * (lane >> 4)) * 2;
#pragma unroll
    for (int f = 0; f < 8; ++f) {
        u32x4 v;
#pragma unroll
        for (int e = 0; e < 4; ++e) v[e] = cvt_pk_bf16(acc[f * 8 + 2 * e], acc[f * 8 + 2 * e + 1]);
        *(GAS u32x4*)(o + (size_t)(f >> 1) * 16 * 16384 + (f & 1) * 64) = v;
    }
}
__device__ __forceinline__ void copy_list(const Params& P, Ctx& c, int grp, int widx, int wpl) {
    const int r = c.rank;
    auto gen = [&](u32 job, int) {
        const int j = (int)job, jb = j & 15;
        Step st; st.meta = 0;
        if (grp == 0) {
            const int u = j >> 4, pt = (u & 3) * 16 + (u >> 2) * 2;
            st.foff = F_P + (u32)(pt * 16 + jb) * 8u; st.slot = OFF_OUT + c.par * OUT_BYTES + (u32)(pt * 16 + jb) * SLOT;
            st.p0 = (int)(OFF_OWN + (u32)((pt * 4 + 3) * 16 + jb) * SLOT); st.p1 = r; st.p2 = (int)(F_OWN + (u32)((pt * 4 + 3) * 16 + jb) * 8u);
        } else {
            const int second = j >= 256, v = (j >> 4) & 15, u = (v & 3) * 4 + (v >> 2), path = (grp - 1) ^ second, pt = 4 * u + 1 + 2 * path, rs = path * 16 + u;
            if (!second) {
                st.foff = F_R + (u32)(rs * 16 + jb) * 8u; st.slot = OFF_OUT + c.par * OUT_BYTES + (u32)(pt * 16 + jb) * SLOT;
                st.p0 = (int)(OFF_REL + c.par * REL_BYTES + (u32)(rs * 16 + jb) * SLOT); st.p1 = r ^ (path ? 1 : 3); st.p2 = (int)(F_P + (u32)(pt * 16 + jb) * 8u);
            } else {
                st.foff = F_P + (u32)(pt * 16 + jb) * 8u; st.slot = OFF_REL + c.par * REL_BYTES + (u32)(rs * 16 + jb) * SLOT;
                st.p0 = (int)(OFF_OWN + (u32)((pt * 4 + 3) * 16 + jb) * SLOT); st.p1 = r; st.p2 = (int)(F_OWN + (u32)((pt * 4 + 3) * 16 + jb) * 8u);
            }
        }
        st.p3 = 0; st.sel = 0;
        return st;
    };
    auto act = [&](Step& st, const Pull& p, rsrc_t rs, bool ok) {
        if (ok) copy_store(c, p, rs, st.slot, (u32)st.p0, st.meta);
        else { u32x4 d[8]; fill_nan(d); store_raw(d, c.own, (u32)st.p0, c.lane); st.meta = META_RAW; }
    };
    auto pub = [&](const Step& st) { post_flag(P, c, st.p1, (u32)st.p2, st.meta); };
    const rsrc_t rs = mk_rsrc(P.ws[(r ^ (grp == 0 ? 2 : grp == 1 ? 1 : 3)) & 7]);
    pull_list<1>(c, 6 + grp, (u32)widx, (u32)wpl, 512u, rs, rs, gen, act, pub);
}
__device__ __forceinline__ void reduce_list(const Params& P, Ctx& c, int h, int widx, int wpl) {
    const int r = c.rank;
    float acc[64];
    auto gen = [&](u32 ujob, int sub) {
        const int job = (int)ujob, sl = (sub ^ job) & 1, jb = job & 15, ot2 = job >> 4;
        const int a = 2 * h + (ot2 & 1), j = ot2 >> 1, ot = a * 16 + j, p = 2 * h + sl, ti = 4 + 2 * p + (a & 1);
        Step st; st.meta = 0;
        st.foff = F_D3 + (u32)((ot * 2 + sl) * 16 + jb) * 8u; st.sel = sl;
        st.slot = OFF_OUT + c.par * OUT_BYTES + (u32)((ti * 16 + j) * 16 + jb) * SLOT;
        st.p0 = sub; st.p1 = a; st.p2 = (1 - (r >> 2)) * 16 + j; st.p3 = jb;
        return st;
    };
    auto act = [&](Step& st, const Pull& p, rsrc_t rs, bool ok) {
        u32x4 d[8];
        if (ok) unpack(d, p, rs, st.slot, st.meta, c.lane, c.kp); else fill_nan(d);
        if (st.p0 == 0) {
#pragma unroll
            for (int q = 0; q < 64; ++q) acc[q] = 0.f;
        }
        acc_add(acc, d);
        if (st.p0 == 1) write_out(P, c, acc, st.p1, st.p2, st.p3);
    };
    auto pub = [&](const Step&) {};
    pull_list<2>(c, 9 + h, (u32)widx, (u32)wpl, 512u, mk_rsrc(P.ws[(r ^ d3_of_pair(2 * h)) & 7]), mk_rsrc(P.ws[(r ^ d3_of_pair(2 * h + 1)) & 7]), gen, act, pub);
}
__device__ __forceinline__ void own_job(const Params& P, const Ctx& c, int n) {
    const int pt = n >> 4, jb = n & 15;
    kp_clear(c.kp, c.lane);
    float acc[64];
#pragma unroll
    for (int i = 0; i < 64; ++i) acc[i] = 0.f;
    for (int part = 0; part < 4; ++part) {
        u32 meta; u32x4 d[8];
        if (wait_flag(c, F_OWN + (u32)((pt * 4 + part) * 16 + jb) * 8u, meta)) {
            Pull p; const u32 slot = OFF_OWN + (u32)((pt * 4 + part) * 16 + jb) * SLOT;
            pull_issue(p, c.own, slot, meta, c.lane); unpack(d, p, c.own, slot, meta, c.lane, c.kp);
        } else fill_nan(d);
        acc_add(acc, d);
    }
    write_out(P, c, acc, pt >> 4, (c.rank >> 2) * 16 + (pt & 15), jb);
}

struct GJob { int ti, tj, t0, nt; };

#define STAMP(i) do {} while (0)

__device__ __forceinline__ void gemm_role(const Params& P, const Ctx& c, LAS unsigned char* lds, int xcd, int idx) {
    const int tid = threadIdx.x, wid = (int)rfl(tid >> 6), lane = c.lane, wr = wid >> 2, wc = wid & 3;
    constexpr int HTB = 16384;
    const u32 voff = (u32)tid * 16u;
    const unsigned ldsw = (unsigned)wid * 1024u;
    const int aoff = wr * 8192 + lane * 16, boff = wc * 4096 + lane * 16;
    LAS volatile int* ctl = (LAS volatile int*)(lds + LDS_CTL);
    int avail = 0; bool bad = false;
#define G_SA(b, h) (((b) * 2 + (h)) * HTB)
#define G_SB(b, h) ((4 + (b) * 2 + (h)) * HTB)
#define G_STAGE(bufoff, gbase) do { _Pragma("unroll") for (int _i = 0; _i < 2; ++_i) \
        __builtin_amdgcn_global_load_lds((GAS const u32*)((GAS const char*)(gbase) + voff + _i * 8192), (LAS u32*)(lds + (bufoff) + ldsw + _i * 8192), 16, 0, AUX_DEV); } while (0)
#define G_LDA(dst, b, h) do { _Pragma("unroll") for (int m = 0; m < 4; ++m) _Pragma("unroll") for (int k = 0; k < 2; ++k) dst[m][k] = *(const LAS bf16x8*)(lds + G_SA(b, h) + aoff + m * 2048 + k * 1024); } while (0)
#define G_LDB(dst, b, h) do { _Pragma("unroll") for (int n = 0; n < 2; ++n) _Pragma("unroll") for (int k = 0; k < 2; ++k) dst[n][k] = *(const LAS bf16x8*)(lds + G_SB(b, h) + boff + n * 2048 + k * 1024); } while (0)
#define G_MMA(ai, bj, At, Bt) do { __builtin_amdgcn_s_setprio(1); _Pragma("unroll") for (int m = 0; m < 4; ++m) _Pragma("unroll") for (int n = 0; n < 2; ++n) _Pragma("unroll") for (int k = 0; k < 2; ++k) \
        acc[ai][bj][m][n] = __builtin_amdgcn_mfma_f32_16x16x32_bf16(Bt[n][k], At[m][k], acc[ai][bj][m][n], 0, 0, 0); __builtin_amdgcn_s_setprio(0); } while (0)
#define G_WAIT_V(n) asm volatile("s_waitcnt vmcnt(" #n ")" ::: "memory")
#define G_WAIT_L(n) asm volatile("s_waitcnt lgkmcnt(" #n ")" ::: "memory")
#define G_BAR __builtin_amdgcn_s_barrier()
#define G_SCHED __builtin_amdgcn_sched_barrier(0)
    auto ensure = [&](const GJob& j, int tneed) {
        if (avail >= tneed) return;
        if (wid == 0) {
            GAS const u32* ra = (GAS const u32*)(c.ws + OFF_RDYA) + j.ti * 64 + lane;
            GAS const u32* rb = (GAS const u32*)(c.ws + OFF_RDYB) + j.tj * 64 + lane;
            GAS u32* tmo = (GAS u32*)(c.ws + OFF_TMO);
            const u32 target = 4u * c.call;
            const u64 below = j.t0 ? ((1ull << j.t0) - 1ull) : 0ull;
            const u64 t0 = __builtin_amdgcn_s_memrealtime();
            int av, bd = 0;
            for (;;) {
                const u32 a = __hip_atomic_load(ra, __ATOMIC_RELAXED, __HIP_MEMORY_SCOPE_AGENT), b = __hip_atomic_load(rb, __ATOMIC_RELAXED, __HIP_MEMORY_SCOPE_AGENT);
                const u64 ok = __builtin_amdgcn_ballot_w64((int)(a - target) >= 0 && (int)(b - target) >= 0) | below;
                av = ok == ~0ull ? 64 : __builtin_ctzll(~ok);
                if (av >= tneed) break;
                if (rfl(__hip_atomic_load(tmo, __ATOMIC_RELAXED, __HIP_MEMORY_SCOPE_AGENT)) == c.call || __builtin_amdgcn_s_memrealtime() - t0 > DEADLINE) {
                    __hip_atomic_store(tmo, c.call, __ATOMIC_RELAXED, __HIP_MEMORY_SCOPE_AGENT); av = 64; bd = 1; break;
                }
                __builtin_amdgcn_s_sleep(4);
            }
            if (lane == 0) { ctl[0] = av; if (bd) ctl[1] = 1; }
            G_WAIT_L(0);
        }
        asm volatile("" ::: "memory"); G_BAR; asm volatile("" ::: "memory");
        avail = (int)rfl((u32)ctl[0]); bad = rfl((u32)ctl[1]) != 0;
    };
    GJob cur, nxt;
    cur.ti = 6 * (xcd & 1) + idx % 6; cur.tj = 4 * (xcd >> 1) + idx / 6; cur.t0 = 0; cur.nt = 64;
    { const int col = xcd * 6 + (idx >> 2), third = col >> 4; nxt.ti = 12 + (idx & 3); nxt.tj = col & 15; nxt.t0 = 22 * third; nxt.nt = third == 2 ? 20 : 22; }
    f32x4 acc[2][2][4][2];
    bf16x8 At[4][2], B0[2][2], B1[2][2];
    constexpr size_t kstep = 32768, hstep = 16384;
    for (int ui = 0; ui < 2; ++ui) {
        const bool has_next = ui == 0;
#pragma unroll
        for (int a = 0; a < 2; ++a)
#pragma unroll
            for (int b = 0; b < 2; ++b)
#pragma unroll
                for (int m = 0; m < 4; ++m)
#pragma unroll
                    for (int n = 0; n < 2; ++n) acc[a][b][m][n] = (f32x4){0.f, 0.f, 0.f, 0.f};
        const char* cA = c.ws + OFF_A + (size_t)(cur.ti * 64 + cur.t0) * kstep;
        const char* cB = c.ws + OFF_B + (size_t)(cur.tj * 64 + cur.t0) * kstep;
        avail = 0;
        ensure(cur, cur.t0 + 2);
        G_STAGE(G_SB(0, 0), cB); G_STAGE(G_SA(0, 0), cA); G_STAGE(G_SB(0, 1), cB + hstep); G_STAGE(G_SA(0, 1), cA + hstep);
        if (wr == 1) G_BAR;
        G_WAIT_V(4); G_BAR;
        G_STAGE(G_SB(1, 0), cB + kstep); G_STAGE(G_SA(1, 0), cA + kstep); G_STAGE(G_SB(1, 1), cB + hstep + kstep);
        G_WAIT_V(6); G_BAR;
        if (ui == 0) STAMP(2);
        const int nt = cur.nt;
        for (int t = 0; t < nt; t += 2) {
            const bool last = (t == nt - 2);
            const char* a1 = cA + (size_t)(t + 1) * kstep;
            const char* a2 = cA + (size_t)(last ? t : t + 2) * kstep; const char* b2 = cB + (size_t)(last ? t : t + 2) * kstep;
            const char* a3 = a2 + kstep; const char* b3 = b2 + kstep;
            if (!last) ensure(cur, cur.t0 + t + 4);
            G_LDB(B0, 0, 0); G_SCHED; G_LDA(At, 0, 0); G_STAGE(G_SA(1, 1), a1 + hstep);
            G_WAIT_L(8); G_BAR; G_WAIT_L(0); G_MMA(0, 0, At, B0); G_BAR; G_SCHED;
            G_LDB(B1, 0, 1); G_STAGE(G_SB(0, 0), b2);
            G_BAR; G_WAIT_L(0); G_MMA(0, 1, At, B1); G_BAR;
            G_LDA(At, 0, 1); G_STAGE(G_SA(0, 0), a2);
            G_BAR; G_WAIT_L(0); G_MMA(1, 0, At, B0); G_BAR; G_SCHED;
            G_STAGE(G_SB(0, 1), b2 + hstep);
            G_WAIT_V(6); G_BAR; G_MMA(1, 1, At, B1); G_BAR;
            G_LDB(B0, 1, 0); G_SCHED; G_LDA(At, 1, 0); G_STAGE(G_SA(0, 1), a2 + hstep);
            G_WAIT_L(8); G_BAR; G_WAIT_L(0); G_MMA(0, 0, At, B0); G_BAR; G_SCHED;
            G_LDB(B1, 1, 1); G_STAGE(G_SB(1, 0), b3);
            G_BAR; G_WAIT_L(0); G_MMA(0, 1, At, B1); G_BAR;
            G_LDA(At, 1, 1); G_STAGE(G_SA(1, 0), a3);
            G_BAR; G_WAIT_L(0); G_MMA(1, 0, At, B0); G_BAR; G_SCHED;
            G_STAGE(G_SB(1, 1), b3 + hstep);
            G_WAIT_V(6); G_BAR; G_MMA(1, 1, At, B1); G_BAR;
        }
        if (wr == 0) G_BAR;
        G_WAIT_V(0); G_BAR;
        STAMP(3 + 2 * ui);
        {
            u32 meta[2], slot[2];
#pragma unroll
            for (int ai = 0; ai < 2; ++ai) {
                u32x4 d[8];
#pragma unroll
                for (int m = 0; m < 4; ++m)
#pragma unroll
                    for (int bj = 0; bj < 2; ++bj) {
                        const f32x4 v0 = acc[ai][bj][m][0], v1 = acc[ai][bj][m][1];
                        d[m * 2 + bj] = (u32x4){cvt_pk_bf16(v0[0], v0[1]), cvt_pk_bf16(v0[2], v0[3]), cvt_pk_bf16(v1[0], v1[1]), cvt_pk_bf16(v1[2], v1[3])};
                    }
                if (bad) fill_nan(d);
                const int jb = wid * 2 + ai;
                if (has_next) { slot[ai] = OFF_OUT + c.par * OUT_BYTES + (u32)((cur.ti * 16 + cur.tj) * 16 + jb) * SLOT; meta[ai] = pack_store(d, c.own, slot[ai], lane, c.kp); }
                else { slot[ai] = OFF_OWN + (u32)((((cur.ti - 12) * 16 + cur.tj) * 4 + (cur.t0 != 0) + (cur.t0 == 44)) * 16 + jb) * SLOT; store_raw(d, c.own, slot[ai], lane); meta[ai] = META_RAW; }
            }
            wait_vm0();
#pragma unroll
            for (int ai = 0; ai < 2; ++ai) {
                const int jb = wid * 2 + ai, r = c.rank;
                int dst; u32 foff;
                if (!has_next) { dst = r; foff = F_OWN + (u32)((((cur.ti - 12) * 16 + cur.tj) * 4 + (cur.t0 != 0) + (cur.t0 == 44)) * 16 + jb) * 8u; }
                else if (cur.ti < 4) {
                    const int pt = cur.ti * 16 + cur.tj;
                    if ((pt & 1) == 0) { dst = r ^ 2; foff = F_P + (u32)(pt * 16 + jb) * 8u; }
                    else { const int path = (pt >> 1) & 1; dst = r ^ (path ? 3 : 1); foff = F_R + (u32)((path * 16 + (pt >> 2)) * 16 + jb) * 8u; }
                } else {
                    const int p = (cur.ti - 4) >> 1, a = 2 * (p >> 1) + ((cur.ti - 4) & 1);
                    dst = r ^ d3_of_pair(p); foff = F_D3 + (u32)(((a * 16 + cur.tj) * 2 + (p & 1)) * 16 + jb) * 8u;
                }
                post_flag(P, c, dst, foff, meta[ai]);
            }
        }
        STAMP(4 + 2 * ui);
        cur = nxt;
    }
#undef G_SA
#undef G_SB
#undef G_STAGE
#undef G_LDA
#undef G_LDB
#undef G_MMA
#undef G_WAIT_V
#undef G_WAIT_L
#undef G_BAR
#undef G_SCHED
}

__global__ __launch_bounds__(NTH, 2) void gemm_rs_kernel(const Params P) {
    extern __shared__ __attribute__((aligned(16))) unsigned char shm[];
    LAS unsigned char* lds = (LAS unsigned char*)shm;
    const int wg = blockIdx.x, xcd = wg & 7, idx = wg >> 3, tid = threadIdx.x, wid = (int)rfl(tid >> 6);
    Ctx c;
    c.rank = P.rank & 7; c.lane = tid & 63;
    c.ws = (char*)P.ws[c.rank]; c.own = mk_rsrc(c.ws);
    LAS volatile u32* ctl = (LAS volatile u32*)(lds + LDS_CTL);
    if (tid == 0) {
        ctl[2] = __hip_atomic_fetch_add((GAS u32*)(c.ws + OFF_CALL + wg * 64), 1u, __ATOMIC_RELAXED, __HIP_MEMORY_SCOPE_AGENT) + 1u;
        ctl[0] = 0; ctl[1] = 0;
    }
    __syncthreads();
    c.call = rfl(ctl[2]); c.par = c.call & 1u;
    c.kp = lds + wid * 8192;
    STAMP(0);
    if (idx < NGEMM / 8) {
        const int gw = (xcd * (NGEMM / 8) + idx) * 8 + wid;
        for (int pj = gw; pj < 4096; pj += GEMM_WAVES) pack_job(P, c, c.kp, pj);
        STAMP(1);
        __syncthreads();
        gemm_role(P, c, lds, xcd, idx);
        if (gw < 1024) own_job(P, c, gw);
        STAMP(7);
    } else {
        const int cw = wid * NCOMM + (idx - NGEMM / 8) * 8 + xcd;
        if (cw >= 6 * IN_WPL && cw >= O_WAVES) return;
        kp_clear(c.kp, c.lane);
#define GR_INMASK 63
        if (cw < 6 * IN_WPL && ((GR_INMASK >> (cw % 6)) & 1)) in_list(P, c, cw % 6, cw / 6, IN_WPL);
        STAMP(3);
        if (cw < 3 * C_WPL) copy_list(P, c, cw % 3, cw / 3, C_WPL);
        else if (cw < O_WAVES) reduce_list(P, c, (cw - 3 * C_WPL) & 1, (cw - 3 * C_WPL) >> 1, 2 * X_WPL);
        STAMP(7);
    }
}
}

extern "C" void kernel_launch(int rank, int nranks, ncclComm_t comm, hipStream_t stream,
                              void* const* d_in, const int* in_sizes, int n_in,
                              void* d_out, int out_size,
                              const int* in_dtypes, int out_dtype,
                              void* const* d_out_all,
                              void* const* d_ws_all, size_t ws_bytes) {
    (void)comm; (void)in_sizes; (void)n_in; (void)out_size; (void)in_dtypes; (void)out_dtype; (void)d_out_all;
    if (nranks != gr::NR || rank < 0 || rank >= gr::NR || ws_bytes < gr::WS_NEED) { fprintf(stderr, "kernel_launch: unexpected configuration\n"); return; }
    static bool init[gr::NR] = {};
    if (!init[rank]) {
        if (hipFuncSetAttribute((const void*)gr::gemm_rs_kernel, hipFuncAttributeMaxDynamicSharedMemorySize, gr::LDS_BYTES) != hipSuccess) { fprintf(stderr, "hipFuncSetAttribute failed\n"); return; }
        init[rank] = true;
    }
    gr::Params P;
    P.x = d_in[0]; P.w = d_in[1]; P.out = d_out; P.rank = rank; P.pad = 0;
    for (int i = 0; i < gr::NR; ++i) P.ws[i] = d_ws_all[i];
    hipLaunchKernelGGL(gr::gemm_rs_kernel, dim3(gr::NWG), dim3(gr::NTH), gr::LDS_BYTES, stream, P);
    hipError_t e = hipGetLastError();
    if (e != hipSuccess) fprintf(stderr, "launch failed: %s\n", hipGetErrorString(e));
}
```

```cpp
#include <hip/hip_runtime.h>
#include <rccl/rccl.h>
#include <cstdio>

namespace gr {
typedef unsigned u32;
typedef unsigned long long u64;
typedef unsigned short u16;
typedef u32 u32x4 __attribute__((ext_vector_type(4)));
typedef u32 u32x2 __attribute__((ext_vector_type(2)));
typedef u16 u16x2 __attribute__((ext_vector_type(2)));
typedef short bf16x8 __attribute__((ext_vector_type(8)));
typedef short s16x4 __attribute__((ext_vector_type(4)));
typedef float f32x4 __attribute__((ext_vector_type(4)));
#define LAS __attribute__((address_space(3)))
#define GAS __attribute__((address_space(1)))
typedef __amdgpu_buffer_rsrc_t rsrc_t;

constexpr int NR = 8;
constexpr int NWG = 256, NTH = 512, NGEMM = 192, NCOMM = NWG - NGEMM;
constexpr int GEMM_WAVES = NGEMM * 8, COMM_WAVES = NCOMM * 8;
constexpr u32 SLOT = 8448;
constexpr u32 PAY_Q2 = 5120, PAY_X = 5632;
constexpr u32 XMAX = 128;
constexpr int KP_LANE = 80, KP_BYTES = 64 * KP_LANE;
constexpr u64 DEADLINE = 100000000ull;

constexpr u32 OFF_CALL = 0;
constexpr u32 OFF_TMO = 16384;
constexpr u32 OFF_RDYA = 20480;
constexpr u32 OFF_RDYB = 24576;
constexpr u32 OFF_TKT = 28672;
constexpr u32 OFF_FLG = 32768;
constexpr u32 F_IN = 0;
constexpr u32 F_D3 = F_IN + 2 * 16 * 64 * 4 * 8;
constexpr u32 F_P = F_D3 + 64 * 2 * 16 * 8;
constexpr u32 F_R = F_P + 64 * 16 * 8;
constexpr u32 F_OWN = F_R + 32 * 16 * 8;
constexpr u32 FLG_BYTES = F_OWN + 64 * 4 * 16 * 8;
constexpr u32 OFF_DBG = 512 * 1024;
constexpr u32 CTL_BYTES = 1024 * 1024;
static_assert(OFF_FLG + 2 * FLG_BYTES <= OFF_DBG, "control region");
constexpr u32 IN_BYTES = 4096 * SLOT;
constexpr u32 OUT_BYTES = 192 * 16 * SLOT;
constexpr u32 REL_BYTES = 32 * 16 * SLOT;
constexpr u32 OFF_IN = CTL_BYTES;
constexpr u32 OFF_OUT = OFF_IN + 2 * IN_BYTES;
constexpr u32 OFF_REL = OFF_OUT + 2 * OUT_BYTES;
constexpr u32 OFF_OWN = OFF_REL + 2 * REL_BYTES;
constexpr u32 OFF_A = OFF_OWN + 64 * 4 * 16 * SLOT;
constexpr u32 OFF_B = OFF_A + 16 * 64 * 32768;
constexpr u32 WS_NEED = OFF_B + 16 * 64 * 32768;
static_assert(WS_NEED <= 257u * 1024u * 1024u, "workspace");
static_assert(OFF_A % 256 == 0 && OFF_OUT % 128 == 0 && OFF_OWN % 128 == 0, "alignment");

constexpr u32 META_RAW = 1u << 16;
#define GR_IN_WPL 16
#define GR_C_WPL 15
#define GR_X_WPL 15
constexpr int IN_WPL = GR_IN_WPL, C_WPL = GR_C_WPL, X_WPL = GR_X_WPL;
constexpr int O_WAVES = 3 * C_WPL + 4 * X_WPL, IN_WAVES = 6 * IN_WPL;
#define GR_SEPOUT 1
#define GR_PACK_TAU 40
#define GR_PACK_DMAX 500
#define GR_PACK_M 0xCCEF
#define GR_LOC0 64
#define GR_LOC1 14
constexpr int PACK_TAU = GR_PACK_TAU, PACK_DMAX = GR_PACK_DMAX;
constexpr u32 PACK_M = GR_PACK_M;
constexpr int O_FIRST = GR_SEPOUT ? IN_WAVES : 0;
static_assert(O_FIRST + O_WAVES <= COMM_WAVES && IN_WAVES <= COMM_WAVES, "waves");
#define GR_GAUX 16
constexpr int AUX_SYS = 17, AUX_DEV = GR_GAUX;
constexpr int LDS_STAGE = 131072, LDS_CTL = LDS_STAGE, LDS_BYTES = LDS_STAGE + 256;

struct Params {
    const void* x;
    const void* w;
    void* out;
    void* ws[NR];
    int rank;
    int pad;
    u32 order[512];
};

__device__ __forceinline__ u32 rfl(u32 v) { return (u32)__builtin_amdgcn_readfirstlane((int)v); }
__device__ __forceinline__ u32 pk_add_s(u32 a, u32 s) { u32 r; asm("v_pk_add_u16 %0, %1, %2" : "=v"(r) : "v"(a), "s"(s)); return r; }
__device__ __forceinline__ u32 pk_sub_s(u32 a, u32 s) { u32 r; asm("v_pk_sub_u16 %0, %1, %2" : "=v"(r) : "v"(a), "s"(s)); return r; }
__device__ __forceinline__ u32 bfi(u32 m, u32 a, u32 b) { u32 r; asm("v_bfi_b32 %0, %1, %2, %3" : "=v"(r) : "s"(m), "v"(a), "v"(b)); return r; }
__device__ __forceinline__ u32 rotl(u32 x, int r) { return __builtin_amdgcn_alignbit(x, x, (u32)(32 - r)); }
__device__ __forceinline__ u32 rotr(u32 x, int r) { return __builtin_amdgcn_alignbit(x, x, (u32)r); }
__device__ __forceinline__ u32 ffbl(u32 x) { u32 r; asm("v_ffbl_b32 %0, %1" : "=v"(r) : "v"(x)); return r; }
__device__ __forceinline__ u32 wave_scan(u32 x) {
    x += (u32)__builtin_amdgcn_update_dpp(0, (int)x, 0x111, 0xF, 0xF, false);
    x += (u32)__builtin_amdgcn_update_dpp(0, (int)x, 0x112, 0xF, 0xF, false);
    x += (u32)__builtin_amdgcn_update_dpp(0, (int)x, 0x114, 0xF, 0xF, false);
    x += (u32)__builtin_amdgcn_update_dpp(0, (int)x, 0x118, 0xF, 0xF, false);
    x += (u32)__builtin_amdgcn_update_dpp(0, (int)x, 0x142, 0xA, 0xF, false);
    x += (u32)__builtin_amdgcn_update_dpp(0, (int)x, 0x143, 0xC, 0xF, false);
    return x;
}
__device__ __forceinline__ u32 cvt_pk_bf16(float lo, float hi) { u32 r; asm volatile("v_cvt_pk_bf16_f32 %0, %1, %2" : "=v"(r) : "v"(lo), "v"(hi)); return r; }
__device__ __forceinline__ rsrc_t mk_rsrc(void* p) { return __builtin_amdgcn_make_buffer_rsrc(p, (short)0, (int)WS_NEED, 0x00020000); }
__device__ __forceinline__ void wait_vm0() { asm volatile("s_waitcnt vmcnt(0)" ::: "memory"); }
__device__ __forceinline__ void wait_vm0_known() { __builtin_amdgcn_s_waitcnt(0x0F70); asm volatile("" ::: "memory"); }

struct Ctx {
    char* ws;
    rsrc_t own;
    u32 call, par;
    int rank, lane;
    LAS unsigned char* kp;
};
#define JLOG(list, job, sub, k) do {} while (0)
#define DT0() do {} while (0)
#define DT(i) do {} while (0)
#define DDEP(d) do {} while (0)

__device__ __forceinline__ u64 flag_load(const Ctx& c, u32 foff) {
    return __hip_atomic_load((GAS const u64*)(c.ws + OFF_FLG + c.par * FLG_BYTES + foff), __ATOMIC_RELAXED, __HIP_MEMORY_SCOPE_SYSTEM);
}
__device__ __forceinline__ bool flag_test(const Ctx& c, u64 v, u32& meta) {
    meta = rfl((u32)v);
    return (int)(rfl((u32)(v >> 32)) - c.call) >= 0;
}
__device__ __forceinline__ bool wait_flag(const Ctx& c, u32 foff, u32& meta) {
    GAS u32* tmo = (GAS u32*)(c.ws + OFF_TMO);
    if (flag_test(c, flag_load(c, foff), meta)) return true;
    const u64 t0 = __builtin_amdgcn_s_memrealtime();
    for (;;) {
        __builtin_amdgcn_s_sleep(2);
        if (flag_test(c, flag_load(c, foff), meta)) return true;
        if (rfl(__hip_atomic_load(tmo, __ATOMIC_RELAXED, __HIP_MEMORY_SCOPE_AGENT)) == c.call || __builtin_amdgcn_s_memrealtime() - t0 > DEADLINE) {
            __hip_atomic_store(tmo, c.call, __ATOMIC_RELAXED, __HIP_MEMORY_SCOPE_AGENT);
            meta = 0; return false;
        }
    }
}
__device__ __forceinline__ void post_flag(const Params& P, const Ctx& c, int dst, u32 foff, u32 meta) {
    if ((unsigned)dst >= (unsigned)NR) return;
    GAS u64* f = (GAS u64*)((char*)P.ws[dst] + OFF_FLG + c.par * FLG_BYTES + foff);
    if (c.lane == 0) __hip_atomic_store(f, ((u64)c.call << 32) | meta, __ATOMIC_RELAXED, __HIP_MEMORY_SCOPE_SYSTEM);
}

__device__ __forceinline__ void store_raw(const u32x4 (&d)[8], rsrc_t rs, u32 slot, int lane) {
#pragma unroll
    for (int v = 0; v < 8; ++v) __builtin_amdgcn_raw_buffer_store_b128(d[v], rs, (int)(slot + v * 1024 + lane * 16), 0, AUX_SYS);
}
__device__ __forceinline__ u32 kz_of(u32 L) { return ((0x8000u - L) & 0x7FFFu) >> 10; }
__device__ __forceinline__ u32 pack_store(const u32x4 (&d)[8], rsrc_t rs, u32 slot, int lane, LAS unsigned char* stage) {
    float m0 = 0.f, m1 = 0.f;
#pragma unroll
    for (int f = 0; f < 8; ++f) {
        asm("v_max3_f32 %0, |%1|, |%2|, %0" : "+v"(m0) : "v"(d[f][0]), "v"(d[f][1]));
        asm("v_max3_f32 %0, |%1|, |%2|, %0" : "+v"(m1) : "v"(d[f][2]), "v"(d[f][3]));
    }
    const u32 b0 = __builtin_bit_cast(u32, m0), b1 = __builtin_bit_cast(u32, m1);
    const u32 sum = (u32)__builtin_amdgcn_readlane((int)wave_scan(((b0 > b1 ? b0 : b1) >> 16) & 0x7FFFu), 63);
    const u32 top = ((sum >> 6) + 32u) >> 7, lo = top > 7u ? top - 7u : 0u, L = lo << 7, LL = L | (L << 16);
    u32 lb[16], hb[16], M[2] = {0u, 0u};
#pragma unroll
    for (int j = 0; j < 16; ++j) {
        const u32 w0 = pk_sub_s(d[j >> 1][(j & 1) * 2], LL), w1 = pk_sub_s(d[j >> 1][(j & 1) * 2 + 1], LL);
        lb[j] = __builtin_amdgcn_perm(w1, w0, 0x06040200u); hb[j] = __builtin_amdgcn_perm(w1, w0, 0x07050301u);
        M[j >> 3] = bfi(0x80808080u, (hb[j] & 0x7C7C7C7Cu) + 0x7C7C7C7Cu, M[j >> 3] >> 1);
    }
    const u32 c0 = (u32)__builtin_popcount(M[0]), c1 = (u32)__builtin_popcount(M[1]), incl = wave_scan(c0 + c1);
    const u32 cnt = (u32)__builtin_amdgcn_readlane((int)incl, 63);
    if (cnt > XMAX) { store_raw(d, rs, slot, lane); return META_RAW; }
    u32 Q[6];
#pragma unroll
    for (int g = 0; g < 2; ++g) {
        const u32* h = hb + 8 * g;
        Q[3 * g + 0] = bfi(0x60606060u, rotl(h[6], 6), bfi(0x1C1C1C1Cu, rotl(h[1], 3), h[0]));
        Q[3 * g + 1] = bfi(0x40404040u, rotl(h[7], 5), bfi(0x20202020u, rotl(h[6], 4), bfi(0x1C1C1C1Cu, rotl(h[3], 3), h[2])));
        Q[3 * g + 2] = bfi(0x60606060u, rotl(h[7], 6), bfi(0x1C1C1C1Cu, rotl(h[5], 3), h[4]));
    }
#pragma unroll
    for (int v = 0; v < 4; ++v)
        __builtin_amdgcn_raw_buffer_store_b128((u32x4){lb[4 * v], lb[4 * v + 1], lb[4 * v + 2], lb[4 * v + 3]}, rs, (int)(slot + v * 1024 + lane * 16), 0, AUX_SYS);
    __builtin_amdgcn_raw_buffer_store_b128((u32x4){Q[0], Q[1], Q[2], Q[3]}, rs, (int)(slot + 4096 + lane * 16), 0, AUX_SYS);
    __builtin_amdgcn_raw_buffer_store_b64((u32x2){Q[4], Q[5]}, rs, (int)(slot + PAY_Q2 + lane * 8), 0, AUX_SYS);
    if (cnt) {
        LAS unsigned char* mine = stage + lane * KP_LANE;
        LAS u16* list = (LAS u16*)(stage + KP_BYTES);
#pragma unroll
        for (int i = 0; i < 4; ++i) *(LAS u32x4*)(mine + 16 * i) = (u32x4){hb[4 * i], hb[4 * i + 1], hb[4 * i + 2], hb[4 * i + 3]};
        const u32 kz = kz_of(L);
        u32 off[2] = {incl - c0 - c1, incl - c1};
        bool bad = false;
        do {
#pragma unroll
            for (int g = 0; g < 2; ++g) {
                const bool has = M[g] != 0u;
                const u32 p = ffbl(M[g]), pos = (u32)(g * 32) + ((p & 7u) << 2) + ((p >> 3) & 3u);
                M[g] &= M[g] - 1u;
                const u32 k5 = ((u32)mine[pos] >> 2) & 31u;
                const bool z = k5 == kz;
                bad |= has && !z && ((k5 + 7u) & 31u) >= 15u;
                list[has ? off[g] : XMAX] = (u16)((((u32)lane * 64u + pos) << 4) | (z ? 8u : k5 & 15u));
                off[g] += has ? 1u : 0u;
            }
        } while (__builtin_amdgcn_ballot_w64((M[0] | M[1]) != 0u));
        if (__builtin_amdgcn_ballot_w64(bad)) { wait_vm0_known(); store_raw(d, rs, slot, lane); return META_RAW; }
        const u32 e = ((LAS const u32*)list)[lane];
        if (2u * (u32)lane < cnt) __builtin_amdgcn_raw_buffer_store_b32(e, rs, (int)(slot + PAY_X + lane * 4), 0, AUX_SYS);
    }
    return lo | (cnt << 8);
}
struct Pull { u32x4 V[5]; u32x2 q; u32 h; };
__device__ __forceinline__ void pull_issue(Pull& p, rsrc_t rs, u32 slot, u32 meta, int lane) {
#pragma unroll
    for (int v = 0; v < 5; ++v) p.V[v] = __builtin_amdgcn_raw_buffer_load_b128(rs, (int)(slot + v * 1024 + lane * 16), 0, AUX_SYS);
    p.q = __builtin_amdgcn_raw_buffer_load_b64(rs, (int)(slot + PAY_Q2 + lane * 8), 0, AUX_SYS);
    const u32 hoff = 2u * (u32)lane < ((meta >> 8) & 0xFFu) ? slot + PAY_X + (u32)lane * 4u : 0x80000000u;
    p.h = __builtin_amdgcn_raw_buffer_load_b32(rs, (int)hoff, 0, AUX_SYS);
}
__device__ __forceinline__ void kp_clear(LAS unsigned char* kp, int lane) {
#pragma unroll
    for (int i = 0; i < 5; ++i) *(LAS u32x4*)(kp + lane * KP_LANE + 16 * i) = (u32x4){0u, 0u, 0u, 0u};
}
__device__ __forceinline__ void unpack(u32x4 (&d)[8], const Pull& p, rsrc_t rs, u32 slot, u32 meta, int lane, LAS unsigned char* kp) {
    if (meta & META_RAW) {
#pragma unroll
        for (int v = 0; v < 5; ++v) d[v] = p.V[v];
#pragma unroll
        for (int v = 5; v < 8; ++v) d[v] = __builtin_amdgcn_raw_buffer_load_b128(rs, (int)(slot + v * 1024 + lane * 16), 0, AUX_SYS);
        wait_vm0_known();
        return;
    }
    const u32 cnt = (meta >> 8) & 0xFFu;
    const u32 L = (meta & 0xFFu) << 7, LL = L | (L << 16), kz = kz_of(L);
    u32 at[2];
#pragma unroll
    for (int i = 0; i < 2; ++i) {
        const u32 e = i ? p.h >> 16 : p.h & 0xFFFFu, pos = e >> 4, k4 = e & 15u;
        at[i] = 2u * (u32)lane + (u32)i < cnt ? pos + (pos >> 6) * 16u : (u32)KP_BYTES;
        kp[at[i]] = (unsigned char)((k4 == 8u ? kz : k4 | ((k4 & 8u) << 1)) << 2);
    }
    u32x4 K[4];
#pragma unroll
    for (int i = 0; i < 4; ++i) K[i] = *(LAS const u32x4*)(kp + lane * KP_LANE + 16 * i);
    kp[at[0]] = 0; kp[at[1]] = 0;
    const u32 Q[6] = {p.V[4][0], p.V[4][1], p.V[4][2], p.V[4][3], p.q[0], p.q[1]};
#pragma unroll
    for (int g = 0; g < 2; ++g) {
        const u32 q0 = Q[3 * g], q1 = Q[3 * g + 1], q2 = Q[3 * g + 2];
        const u32x4 k0 = K[2 * g], k1 = K[2 * g + 1];
        u32 h[8];
        h[0] = (q0 & 0x83838383u) | k0[0]; h[1] = (rotr(q0, 3) & 0x83838383u) | k0[1];
        h[2] = (q1 & 0x83838383u) | k0[2]; h[3] = (rotr(q1, 3) & 0x83838383u) | k0[3];
        h[4] = (q2 & 0x83838383u) | k1[0]; h[5] = (rotr(q2, 3) & 0x83838383u) | k1[1];
        h[6] = (rotr(q0, 6) & 0x81818181u) | ((rotr(q1, 4) & 0x02020202u) | k1[2]);
        h[7] = (rotr(q2, 6) & 0x81818181u) | ((rotr(q1, 5) & 0x02020202u) | k1[3]);
#pragma unroll
        for (int j = 0; j < 8; ++j) {
            const u32 B = p.V[2 * g + (j >> 2)][j & 3];
            d[4 * g + (j >> 1)][(j & 1) * 2] = pk_add_s(__builtin_amdgcn_perm(h[j], B, 0x05010400u), LL);
            d[4 * g + (j >> 1)][(j & 1) * 2 + 1] = pk_add_s(__builtin_amdgcn_perm(h[j], B, 0x07030602u), LL);
        }
    }
}
__device__ __forceinline__ void fill_nan(u32x4 (&d)[8]) {
#pragma unroll
    for (int v = 0; v < 8; ++v) d[v] = (u32x4){0x7FC07FC0u, 0x7FC07FC0u, 0x7FC07FC0u, 0x7FC07FC0u};
}
__device__ __forceinline__ void copy_store(const Ctx& c, const Pull& p, rsrc_t rs, u32 slot, u32 dst, u32 meta) {
    const int lane = c.lane;
    if (meta & META_RAW) {
        u32x4 a[3];
#pragma unroll
        for (int v = 0; v < 3; ++v) a[v] = __builtin_amdgcn_raw_buffer_load_b128(rs, (int)(slot + (5 + v) * 1024 + lane * 16), 0, AUX_SYS);
        wait_vm0_known();
#pragma unroll
        for (int v = 0; v < 3; ++v) __builtin_amdgcn_raw_buffer_store_b128(a[v], c.own, (int)(dst + (5 + v) * 1024 + lane * 16), 0, AUX_SYS);
    } else {
        __builtin_amdgcn_raw_buffer_store_b64(p.q, c.own, (int)(dst + PAY_Q2 + lane * 8), 0, AUX_SYS);
        if (2u * (u32)lane < ((meta >> 8) & 0xFFu)) __builtin_amdgcn_raw_buffer_store_b32(p.h, c.own, (int)(dst + PAY_X + lane * 4), 0, AUX_SYS);
    }
#pragma unroll
    for (int v = 0; v < 5; ++v) __builtin_amdgcn_raw_buffer_store_b128(p.V[v], c.own, (int)(dst + v * 1024 + lane * 16), 0, AUX_SYS);
}

__device__ __forceinline__ u32 ticket_issue(const Ctx& c, u32 off) {
    u32 r; u64 sv; const u32 one = 1u;
    asm volatile("s_mov_b64 %1, exec\n\ts_mov_b64 exec, 1\n\tglobal_atomic_add %0, %2, %3, %4 sc0\n\ts_mov_b64 exec, %1"
                 : "=&v"(r), "=&s"(sv) : "v"(off), "v"(one), "s"(c.ws) : "memory");
    return r;
}
__device__ __forceinline__ u32 ticket_get(u32 r) { asm volatile("" : "+v"(r) :: "memory"); return rfl(r); }

struct Step { u32 foff, slot, meta; int sel, p0, p1, p2, p3; };
template <int SPT, class Gen, class Act, class Pub>
__device__ __forceinline__ void pull_list(Ctx& c, int list, u32 first, u32 nwaves, u32 njobs, rsrc_t rs0, rsrc_t rs1, Gen gen, Act act, Pub pub) {
    DT0();
    const u32 tkoff = OFF_TKT + (u32)list * 128u, tkbase = nwaves - (c.call - 1u) * njobs;
    u32 jc = first, jn = 0, tk = 0;
    if (SPT == 1) tk = ticket_issue(c, tkoff);
    Step cur = gen(jc, 0), prev = cur;
    bool ok = wait_flag(c, cur.foff, cur.meta), have_prev = false, more = true;
    if (SPT == 1) { wait_vm0_known(); jn = ticket_get(tk) + tkbase; }
    DT(1);
    Pull pa, pb;
    JLOG(list, jc, 0, 0);
    pull_issue(pa, cur.sel ? rs1 : rs0, cur.slot, cur.meta, c.lane);
    auto half = [&](const Pull& p, Pull& pn, int sub) {
        const bool same = SPT == 2 && sub == 0;
        more = same || jn < njobs;
        Step nxt = gen(same || !more ? jc : jn, same ? 1 : 0);
        const bool draw = SPT == 1 ? more : same;
        if (draw) tk = ticket_issue(c, tkoff);
        const u64 fv = flag_load(c, nxt.foff);
        wait_vm0_known();
        DT(0);
        JLOG(list, jc, sub, 1);
        const u32 jd = draw ? ticket_get(tk) + tkbase : njobs;
        if (have_prev) pub(prev);
        const bool nready = more && flag_test(c, fv, nxt.meta);
        if (nready) { JLOG(list, same ? jc : jn, same ? 1 : 0, 0); pull_issue(pn, nxt.sel ? rs1 : rs0, nxt.slot, nxt.meta, c.lane); }
        DT(3);
        act(cur, p, cur.sel ? rs1 : rs0, ok);
        prev = cur; have_prev = true; ok = true;
        DT(2);
        if (more && !nready) {
            wait_vm0_known(); pub(prev); have_prev = false;
            ok = wait_flag(c, nxt.foff, nxt.meta);
            JLOG(list, same ? jc : jn, same ? 1 : 0, 0);
            pull_issue(pn, nxt.sel ? rs1 : rs0, nxt.slot, nxt.meta, c.lane);
            DT(1);
        }
        cur = nxt;
        if (SPT == 1) { jc = jn; jn = jd; }
        else if (same) jn = jd; else jc = jn;
    };
    for (;;) {
        half(pa, pb, 0); if (!more) break;
        half(pb, pa, 1); if (!more) break;
    }
    wait_vm0_known();
    if (have_prev) pub(prev);
}

__host__ __device__ __forceinline__ int cq_of(int q, int r) {
    return ((r >> 1) ^ ((q & 1) ? (r >> 2) : 0) ^ ((q & 2) ? r : 0)) & 1;
}
__host__ __device__ __forceinline__ int link_d(int link) { return (0x765431 >> (4 * link)) & 7; }
__host__ __device__ __forceinline__ int link_q(int link, int i) { return ((i ? 0x233231 : 0x110020) >> (4 * link)) & 3; }
__host__ __device__ __forceinline__ int link_of_d(int d) { return d == 1 ? 0 : d - 2; }
__host__ __device__ __forceinline__ int t_of(int d, int q, int kq) {
    if (d == 0) return kq * 4 + q;
    const int link = link_of_d(d);
    return 16 + (2 * kq + (link_q(link, 0) == q ? 0 : 1)) * 6 + link;
}
__host__ __device__ __forceinline__ int d3_pair(int d) { return (d & 1) * 2 + ((d >> 1) & 1); }
__host__ __device__ __forceinline__ int d3_of_pair(int p) { return 4 + (p >> 1) + (p & 1) * 2; }
__host__ __device__ __forceinline__ int rowtile_of(int rc, int o, int a) {
    const int d = rc ^ o;
    return d == 0 ? 12 + a : d == 2 ? a : 4 + 2 * d3_pair(d) + (a & 1);
}
__device__ __forceinline__ void block_of_rowtile(int rc, int i, int& o, int& a) {
    if (i >= 12) { o = rc; a = i - 12; }
    else if (i < 4) { o = rc ^ 2; a = i; }
    else { const int p = (i - 4) >> 1; o = rc ^ d3_of_pair(p); a = 2 * (p >> 1) + ((i - 4) & 1); }
}
__device__ __forceinline__ u32 raw_frag_off(int type, int sub, int f) {
    return type == 0 ? (u32)(sub * 8192 + f * 1024) : (u32)((f >> 2) * 16384 + sub * 4096 + (f & 3) * 1024);
}
__device__ __forceinline__ void store_operand(const Ctx& c, const u32x4 (&d)[8], int type, int tile, int t, int sub) {
    const u32 base = (type == 0 ? OFF_A : OFF_B) + (u32)(tile * 64 + t) * 32768u + (u32)c.lane * 16u;
#pragma unroll
    for (int f = 0; f < 8; ++f) __builtin_amdgcn_raw_buffer_store_b128(d[f], c.own, (int)(base + raw_frag_off(type, sub, f)), 0, AUX_SYS);
}
__device__ __forceinline__ void count_operand(const Ctx& c, int type, int tile, int t) {
    GAS u32* p = (GAS u32*)(c.ws + (type == 0 ? OFF_RDYA : OFF_RDYB)) + tile * 64 + t;
    if (c.lane == 0) __hip_atomic_fetch_add(p, 1u, __ATOMIC_RELAXED, __HIP_MEMORY_SCOPE_AGENT);
}

__host__ __device__ __forceinline__ void consumers_of(int s, int pj, int (&rc)[2], int (&tl)[2], int (&tt)[2]) {
    const int kq = pj >> 10, q = pj & 3, type = (pj >> 2) & 1, g = (pj & 1023) >> 3;
    const int cs = cq_of(q, s);
#pragma unroll
    for (int i = 0; i < 2; ++i) {
        int e1, gam;
        if (type == 0) { const int o = g >> 4, h = (g >> 3) & 1; gam = i; e1 = h == 0 ? (o & 1) : ((o ^ (o >> 2) ^ gam) & 1); }
        else { gam = g >> 6; e1 = i; }
        const int e2 = (cs ^ ((q & 2) ? e1 : 0) ^ ((q & 1) ? gam : 0)) & 1;
        rc[i] = e1 | (e2 << 1) | (gam << 2);
        tl[i] = type == 0 ? rowtile_of(rc[i], g >> 4, (g >> 2) & 3) : ((g >> 2) & 15);
        tt[i] = t_of(rc[i] ^ s, q, kq);
    }
}
__host__ __device__ __forceinline__ int need_of(int s, int pj) {
    int rc[2], tl[2], tt[2], need = 1024;
    consumers_of(s, pj, rc, tl, tt);
    const int type = (pj >> 2) & 1, sub = (pj >> 3) & 3;
    for (int i = 0; i < 2; ++i) {
        const int pos = type == 0 && tl[i] >= 12 ? 896 : rc[i] == s ? GR_LOC0 + GR_LOC1 * tt[i] : ((tt[i] - 16) / 6) * 112 + type * 48 + tl[i] * 4 + sub;
        if (pos < need) need = pos;
    }
    return need;
}
__device__ __forceinline__ void pack_issue(const Params& P, int lane, LAS unsigned char* tile, int pj, u32x4 (&d)[8]) {
    const int kq = pj >> 10, q = pj & 3, type = (pj >> 2) & 1, g = (pj & 1023) >> 3, kt = 4 * q + kq;
    if (type == 0) {
        GAS const char* src = (GAS const char*)P.x + (size_t)(64 * g + (lane & 15)) * 2048 + (64 * kt + 8 * (lane >> 4)) * 2;
#pragma unroll
        for (int f = 0; f < 8; ++f) d[f] = *(GAS const u32x4*)(src + (f >> 1) * 16 * 2048 + (f & 1) * 64);
    } else {
        GAS const char* src = (GAS const char*)P.w + (size_t)(64 * kt + (lane >> 3)) * 16384 + (64 * g) * 2 + (lane & 7) * 16;
#pragma unroll
        for (int v = 0; v < 8; ++v)
            __builtin_amdgcn_global_load_lds((GAS const u32*)(src + (size_t)v * 8 * 16384), (LAS u32*)(tile + v * 1024), 16, 0, 0);
    }
}
__device__ __forceinline__ void pack_job(const Params& P, const Ctx& c, LAS unsigned char* tile, LAS unsigned char* tile2, int pj, int next, u32x4 (&in)[8]) {
    const int kq = pj >> 10, q = pj & 3, type = (pj >> 2) & 1, g = (pj & 1023) >> 3, kt = 4 * q + kq, lane = c.lane, s = c.rank;
    u32x4 d[8];
    if (type == 0) {
#pragma unroll
        for (int f = 0; f < 8; ++f) d[f] = in[f];
    } else {
        wait_vm0();
        const int lg = lane >> 4, lq = (lane >> 2) & 3, lp = lane & 3;
        LAS unsigned char* b = tile + (8 * lg + lq) * 128 + lp * 16;
#pragma unroll
        for (int f = 0; f < 8; ++f) {
            const int o = (f & 1) * 32 * 128 + (f >> 2) * 64 + ((f >> 1) & 1) * 8;
            const s16x4 lo4 = __builtin_amdgcn_ds_read_tr16_b64_v4i16((LAS s16x4*)(b + o));
            const s16x4 hi4 = __builtin_amdgcn_ds_read_tr16_b64_v4i16((LAS s16x4*)(b + o + 4 * 128));
            const u32x2 l2 = __builtin_bit_cast(u32x2, lo4), h2 = __builtin_bit_cast(u32x2, hi4);
            d[f] = (u32x4){l2[0], l2[1], h2[0], h2[1]};
        }
        asm volatile("s_waitcnt lgkmcnt(0)" ::: "memory");
    }
    if (next >= 0) pack_issue(P, lane, tile2, next, in);
    int rc[2], tl[2], tt[2];
    consumers_of(s, pj, rc, tl, tt);
    const int sub = g & 3;
    const u32 meta = pack_store(d, c.own, OFF_IN + c.par * IN_BYTES + (u32)(type * 2048 + kt * 128 + g) * SLOT, lane, tile);
    if (rc[0] == s) store_operand(c, d, type, tl[0], tt[0], sub);
    if (rc[1] == s) store_operand(c, d, type, tl[1], tt[1], sub);
    wait_vm0();
#pragma unroll
    for (int i = 0; i < 2; ++i) {
        if (rc[i] == s) count_operand(c, type, tl[i], tt[i]);
        else post_flag(P, c, rc[i], F_IN + (u32)(((type * 16 + tl[i]) * 64 + tt[i]) * 4 + sub) * 8u, meta);
    }
}
__device__ __forceinline__ int pack_pick(const Params& P, int grp, int u, int r, int& urg) {
    int off = 0;
#pragma unroll
    for (int i = 0; i < 6; ++i) if (i < r) off += 384 * __builtin_popcount((PACK_M >> (4 * i)) & 15u);
    const u32 m = (PACK_M >> (4 * r)) & 15u;
    const int i = off + 384 * __builtin_popcount(m & ((1u << grp) - 1u)) + u;
    if (!((m >> grp) & 1u) || i >= 4096) return -1;
    const u32 e = (P.order[(i & 1023) >> 1] >> (16 * (i & 1))) & 0xFFFFu;
    urg = i < 1024 ? (int)(e >> 10) : 63;
    return (i & ~1023) | (int)(e & 1023u);
}
__device__ __forceinline__ void pack_list(const Params& P, const Ctx& c, int j0, int j1, int j2, int j3, int j4, u32x4 (&in)[8]) {
    for (u32 buf = 0; j0 >= 0; buf ^= 8192u) { pack_job(P, c, c.kp + buf, c.kp + (buf ^ 8192u), j0, j1, in); j0 = j1; j1 = j2; j2 = j3; j3 = j4; j4 = -1; }
}

__device__ __forceinline__ void in_list(const Params& P, Ctx& c, int link, int widx, int wpl) {
    const int s = c.rank ^ link_d(link);
    auto gen = [&](u32 job, int) {
        const int j = (int)job;
        int kc, type, tile, sub;
        if (j < 896) { kc = j / 112; const int u = j % 112; if (u < 48) { type = 0; tile = u >> 2; } else { type = 1; tile = (u - 48) >> 2; } sub = u & 3; }
        else { const int m = j - 896; kc = m >> 4; type = 0; tile = 12 + ((m >> 2) & 3); sub = m & 3; }
        const int t = 16 + kc * 6 + link, kt = 4 * link_q(link, kc & 1) + (kc >> 1);
        int g;
        if (type == 0) { int o, a; block_of_rowtile(c.rank, tile, o, a); g = o * 16 + a * 4 + sub; } else g = (c.rank >> 2) * 64 + tile * 4 + sub;
        Step st; st.foff = F_IN + (u32)(((type * 16 + tile) * 64 + t) * 4 + sub) * 8u;
        st.slot = OFF_IN + c.par * IN_BYTES + (u32)(type * 2048 + kt * 128 + g) * SLOT;
        st.sel = 0; st.meta = 0; st.p0 = type; st.p1 = tile; st.p2 = t; st.p3 = sub;
        return st;
    };
    auto act = [&](Step& st, const Pull& p, rsrc_t rs, bool ok) {
        u32x4 d[8];
        if (ok) unpack(d, p, rs, st.slot, st.meta, c.lane, c.kp); else fill_nan(d);
        store_operand(c, d, st.p0, st.p1, st.p2, st.p3);
    };
    auto pub = [&](const Step& st) { count_operand(c, st.p0, st.p1, st.p2); };
    const rsrc_t rs = mk_rsrc(P.ws[s & 7]);
    pull_list<1>(c, link, (u32)widx, (u32)wpl, 1024u, rs, rs, gen, act, pub);
}

__device__ __forceinline__ void acc_add(float (&acc)[64], const u32x4 (&d)[8]) {
#pragma unroll
    for (int q = 0; q < 32; ++q) { const u32 v = d[q >> 2][q & 3]; acc[2 * q] += __builtin_bit_cast(float, v << 16); acc[2 * q + 1] += __builtin_bit_cast(float, v & 0xFFFF0000u); }
}
__device__ __forceinline__ void write_out(const Params& P, const Ctx& c, const float (&acc)[64], int a, int cj, int jb) {
    const int w = jb >> 1, ai = jb & 1, lane = c.lane;
    GAS char* o = (GAS char*)P.out + (size_t)(a * 256 + ai * 128 + (w >> 2) * 64 + (lane & 15)) * 16384 + (cj * 256 + (w & 3) * 64 + 8 * (lane >> 4)) * 2;
#pragma unroll
    for (int f = 0; f < 8; ++f) {
        u32x4 v;
#pragma unroll
        for (int e = 0; e < 4; ++e) v[e] = cvt_pk_bf16(acc[f * 8 + 2 * e], acc[f * 8 + 2 * e + 1]);
        *(GAS u32x4*)(o + (size_t)(f >> 1) * 16 * 16384 + (f & 1) * 64) = v;
    }
}
__device__ __forceinline__ void copy_list(const Params& P, Ctx& c, int grp, int widx, int wpl) {
    const int r = c.rank;
    auto gen = [&](u32 job, int) {
        const int j = (int)job, jb = j & 15;
        Step st; st.meta = 0;
        if (grp == 0) {
            const int u = j >> 4, pt = (u & 3) * 16 + (u >> 2) * 2;
            st.foff = F_P + (u32)(pt * 16 + jb) * 8u; st.slot = OFF_OUT + c.par * OUT_BYTES + (u32)(pt * 16 + jb) * SLOT;
            st.p0 = (int)(OFF_OWN + (u32)((pt * 4 + 3) * 16 + jb) * SLOT); st.p1 = r; st.p2 = (int)(F_OWN + (u32)((pt * 4 + 3) * 16 + jb) * 8u);
        } else {
            const int second = j >= 256, v = (j >> 4) & 15, u = (v & 3) * 4 + (v >> 2), path = (grp - 1) ^ second, pt = 4 * u + 1 + 2 * path, rs = path * 16 + u;
            if (!second) {
                st.foff = F_R + (u32)(rs * 16 + jb) * 8u; st.slot = OFF_OUT + c.par * OUT_BYTES + (u32)(pt * 16 + jb) * SLOT;
                st.p0 = (int)(OFF_REL + c.par * REL_BYTES + (u32)(rs * 16 + jb) * SLOT); st.p1 = r ^ (path ? 1 : 3); st.p2 = (int)(F_P + (u32)(pt * 16 + jb) * 8u);
            } else {
                st.foff = F_P + (u32)(pt * 16 + jb) * 8u; st.slot = OFF_REL + c.par * REL_BYTES + (u32)(rs * 16 + jb) * SLOT;
                st.p0 = (int)(OFF_OWN + (u32)((pt * 4 + 3) * 16 + jb) * SLOT); st.p1 = r; st.p2 = (int)(F_OWN + (u32)((pt * 4 + 3) * 16 + jb) * 8u);
            }
        }
        st.p3 = 0; st.sel = 0;
        return st;
    };
    auto act = [&](Step& st, const Pull& p, rsrc_t rs, bool ok) {
        if (ok) copy_store(c, p, rs, st.slot, (u32)st.p0, st.meta);
        else { u32x4 d[8]; fill_nan(d); store_raw(d, c.own, (u32)st.p0, c.lane); st.meta = META_RAW; }
    };
    auto pub = [&](const Step& st) { post_flag(P, c, st.p1, (u32)st.p2, st.meta); };
    const rsrc_t rs = mk_rsrc(P.ws[(r ^ (grp == 0 ? 2 : grp == 1 ? 1 : 3)) & 7]);
    pull_list<1>(c, 6 + grp, (u32)widx, (u32)wpl, 512u, rs, rs, gen, act, pub);
}
__device__ __forceinline__ void reduce_list(const Params& P, Ctx& c, int h, int widx, int wpl) {
    const int r = c.rank;
    float acc[64];
    auto gen = [&](u32 ujob, int sub) {
        const int job = (int)ujob, sl = (sub ^ job) & 1, jb = job & 15, ot2 = job >> 4;
        const int a = 2 * h + (ot2 & 1), j = ot2 >> 1, ot = a * 16 + j, p = 2 * h + sl, ti = 4 + 2 * p + (a & 1);
        Step st; st.meta = 0;
        st.foff = F_D3 + (u32)((ot * 2 + sl) * 16 + jb) * 8u; st.sel = sl;
        st.slot = OFF_OUT + c.par * OUT_BYTES + (u32)((ti * 16 + j) * 16 + jb) * SLOT;
        st.p0 = sub; st.p1 = a; st.p2 = (1 - (r >> 2)) * 16 + j; st.p3 = jb;
        return st;
    };
    auto act = [&](Step& st, const Pull& p, rsrc_t rs, bool ok) {
        u32x4 d[8];
        if (ok) unpack(d, p, rs, st.slot, st.meta, c.lane, c.kp); else fill_nan(d);
        if (st.p0 == 0) {
#pragma unroll
            for (int q = 0; q < 64; ++q) acc[q] = 0.f;
        }
        acc_add(acc, d);
        if (st.p0 == 1) write_out(P, c, acc, st.p1, st.p2, st.p3);
    };
    auto pub = [&](const Step&) {};
    pull_list<2>(c, 9 + h, (u32)widx, (u32)wpl, 512u, mk_rsrc(P.ws[(r ^ d3_of_pair(2 * h)) & 7]), mk_rsrc(P.ws[(r ^ d3_of_pair(2 * h + 1)) & 7]), gen, act, pub);
}
__device__ __forceinline__ void own_job(const Params& P, const Ctx& c, int n) {
    const int pt = n >> 4, jb = n & 15;
    kp_clear(c.kp, c.lane);
    float acc[64];
#pragma unroll
    for (int i = 0; i < 64; ++i) acc[i] = 0.f;
    for (int part = 0; part < 4; ++part) {
        u32 meta; u32x4 d[8];
        if (wait_flag(c, F_OWN + (u32)((pt * 4 + part) * 16 + jb) * 8u, meta)) {
            Pull p; const u32 slot = OFF_OWN + (u32)((pt * 4 + part) * 16 + jb) * SLOT;
            pull_issue(p, c.own, slot, meta, c.lane); unpack(d, p, c.own, slot, meta, c.lane, c.kp);
        } else fill_nan(d);
        acc_add(acc, d);
    }
    write_out(P, c, acc, pt >> 4, (c.rank >> 2) * 16 + (pt & 15), jb);
}

struct GJob { int ti, tj, t0, nt; };

#define STAMP(i) do {} while (0)

__device__ __forceinline__ void gemm_role(const Params& P, const Ctx& c, LAS unsigned char* lds, int xcd, int idx) {
    const int tid = threadIdx.x, wid = (int)rfl(tid >> 6), lane = c.lane, wr = wid >> 2, wc = wid & 3;
    constexpr int HTB = 16384;
    const u32 voff = (u32)tid * 16u;
    const unsigned ldsw = (unsigned)wid * 1024u;
    const int aoff = wr * 8192 + lane * 16, boff = wc * 4096 + lane * 16;
    LAS volatile int* ctl = (LAS volatile int*)(lds + LDS_CTL);
    int avail = 0; bool bad = false;
#define G_SA(b, h) (((b) * 2 + (h)) * HTB)
#define G_SB(b, h) ((4 + (b) * 2 + (h)) * HTB)
#define G_STAGE(bufoff, gbase) do { _Pragma("unroll") for (int _i = 0; _i < 2; ++_i) \
        __builtin_amdgcn_global_load_lds((GAS const u32*)((GAS const char*)(gbase) + voff + _i * 8192), (LAS u32*)(lds + (bufoff) + ldsw + _i * 8192), 16, 0, AUX_DEV); } while (0)
#define G_LDA(dst, b, h) do { _Pragma("unroll") for (int m = 0; m < 4; ++m) _Pragma("unroll") for (int k = 0; k < 2; ++k) dst[m][k] = *(const LAS bf16x8*)(lds + G_SA(b, h) + aoff + m * 2048 + k * 1024); } while (0)
#define G_LDB(dst, b, h) do { _Pragma("unroll") for (int n = 0; n < 2; ++n) _Pragma("unroll") for (int k = 0; k < 2; ++k) dst[n][k] = *(const LAS bf16x8*)(lds + G_SB(b, h) + boff + n * 2048 + k * 1024); } while (0)
#define G_MMA(ai, bj, At, Bt) do { __builtin_amdgcn_s_setprio(1); _Pragma("unroll") for (int m = 0; m < 4; ++m) _Pragma("unroll") for (int n = 0; n < 2; ++n) _Pragma("unroll") for (int k = 0; k < 2; ++k) \
        acc[ai][bj][m][n] = __builtin_amdgcn_mfma_f32_16x16x32_bf16(Bt[n][k], At[m][k], acc[ai][bj][m][n], 0, 0, 0); __builtin_amdgcn_s_setprio(0); } while (0)
#define G_WAIT_V(n) asm volatile("s_waitcnt vmcnt(" #n ")" ::: "memory")
#define G_WAIT_L(n) asm volatile("s_waitcnt lgkmcnt(" #n ")" ::: "memory")
#define G_BAR __builtin_amdgcn_s_barrier()
#define G_SCHED __builtin_amdgcn_sched_barrier(0)
    auto ensure = [&](const GJob& j, int tneed) {
        if (avail >= tneed) return;
        if (wid == 0) {
            GAS const u32* ra = (GAS const u32*)(c.ws + OFF_RDYA) + j.ti * 64 + lane;
            GAS const u32* rb = (GAS const u32*)(c.ws + OFF_RDYB) + j.tj * 64 + lane;
            GAS u32* tmo = (GAS u32*)(c.ws + OFF_TMO);
            const u32 target = 4u * c.call;
            const u64 below = j.t0 ? ((1ull << j.t0) - 1ull) : 0ull;
            const u64 t0 = __builtin_amdgcn_s_memrealtime();
            int av, bd = 0;
            for (;;) {
                const u32 a = __hip_atomic_load(ra, __ATOMIC_RELAXED, __HIP_MEMORY_SCOPE_AGENT), b = __hip_atomic_load(rb, __ATOMIC_RELAXED, __HIP_MEMORY_SCOPE_AGENT);
                const u64 ok = __builtin_amdgcn_ballot_w64((int)(a - target) >= 0 && (int)(b - target) >= 0) | below;
                av = ok == ~0ull ? 64 : __builtin_ctzll(~ok);
                if (av >= tneed) break;
                if (rfl(__hip_atomic_load(tmo, __ATOMIC_RELAXED, __HIP_MEMORY_SCOPE_AGENT)) == c.call || __builtin_amdgcn_s_memrealtime() - t0 > DEADLINE) {
                    __hip_atomic_store(tmo, c.call, __ATOMIC_RELAXED, __HIP_MEMORY_SCOPE_AGENT); av = 64; bd = 1; break;
                }
                __builtin_amdgcn_s_sleep(4);
            }
            if (lane == 0) { ctl[0] = av; if (bd) ctl[1] = 1; }
            G_WAIT_L(0);
        }
        asm volatile("" ::: "memory"); G_BAR; asm volatile("" ::: "memory");
        avail = (int)rfl((u32)ctl[0]); bad = rfl((u32)ctl[1]) != 0;
    };
    GJob cur, nxt;
    cur.ti = 6 * (xcd & 1) + idx % 6; cur.tj = (xcd >> 1) + 4 * (idx / 6); cur.t0 = 0; cur.nt = 64;
    { const int col = xcd * 6 + (idx >> 2), third = col >> 4; nxt.ti = 12 + (idx & 3); nxt.tj = col & 15; nxt.t0 = 22 * third; nxt.nt = third == 2 ? 20 : 22; }
    f32x4 acc[2][2][4][2];
    bf16x8 At[4][2], B0[2][2], B1[2][2];
    constexpr size_t kstep = 32768, hstep = 16384;
    for (int ui = 0; ui < 2; ++ui) {
        const bool has_next = ui == 0;
#pragma unroll
        for (int a = 0; a < 2; ++a)
#pragma unroll
            for (int b = 0; b < 2; ++b)
#pragma unroll
                for (int m = 0; m < 4; ++m)
#pragma unroll
                    for (int n = 0; n < 2; ++n) acc[a][b][m][n] = (f32x4){0.f, 0.f, 0.f, 0.f};
        const char* cA = c.ws + OFF_A + (size_t)(cur.ti * 64 + cur.t0) * kstep;
        const char* cB = c.ws + OFF_B + (size_t)(cur.tj * 64 + cur.t0) * kstep;
        avail = 0;
        ensure(cur, cur.t0 + 2);
        G_STAGE(G_SB(0, 0), cB); G_STAGE(G_SA(0, 0), cA); G_STAGE(G_SB(0, 1), cB + hstep); G_STAGE(G_SA(0, 1), cA + hstep);
        if (wr == 1) G_BAR;
        G_WAIT_V(4); G_BAR;
        G_STAGE(G_SB(1, 0), cB + kstep); G_STAGE(G_SA(1, 0), cA + kstep); G_STAGE(G_SB(1, 1), cB + hstep + kstep);
        G_WAIT_V(6); G_BAR;
        if (ui == 0) STAMP(2);
        const int nt = cur.nt;
        for (int t = 0; t < nt; t += 2) {
            const bool last = (t == nt - 2);
            const char* a1 = cA + (size_t)(t + 1) * kstep;
            const char* a2 = cA + (size_t)(last ? t : t + 2) * kstep; const char* b2 = cB + (size_t)(last ? t : t + 2) * kstep;
            const char* a3 = a2 + kstep; const char* b3 = b2 + kstep;
            if (!last) ensure(cur, cur.t0 + t + 4);
            G_LDB(B0, 0, 0); G_SCHED; G_LDA(At, 0, 0); G_STAGE(G_SA(1, 1), a1 + hstep);
            G_WAIT_L(8); G_BAR; G_WAIT_L(0); G_MMA(0, 0, At, B0); G_BAR; G_SCHED;
            G_LDB(B1, 0, 1); G_STAGE(G_SB(0, 0), b2);
            G_BAR; G_WAIT_L(0); G_MMA(0, 1, At, B1); G_BAR;
            G_LDA(At, 0, 1); G_STAGE(G_SA(0, 0), a2);
            G_BAR; G_WAIT_L(0); G_MMA(1, 0, At, B0); G_BAR; G_SCHED;
            G_STAGE(G_SB(0, 1), b2 + hstep);
            G_WAIT_V(6); G_BAR; G_MMA(1, 1, At, B1); G_BAR;
            G_LDB(B0, 1, 0); G_SCHED; G_LDA(At, 1, 0); G_STAGE(G_SA(0, 1), a2 + hstep);
            G_WAIT_L(8); G_BAR; G_WAIT_L(0); G_MMA(0, 0, At, B0); G_BAR; G_SCHED;
            G_LDB(B1, 1, 1); G_STAGE(G_SB(1, 0), b3);
            G_BAR; G_WAIT_L(0); G_MMA(0, 1, At, B1); G_BAR;
            G_LDA(At, 1, 1); G_STAGE(G_SA(1, 0), a3);
            G_BAR; G_WAIT_L(0); G_MMA(1, 0, At, B0); G_BAR; G_SCHED;
            G_STAGE(G_SB(1, 1), b3 + hstep);
            G_WAIT_V(6); G_BAR; G_MMA(1, 1, At, B1); G_BAR;
        }
        if (wr == 0) G_BAR;
        G_WAIT_V(0); G_BAR;
        STAMP(3 + 2 * ui);
        {
            u32 meta[2], slot[2];
#pragma unroll
            for (int ai = 0; ai < 2; ++ai) {
                u32x4 d[8];
#pragma unroll
                for (int m = 0; m < 4; ++m)
#pragma unroll
                    for (int bj = 0; bj < 2; ++bj) {
                        const f32x4 v0 = acc[ai][bj][m][0], v1 = acc[ai][bj][m][1];
                        d[m * 2 + bj] = (u32x4){cvt_pk_bf16(v0[0], v0[1]), cvt_pk_bf16(v0[2], v0[3]), cvt_pk_bf16(v1[0], v1[1]), cvt_pk_bf16(v1[2], v1[3])};
                    }
                if (bad) fill_nan(d);
                const int jb = wid * 2 + ai;
                if (has_next) { slot[ai] = OFF_OUT + c.par * OUT_BYTES + (u32)((cur.ti * 16 + cur.tj) * 16 + jb) * SLOT; meta[ai] = pack_store(d, c.own, slot[ai], lane, c.kp); }
                else { slot[ai] = OFF_OWN + (u32)((((cur.ti - 12) * 16 + cur.tj) * 4 + (cur.t0 != 0) + (cur.t0 == 44)) * 16 + jb) * SLOT; store_raw(d, c.own, slot[ai], lane); meta[ai] = META_RAW; }
            }
            wait_vm0();
#pragma unroll
            for (int ai = 0; ai < 2; ++ai) {
                const int jb = wid * 2 + ai, r = c.rank;
                int dst; u32 foff;
                if (!has_next) { dst = r; foff = F_OWN + (u32)((((cur.ti - 12) * 16 + cur.tj) * 4 + (cur.t0 != 0) + (cur.t0 == 44)) * 16 + jb) * 8u; }
                else if (cur.ti < 4) {
                    const int pt = cur.ti * 16 + cur.tj;
                    if ((pt & 1) == 0) { dst = r ^ 2; foff = F_P + (u32)(pt * 16 + jb) * 8u; }
                    else { const int path = (pt >> 1) & 1; dst = r ^ (path ? 3 : 1); foff = F_R + (u32)((path * 16 + (pt >> 2)) * 16 + jb) * 8u; }
                } else {
                    const int p = (cur.ti - 4) >> 1, a = 2 * (p >> 1) + ((cur.ti - 4) & 1);
                    dst = r ^ d3_of_pair(p); foff = F_D3 + (u32)(((a * 16 + cur.tj) * 2 + (p & 1)) * 16 + jb) * 8u;
                }
                post_flag(P, c, dst, foff, meta[ai]);
            }
        }
        STAMP(4 + 2 * ui);
        cur = nxt;
    }
#undef G_SA
#undef G_SB
#undef G_STAGE
#undef G_LDA
#undef G_LDB
#undef G_MMA
#undef G_WAIT_V
#undef G_WAIT_L
#undef G_BAR
#undef G_SCHED
}

__global__ __launch_bounds__(NTH, 2) void gemm_rs_kernel(const Params P) {
    extern __shared__ __attribute__((aligned(16))) unsigned char shm[];
    LAS unsigned char* lds = (LAS unsigned char*)shm;
    const int wg = blockIdx.x, xcd = wg & 7, idx = wg >> 3, tid = threadIdx.x, wid = (int)rfl(tid >> 6);
    Ctx c;
    c.rank = P.rank & 7; c.lane = tid & 63;
    c.ws = (char*)P.ws[c.rank]; c.own = mk_rsrc(c.ws);
    c.kp = lds + wid * 16384;
    const bool gemm = idx < NGEMM / 8;
    const int gw = (xcd * (NGEMM / 8) + idx) * 8 + wid;
    const int cw = wid * NCOMM + (idx - NGEMM / 8) * 8 + xcd;
    int pj[5] = {-1, -1, -1, -1, -1}, pdelay = 0;
    u32x4 pin[8];
    const u64 tstart = __builtin_amdgcn_s_memrealtime();
    if (gemm) {
        const int grp = idx / 6, u = (xcd * 6 + idx % 6) * 8 + wid;
        int urg = 0, dummy;
        pj[0] = pack_pick(P, grp, u, 0, urg);
#pragma unroll
        for (int r = 1; r < 5; ++r) pj[r] = pack_pick(P, grp, u, r, dummy);
        pdelay = (urg * 4 - 16) * PACK_TAU / 16; pdelay = pdelay < 0 ? 0 : pdelay > PACK_DMAX ? PACK_DMAX : pdelay;
    }
    if (pj[0] >= 0 && pdelay == 0) pack_issue(P, c.lane, c.kp, pj[0], pin);
    LAS volatile u32* ctl = (LAS volatile u32*)(lds + LDS_CTL);
    if (tid == 0) {
        ctl[2] = __hip_atomic_fetch_add((GAS u32*)(c.ws + OFF_CALL + wg * 64), 1u, __ATOMIC_RELAXED, __HIP_MEMORY_SCOPE_AGENT) + 1u;
        ctl[0] = 0; ctl[1] = 0;
    }
    asm volatile("s_waitcnt lgkmcnt(0)" ::: "memory"); __builtin_amdgcn_s_barrier(); asm volatile("" ::: "memory");
    c.call = rfl(ctl[2]); c.par = c.call & 1u;
    STAMP(0);
    if (pdelay) {
        while (__builtin_amdgcn_s_memrealtime() - tstart < (u64)pdelay) __builtin_amdgcn_s_sleep(1);
        pack_issue(P, c.lane, c.kp, pj[0], pin);
    }
    pack_list(P, c, pj[0], pj[1], pj[2], pj[3], pj[4], pin);
    STAMP(1);
    if (gemm) {
        __syncthreads();
        gemm_role(P, c, lds, xcd, idx);
        if (gw < 1024) own_job(P, c, gw);
        STAMP(7);
    } else {
        const int ow = cw - O_FIRST;
        if (cw >= IN_WAVES && (ow < 0 || ow >= O_WAVES)) return;
        kp_clear(c.kp, c.lane);
#define GR_INMASK 63
        if (cw < IN_WAVES && ((GR_INMASK >> (cw % 6)) & 1)) in_list(P, c, cw % 6, cw / 6, IN_WPL);
        STAMP(3);
        if (ow >= 0 && ow < O_WAVES) {
            if (ow < 3 * C_WPL) copy_list(P, c, ow % 3, ow / 3, C_WPL);
            else reduce_list(P, c, (ow - 3 * C_WPL) & 1, (ow - 3 * C_WPL) >> 1, 2 * X_WPL);
        }
        STAMP(7);
    }
}
}

extern "C" void kernel_launch(int rank, int nranks, ncclComm_t comm, hipStream_t stream,
                              void* const* d_in, const int* in_sizes, int n_in,
                              void* d_out, int out_size,
                              const int* in_dtypes, int out_dtype,
                              void* const* d_out_all,
                              void* const* d_ws_all, size_t ws_bytes) {
    (void)comm; (void)in_sizes; (void)n_in; (void)out_size; (void)in_dtypes; (void)out_dtype; (void)d_out_all;
    if (nranks != gr::NR || rank < 0 || rank >= gr::NR || ws_bytes < gr::WS_NEED) { fprintf(stderr, "kernel_launch: unexpected configuration\n"); return; }
    static bool init[gr::NR] = {};
    if (!init[rank]) {
        if (hipFuncSetAttribute((const void*)gr::gemm_rs_kernel, hipFuncAttributeMaxDynamicSharedMemorySize, gr::LDS_BYTES) != hipSuccess) { fprintf(stderr, "hipFuncSetAttribute failed\n"); return; }
        init[rank] = true;
    }
    gr::Params P;
    P.x = d_in[0]; P.w = d_in[1]; P.out = d_out; P.rank = rank; P.pad = 0;
    for (int i = 0; i < gr::NR; ++i) P.ws[i] = d_ws_all[i];
    {
        unsigned short ord[1024]; int need[1024], first[1026] = {};
        for (int j = 0; j < 1024; ++j) { need[j] = gr::need_of(rank, j); ++first[need[j] + 1]; }
        for (int v = 0; v < 1025; ++v) first[v + 1] += first[v];
        for (int j = 0; j < 1024; ++j) ord[first[need[j]]++] = (unsigned short)(j | ((need[j] / 4 > 63 ? 63 : need[j] / 4) << 10));
        for (int i = 0; i < 512; ++i) P.order[i] = (unsigned)ord[2 * i] | ((unsigned)ord[2 * i + 1] << 16);
    }
    hipLaunchKernelGGL(gr::gemm_rs_kernel, dim3(gr::NWG), dim3(gr::NTH), gr::LDS_BYTES, stream, P);
    hipError_t e = hipGetLastError();
    if (e != hipSuccess) fprintf(stderr, "launch failed: %s\n", hipGetErrorString(e));
}
```

```cpp
#include <hip/hip_runtime.h>
#include <rccl/rccl.h>
#include <cstdio>

namespace gr {
typedef unsigned u32;
typedef unsigned long long u64;
typedef unsigned short u16;
typedef u32 u32x4 __attribute__((ext_vector_type(4)));
typedef u32 u32x2 __attribute__((ext_vector_type(2)));
typedef u16 u16x2 __attribute__((ext_vector_type(2)));
typedef short bf16x8 __attribute__((ext_vector_type(8)));
typedef short s16x4 __attribute__((ext_vector_type(4)));
typedef float f32x4 __attribute__((ext_vector_type(4)));
#define LAS __attribute__((address_space(3)))
#define GAS __attribute__((address_space(1)))
typedef __amdgpu_buffer_rsrc_t rsrc_t;

constexpr int NR = 8;
constexpr int NWG = 256, NTH = 512, NGEMM = 192, NCOMM = NWG - NGEMM;
constexpr int GEMM_WAVES = NGEMM * 8, COMM_WAVES = NCOMM * 8;
constexpr u32 SLOT = 8448;
constexpr u32 PAY_Q2 = 5120, PAY_X = 5632;
constexpr u32 XMAX = 128;
constexpr int KP_LANE = 80, KP_BYTES = 64 * KP_LANE;
constexpr u64 DEADLINE = 100000000ull;

constexpr u32 OFF_CALL = 0;
constexpr u32 OFF_TMO = 16384;
constexpr u32 OFF_RDYA = 20480;
constexpr u32 OFF_RDYB = 24576;
constexpr u32 OFF_TKT = 28672;
constexpr u32 OFF_FLG = 32768;
constexpr u32 F_IN = 0;
constexpr u32 F_D3 = F_IN + 2 * 16 * 64 * 4 * 8;
constexpr u32 F_P = F_D3 + 64 * 2 * 16 * 8;
constexpr u32 F_R = F_P + 64 * 16 * 8;
constexpr u32 F_OWN = F_R + 32 * 16 * 8;
constexpr u32 FLG_BYTES = F_OWN + 64 * 4 * 16 * 8;
constexpr u32 OFF_DBG = 512 * 1024;
constexpr u32 CTL_BYTES = 1024 * 1024;
static_assert(OFF_FLG + 2 * FLG_BYTES <= OFF_DBG, "control region");
constexpr u32 IN_BYTES = 4096 * SLOT;
constexpr u32 OUT_BYTES = 192 * 16 * SLOT;
constexpr u32 REL_BYTES = 32 * 16 * SLOT;
constexpr u32 OFF_IN = CTL_BYTES;
constexpr u32 OFF_OUT = OFF_IN + 2 * IN_BYTES;
constexpr u32 OFF_REL = OFF_OUT + 2 * OUT_BYTES;
constexpr u32 OFF_OWN = OFF_REL + 2 * REL_BYTES;
constexpr u32 OFF_A = OFF_OWN + 64 * 4 * 16 * SLOT;
constexpr u32 OFF_B = OFF_A + 16 * 64 * 32768;
constexpr u32 WS_NEED = OFF_B + 16 * 64 * 32768;
static_assert(WS_NEED <= 257u * 1024u * 1024u, "workspace");
static_assert(OFF_A % 256 == 0 && OFF_OUT % 128 == 0 && OFF_OWN % 128 == 0, "alignment");

constexpr u32 META_RAW = 1u << 16;
#define GR_IN_WPL 17
#define GR_C_WPL 15
#define GR_X_WPL 15
constexpr int IN_WPL = GR_IN_WPL, C_WPL = GR_C_WPL, X_WPL = GR_X_WPL;
constexpr int O_WAVES = 3 * C_WPL + 4 * X_WPL, IN_WAVES = 6 * IN_WPL;
#define GR_SEPOUT 1
#define GR_PACK_TAU 60
#define GR_PACK_DMAX 500
#define GR_PACK_M 0xCCEF
#define GR_LOC0 64
#define GR_LOC1 14
#define GR_CH 78
#define GR_D1 0
#define GR_D2 0
#define GR_D3 0
#define GR_DOWN 10000
#define GR_TPERM 0
#define GR_SLEEP 2
#define GR_GSLEEP 4
#define GR_TMOMASK 15
#define GR_TOPW 1
#define GR_XGRP 0
constexpr int PACK_TAU = GR_PACK_TAU, PACK_DMAX = GR_PACK_DMAX;
constexpr u32 PACK_M = GR_PACK_M;
constexpr int O_FIRST = GR_SEPOUT ? IN_WAVES : 0;
static_assert(O_FIRST + O_WAVES <= COMM_WAVES && IN_WAVES <= COMM_WAVES, "waves");
#define GR_GAUX 16
constexpr int AUX_SYS = 17, AUX_DEV = GR_GAUX;
constexpr int LDS_STAGE = 131072, LDS_CTL = LDS_STAGE, LDS_BYTES = LDS_STAGE + 256;

struct Params {
    const void* x;
    const void* w;
    void* out;
    void* ws[NR];
    int rank;
    int pad;
    u32 inord[16];
    u32 order[512];
};

__device__ __forceinline__ u32 rfl(u32 v) { return (u32)__builtin_amdgcn_readfirstlane((int)v); }
__device__ __forceinline__ u32 pk_add_s(u32 a, u32 s) { u32 r; asm("v_pk_add_u16 %0, %1, %2" : "=v"(r) : "v"(a), "s"(s)); return r; }
__device__ __forceinline__ u32 pk_sub_s(u32 a, u32 s) { u32 r; asm("v_pk_sub_u16 %0, %1, %2" : "=v"(r) : "v"(a), "s"(s)); return r; }
__device__ __forceinline__ u32 bfi(u32 m, u32 a, u32 b) { u32 r; asm("v_bfi_b32 %0, %1, %2, %3" : "=v"(r) : "s"(m), "v"(a), "v"(b)); return r; }
__device__ __forceinline__ u32 rotl(u32 x, int r) { return __builtin_amdgcn_alignbit(x, x, (u32)(32 - r)); }
__device__ __forceinline__ u32 rotr(u32 x, int r) { return __builtin_amdgcn_alignbit(x, x, (u32)r); }
__device__ __forceinline__ u32 ffbl(u32 x) { u32 r; asm("v_ffbl_b32 %0, %1" : "=v"(r) : "v"(x)); return r; }
__device__ __forceinline__ u32 wave_scan(u32 x) {
    x += (u32)__builtin_amdgcn_update_dpp(0, (int)x, 0x111, 0xF, 0xF, false);
    x += (u32)__builtin_amdgcn_update_dpp(0, (int)x, 0x112, 0xF, 0xF, false);
    x += (u32)__builtin_amdgcn_update_dpp(0, (int)x, 0x114, 0xF, 0xF, false);
    x += (u32)__builtin_amdgcn_update_dpp(0, (int)x, 0x118, 0xF, 0xF, false);
    x += (u32)__builtin_amdgcn_update_dpp(0, (int)x, 0x142, 0xA, 0xF, false);
    x += (u32)__builtin_amdgcn_update_dpp(0, (int)x, 0x143, 0xC, 0xF, false);
    return x;
}
__device__ __forceinline__ u32 cvt_pk_bf16(float lo, float hi) { u32 r; asm volatile("v_cvt_pk_bf16_f32 %0, %1, %2" : "=v"(r) : "v"(lo), "v"(hi)); return r; }
__device__ __forceinline__ rsrc_t mk_rsrc(void* p) { return __builtin_amdgcn_make_buffer_rsrc(p, (short)0, (int)WS_NEED, 0x00020000); }
__device__ __forceinline__ void wait_vm0() { asm volatile("s_waitcnt vmcnt(0)" ::: "memory"); }
__device__ __forceinline__ void wait_vm0_known() { __builtin_amdgcn_s_waitcnt(0x0F70); asm volatile("" ::: "memory"); }

struct Ctx {
    char* ws;
    rsrc_t own;
    u32 call, par;
    int rank, lane;
    LAS unsigned char* kp;
};
#define JLOG(list, job, sub, k) do {} while (0)
#define DT0() do {} while (0)
#define DT(i) do {} while (0)
#define DDEP(d) do {} while (0)

__device__ __forceinline__ u64 flag_load(const Ctx& c, u32 foff) {
    return __hip_atomic_load((GAS const u64*)(c.ws + OFF_FLG + c.par * FLG_BYTES + foff), __ATOMIC_RELAXED, __HIP_MEMORY_SCOPE_SYSTEM);
}
__device__ __forceinline__ bool flag_test(const Ctx& c, u64 v, u32& meta) {
    meta = rfl((u32)v);
    return (int)(rfl((u32)(v >> 32)) - c.call) >= 0;
}
__device__ __forceinline__ bool wait_flag(const Ctx& c, u32 foff, u32& meta) {
    GAS u32* tmo = (GAS u32*)(c.ws + OFF_TMO);
    if (flag_test(c, flag_load(c, foff), meta)) return true;
    const u64 t0 = __builtin_amdgcn_s_memrealtime();
    for (u32 it = 1;; ++it) {
        __builtin_amdgcn_s_sleep(GR_SLEEP);
        if (flag_test(c, flag_load(c, foff), meta)) return true;
        if ((it & GR_TMOMASK) == 0 && (rfl(__hip_atomic_load(tmo, __ATOMIC_RELAXED, __HIP_MEMORY_SCOPE_AGENT)) == c.call || __builtin_amdgcn_s_memrealtime() - t0 > DEADLINE)) {
            __hip_atomic_store(tmo, c.call, __ATOMIC_RELAXED, __HIP_MEMORY_SCOPE_AGENT);
            meta = 0; return false;
        }
    }
}
__device__ __forceinline__ void post_flag(const Params& P, const Ctx& c, int dst, u32 foff, u32 meta) {
    if ((unsigned)dst >= (unsigned)NR) return;
    GAS u64* f = (GAS u64*)((char*)P.ws[dst] + OFF_FLG + c.par * FLG_BYTES + foff);
    if (c.lane == 0) __hip_atomic_store(f, ((u64)c.call << 32) | meta, __ATOMIC_RELAXED, __HIP_MEMORY_SCOPE_SYSTEM);
}

__device__ __forceinline__ void store_raw(const u32x4 (&d)[8], rsrc_t rs, u32 slot, int lane) {
#pragma unroll
    for (int v = 0; v < 8; ++v) __builtin_amdgcn_raw_buffer_store_b128(d[v], rs, (int)(slot + v * 1024 + lane * 16), 0, AUX_SYS);
}
__device__ __forceinline__ u32 kz_of(u32 L) { return ((0x8000u - L) & 0x7FFFu) >> 10; }
__device__ __forceinline__ u32 pack_store(const u32x4 (&d)[8], rsrc_t rs, u32 slot, int lane, LAS unsigned char* stage) {
    float m0 = 0.f, m1 = 0.f;
#pragma unroll
    for (int f = 0; f < 8; ++f) {
        asm("v_max3_f32 %0, |%1|, |%2|, %0" : "+v"(m0) : "v"(d[f][0]), "v"(d[f][1]));
        asm("v_max3_f32 %0, |%1|, |%2|, %0" : "+v"(m1) : "v"(d[f][2]), "v"(d[f][3]));
    }
    const u32 b0 = __builtin_bit_cast(u32, m0), b1 = __builtin_bit_cast(u32, m1);
    const u32 sum = (u32)__builtin_amdgcn_readlane((int)wave_scan(((b0 > b1 ? b0 : b1) >> 16) & 0x7FFFu), 63);
    const u32 top = ((sum >> 6) + 32u) >> 7, lo = top > 7u ? top - 7u : 0u, L = lo << 7, LL = L | (L << 16);
    u32 lb[16], hb[16], M[2] = {0u, 0u};
#pragma unroll
    for (int j = 0; j < 16; ++j) {
        const u32 w0 = pk_sub_s(d[j >> 1][(j & 1) * 2], LL), w1 = pk_sub_s(d[j >> 1][(j & 1) * 2 + 1], LL);
        lb[j] = __builtin_amdgcn_perm(w1, w0, 0x06040200u); hb[j] = __builtin_amdgcn_perm(w1, w0, 0x07050301u);
        M[j >> 3] = bfi(0x80808080u, (hb[j] & 0x7C7C7C7Cu) + 0x7C7C7C7Cu, M[j >> 3] >> 1);
    }
    const u32 c0 = (u32)__builtin_popcount(M[0]), c1 = (u32)__builtin_popcount(M[1]), incl = wave_scan(c0 + c1);
    const u32 cnt = (u32)__builtin_amdgcn_readlane((int)incl, 63);
    if (cnt > XMAX) { store_raw(d, rs, slot, lane); return META_RAW; }
    u32 Q[6];
#pragma unroll
    for (int g = 0; g < 2; ++g) {
        const u32* h = hb + 8 * g;
        Q[3 * g + 0] = bfi(0x60606060u, rotl(h[6], 6), bfi(0x1C1C1C1Cu, rotl(h[1], 3), h[0]));
        Q[3 * g + 1] = bfi(0x40404040u, rotl(h[7], 5), bfi(0x20202020u, rotl(h[6], 4), bfi(0x1C1C1C1Cu, rotl(h[3], 3), h[2])));
        Q[3 * g + 2] = bfi(0x60606060u, rotl(h[7], 6), bfi(0x1C1C1C1Cu, rotl(h[5], 3), h[4]));
    }
#pragma unroll
    for (int v = 0; v < 4; ++v)
        __builtin_amdgcn_raw_buffer_store_b128((u32x4){lb[4 * v], lb[4 * v + 1], lb[4 * v + 2], lb[4 * v + 3]}, rs, (int)(slot + v * 1024 + lane * 16), 0, AUX_SYS);
    __builtin_amdgcn_raw_buffer_store_b128((u32x4){Q[0], Q[1], Q[2], Q[3]}, rs, (int)(slot + 4096 + lane * 16), 0, AUX_SYS);
    __builtin_amdgcn_raw_buffer_store_b64((u32x2){Q[4], Q[5]}, rs, (int)(slot + PAY_Q2 + lane * 8), 0, AUX_SYS);
    if (cnt) {
        LAS unsigned char* mine = stage + lane * KP_LANE;
        LAS u16* list = (LAS u16*)(stage + KP_BYTES);
#pragma unroll
        for (int i = 0; i < 4; ++i) *(LAS u32x4*)(mine + 16 * i) = (u32x4){hb[4 * i], hb[4 * i + 1], hb[4 * i + 2], hb[4 * i + 3]};
        const u32 kz = kz_of(L);
        u32 off[2] = {incl - c0 - c1, incl - c1};
        bool bad = false;
        do {
#pragma unroll
            for (int g = 0; g < 2; ++g) {
                const bool has = M[g] != 0u;
                const u32 p = ffbl(M[g]), pos = (u32)(g * 32) + ((p & 7u) << 2) + ((p >> 3) & 3u);
                M[g] &= M[g] - 1u;
                const u32 k5 = ((u32)mine[pos] >> 2) & 31u;
                const bool z = k5 == kz;
                bad |= has && !z && ((k5 + 7u) & 31u) >= 15u;
                list[has ? off[g] : XMAX] = (u16)((((u32)lane * 64u + pos) << 4) | (z ? 8u : k5 & 15u));
                off[g] += has ? 1u : 0u;
            }
        } while (__builtin_amdgcn_ballot_w64((M[0] | M[1]) != 0u));
        if (__builtin_amdgcn_ballot_w64(bad)) { wait_vm0_known(); store_raw(d, rs, slot, lane); return META_RAW; }
        const u32 e = ((LAS const u32*)list)[lane];
        if (2u * (u32)lane < cnt) __builtin_amdgcn_raw_buffer_store_b32(e, rs, (int)(slot + PAY_X + lane * 4), 0, AUX_SYS);
    }
    return lo | (cnt << 8);
}
struct Pull { u32x4 V[5]; u32x2 q; u32 h; };
__device__ __forceinline__ void pull_issue(Pull& p, rsrc_t rs, u32 slot, u32 meta, int lane) {
#pragma unroll
    for (int v = 0; v < 5; ++v) p.V[v] = __builtin_amdgcn_raw_buffer_load_b128(rs, (int)(slot + v * 1024 + lane * 16), 0, AUX_SYS);
    p.q = __builtin_amdgcn_raw_buffer_load_b64(rs, (int)(slot + PAY_Q2 + lane * 8), 0, AUX_SYS);
    const u32 hoff = 2u * (u32)lane < ((meta >> 8) & 0xFFu) ? slot + PAY_X + (u32)lane * 4u : 0x80000000u;
    p.h = __builtin_amdgcn_raw_buffer_load_b32(rs, (int)hoff, 0, AUX_SYS);
}
__device__ __forceinline__ void kp_clear(LAS unsigned char* kp, int lane) {
#pragma unroll
    for (int i = 0; i < 5; ++i) *(LAS u32x4*)(kp + lane * KP_LANE + 16 * i) = (u32x4){0u, 0u, 0u, 0u};
}
__device__ __forceinline__ void unpack(u32x4 (&d)[8], const Pull& p, rsrc_t rs, u32 slot, u32 meta, int lane, LAS unsigned char* kp) {
    if (meta & META_RAW) {
#pragma unroll
        for (int v = 0; v < 5; ++v) d[v] = p.V[v];
#pragma unroll
        for (int v = 5; v < 8; ++v) d[v] = __builtin_amdgcn_raw_buffer_load_b128(rs, (int)(slot + v * 1024 + lane * 16), 0, AUX_SYS);
        wait_vm0_known();
        return;
    }
    const u32 cnt = (meta >> 8) & 0xFFu;
    const u32 L = (meta & 0xFFu) << 7, LL = L | (L << 16), kz = kz_of(L);
    u32 at[2];
#pragma unroll
    for (int i = 0; i < 2; ++i) {
        const u32 e = i ? p.h >> 16 : p.h & 0xFFFFu, pos = e >> 4, k4 = e & 15u;
        at[i] = 2u * (u32)lane + (u32)i < cnt ? pos + (pos >> 6) * 16u : (u32)KP_BYTES;
        kp[at[i]] = (unsigned char)((k4 == 8u ? kz : k4 | ((k4 & 8u) << 1)) << 2);
    }
    u32x4 K[4];
#pragma unroll
    for (int i = 0; i < 4; ++i) K[i] = *(LAS const u32x4*)(kp + lane * KP_LANE + 16 * i);
    kp[at[0]] = 0; kp[at[1]] = 0;
    const u32 Q[6] = {p.V[4][0], p.V[4][1], p.V[4][2], p.V[4][3], p.q[0], p.q[1]};
#pragma unroll
    for (int g = 0; g < 2; ++g) {
        const u32 q0 = Q[3 * g], q1 = Q[3 * g + 1], q2 = Q[3 * g + 2];
        const u32x4 k0 = K[2 * g], k1 = K[2 * g + 1];
        u32 h[8];
        h[0] = (q0 & 0x83838383u) | k0[0]; h[1] = (rotr(q0, 3) & 0x83838383u) | k0[1];
        h[2] = (q1 & 0x83838383u) | k0[2]; h[3] = (rotr(q1, 3) & 0x83838383u) | k0[3];
        h[4] = (q2 & 0x83838383u) | k1[0]; h[5] = (rotr(q2, 3) & 0x83838383u) | k1[1];
        h[6] = (rotr(q0, 6) & 0x81818181u) | ((rotr(q1, 4) & 0x02020202u) | k1[2]);
        h[7] = (rotr(q2, 6) & 0x81818181u) | ((rotr(q1, 5) & 0x02020202u) | k1[3]);
#pragma unroll
        for (int j = 0; j < 8; ++j) {
            const u32 B = p.V[2 * g + (j >> 2)][j & 3];
            d[4 * g + (j >> 1)][(j & 1) * 2] = pk_add_s(__builtin_amdgcn_perm(h[j], B, 0x05010400u), LL);
            d[4 * g + (j >> 1)][(j & 1) * 2 + 1] = pk_add_s(__builtin_amdgcn_perm(h[j], B, 0x07030602u), LL);
        }
    }
}
__device__ __forceinline__ void fill_nan(u32x4 (&d)[8]) {
#pragma unroll
    for (int v = 0; v < 8; ++v) d[v] = (u32x4){0x7FC07FC0u, 0x7FC07FC0u, 0x7FC07FC0u, 0x7FC07FC0u};
}
__device__ __forceinline__ void copy_store(const Ctx& c, const Pull& p, rsrc_t rs, u32 slot, u32 dst, u32 meta) {
    const int lane = c.lane;
    if (meta & META_RAW) {
        u32x4 a[3];
#pragma unroll
        for (int v = 0; v < 3; ++v) a[v] = __builtin_amdgcn_raw_buffer_load_b128(rs, (int)(slot + (5 + v) * 1024 + lane * 16), 0, AUX_SYS);
        wait_vm0_known();
#pragma unroll
        for (int v = 0; v < 3; ++v) __builtin_amdgcn_raw_buffer_store_b128(a[v], c.own, (int)(dst + (5 + v) * 1024 + lane * 16), 0, AUX_SYS);
    } else {
        __builtin_amdgcn_raw_buffer_store_b64(p.q, c.own, (int)(dst + PAY_Q2 + lane * 8), 0, AUX_SYS);
        if (2u * (u32)lane < ((meta >> 8) & 0xFFu)) __builtin_amdgcn_raw_buffer_store_b32(p.h, c.own, (int)(dst + PAY_X + lane * 4), 0, AUX_SYS);
    }
#pragma unroll
    for (int v = 0; v < 5; ++v) __builtin_amdgcn_raw_buffer_store_b128(p.V[v], c.own, (int)(dst + v * 1024 + lane * 16), 0, AUX_SYS);
}

__device__ __forceinline__ u32 ticket_issue(const Ctx& c, u32 off) {
    u32 r; u64 sv; const u32 one = 1u;
    asm volatile("s_mov_b64 %1, exec\n\ts_mov_b64 exec, 1\n\tglobal_atomic_add %0, %2, %3, %4 sc0\n\ts_mov_b64 exec, %1"
                 : "=&v"(r), "=&s"(sv) : "v"(off), "v"(one), "s"(c.ws) : "memory");
    return r;
}
__device__ __forceinline__ u32 ticket_get(u32 r) { asm volatile("" : "+v"(r) :: "memory"); return rfl(r); }

struct Step { u32 foff, slot, meta; int sel, p0, p1, p2, p3; };
template <int SPT, class Gen, class Act, class Pub>
__device__ __forceinline__ void pull_list(Ctx& c, int list, u32 first, u32 nwaves, u32 njobs, rsrc_t rs0, rsrc_t rs1, Gen gen, Act act, Pub pub) {
    DT0();
    const u32 tkoff = OFF_TKT + (u32)list * 128u, tkbase = nwaves - (c.call - 1u) * njobs;
    u32 jc = first, jn = 0, tk = 0;
    if (SPT == 1) tk = ticket_issue(c, tkoff);
    Step cur = gen(jc, 0), prev = cur;
    bool ok = wait_flag(c, cur.foff, cur.meta), have_prev = false, more = true;
    if (SPT == 1) { wait_vm0_known(); jn = ticket_get(tk) + tkbase; }
    DT(1);
    Pull pa, pb;
    JLOG(list, jc, 0, 0);
    pull_issue(pa, cur.sel ? rs1 : rs0, cur.slot, cur.meta, c.lane);
    auto half = [&](const Pull& p, Pull& pn, int sub) {
        const bool same = SPT == 2 && sub == 0;
        more = same || jn < njobs;
        Step nxt = gen(same || !more ? jc : jn, same ? 1 : 0);
        const bool draw = SPT == 1 ? more : same;
        if (draw) tk = ticket_issue(c, tkoff);
        const u64 fv = flag_load(c, nxt.foff);
        wait_vm0_known();
        DT(0);
        JLOG(list, jc, sub, 1);
        const u32 jd = draw ? ticket_get(tk) + tkbase : njobs;
        if (have_prev) pub(prev);
        const bool nready = more && flag_test(c, fv, nxt.meta);
        if (nready) { JLOG(list, same ? jc : jn, same ? 1 : 0, 0); pull_issue(pn, nxt.sel ? rs1 : rs0, nxt.slot, nxt.meta, c.lane); }
        DT(3);
        act(cur, p, cur.sel ? rs1 : rs0, ok);
        prev = cur; have_prev = true; ok = true;
        DT(2);
        if (more && !nready) {
            wait_vm0_known(); pub(prev); have_prev = false;
            ok = wait_flag(c, nxt.foff, nxt.meta);
            JLOG(list, same ? jc : jn, same ? 1 : 0, 0);
            pull_issue(pn, nxt.sel ? rs1 : rs0, nxt.slot, nxt.meta, c.lane);
            DT(1);
        }
        cur = nxt;
        if (SPT == 1) { jc = jn; jn = jd; }
        else if (same) jn = jd; else jc = jn;
    };
    for (;;) {
        half(pa, pb, 0); if (!more) break;
        half(pb, pa, 1); if (!more) break;
    }
    wait_vm0_known();
    if (have_prev) pub(prev);
}

__host__ __device__ __forceinline__ int cq_of(int q, int r) {
    return ((r >> 1) ^ ((q & 1) ? (r >> 2) : 0) ^ ((q & 2) ? r : 0)) & 1;
}
__host__ __device__ __forceinline__ int link_d(int link) { return (0x765431 >> (4 * link)) & 7; }
__host__ __device__ __forceinline__ int link_q(int link, int i) { return ((i ? 0x233231 : 0x110020) >> (4 * link)) & 3; }
__host__ __device__ __forceinline__ int link_of_d(int d) { return d == 1 ? 0 : d - 2; }
__host__ __device__ __forceinline__ int t_chunk(int kc) { return GR_TPERM && kc < 2 ? 8 + 10 * kc : 16 + 6 * kc; }
__host__ __device__ __forceinline__ int t_own(int kq) { return GR_TPERM && kq >= 2 ? 10 * kq - 6 : 4 * kq; }
__host__ __device__ __forceinline__ int chunk_of(int d, int q, int kq) { return 2 * kq + (link_q(link_of_d(d), 0) == q ? 0 : 1); }
__host__ __device__ __forceinline__ int t_of(int d, int q, int kq) { return d == 0 ? t_own(kq) + q : t_chunk(chunk_of(d, q, kq)) + link_of_d(d); }
__host__ __device__ __forceinline__ int d3_pair(int d) { return (d & 1) * 2 + ((d >> 1) & 1); }
__host__ __device__ __forceinline__ int d3_of_pair(int p) { return 4 + (p >> 1) + (p & 1) * 2; }
__host__ __device__ __forceinline__ int rowtile_of(int rc, int o, int a) {
    const int d = rc ^ o;
    return d == 0 ? 12 + a : d == 2 ? a : 4 + 2 * d3_pair(d) + (a & 1);
}
__device__ __forceinline__ void block_of_rowtile(int rc, int i, int& o, int& a) {
    if (i >= 12) { o = rc; a = i - 12; }
    else if (i < 4) { o = rc ^ 2; a = i; }
    else { const int p = (i - 4) >> 1; o = rc ^ d3_of_pair(p); a = 2 * (p >> 1) + ((i - 4) & 1); }
}
__device__ __forceinline__ u32 raw_frag_off(int type, int sub, int f) {
    return type == 0 ? (u32)(sub * 8192 + f * 1024) : (u32)((f >> 2) * 16384 + sub * 4096 + (f & 3) * 1024);
}
__device__ __forceinline__ void store_operand(const Ctx& c, const u32x4 (&d)[8], int type, int tile, int t, int sub) {
    const u32 base = (type == 0 ? OFF_A : OFF_B) + (u32)(tile * 64 + t) * 32768u + (u32)c.lane * 16u;
#pragma unroll
    for (int f = 0; f < 8; ++f) __builtin_amdgcn_raw_buffer_store_b128(d[f], c.own, (int)(base + raw_frag_off(type, sub, f)), 0, AUX_SYS);
}
__device__ __forceinline__ void count_operand(const Ctx& c, int type, int tile, int t) {
    GAS u32* p = (GAS u32*)(c.ws + (type == 0 ? OFF_RDYA : OFF_RDYB)) + tile * 64 + t;
    if (c.lane == 0) __hip_atomic_fetch_add(p, 1u, __ATOMIC_RELAXED, __HIP_MEMORY_SCOPE_AGENT);
}

__host__ __device__ __forceinline__ void consumers_of(int s, int pj, int (&rc)[2], int (&tl)[2], int (&tt)[2]) {
    const int kq = pj >> 10, q = pj & 3, type = (pj >> 2) & 1, g = (pj & 1023) >> 3;
    const int cs = cq_of(q, s);
#pragma unroll
    for (int i = 0; i < 2; ++i) {
        int e1, gam;
        if (type == 0) { const int o = g >> 4, h = (g >> 3) & 1; gam = i; e1 = h == 0 ? (o & 1) : ((o ^ (o >> 2) ^ gam) & 1); }
        else { gam = g >> 6; e1 = i; }
        const int e2 = (cs ^ ((q & 2) ? e1 : 0) ^ ((q & 1) ? gam : 0)) & 1;
        rc[i] = e1 | (e2 << 1) | (gam << 2);
        tl[i] = type == 0 ? rowtile_of(rc[i], g >> 4, (g >> 2) & 3) : ((g >> 2) & 15);
        tt[i] = t_of(rc[i] ^ s, q, kq);
    }
}
__host__ inline int need_of(int s, int pj, const int (&blockpos)[64]) {
    int rc[2], tl[2], tt[2], need = 1023;
    consumers_of(s, pj, rc, tl, tt);
    const int kq = pj >> 10, q = pj & 3, type = (pj >> 2) & 1, sub = (pj >> 3) & 3;
    for (int i = 0; i < 2; ++i) {
        const int kind = type == 0 ? (tl[i] >= 12 ? 7 : tl[i] >> 2) : 3 + (tl[i] >> 2);
        const int pos = rc[i] != s ? blockpos[chunk_of(rc[i] ^ s, q, kq) * 8 + kind] * 16 + (tl[i] & 3) * 4 + sub : kind == 7 ? 896 : GR_LOC0 + GR_LOC1 * tt[i];
        if (pos < need) need = pos;
    }
    return need;
}
__device__ __forceinline__ void pack_issue(const Params& P, int lane, LAS unsigned char* tile, int pj, u32x4 (&d)[8]) {
    const int kq = pj >> 10, q = pj & 3, type = (pj >> 2) & 1, g = (pj & 1023) >> 3, kt = 4 * q + kq;
    if (type == 0) {
        GAS const char* src = (GAS const char*)P.x + (size_t)(64 * g + (lane & 15)) * 2048 + (64 * kt + 8 * (lane >> 4)) * 2;
#pragma unroll
        for (int f = 0; f < 8; ++f) d[f] = *(GAS const u32x4*)(src + (f >> 1) * 16 * 2048 + (f & 1) * 64);
    } else {
        GAS const char* src = (GAS const char*)P.w + (size_t)(64 * kt + (lane >> 3)) * 16384 + (64 * g) * 2 + (lane & 7) * 16;
#pragma unroll
        for (int v = 0; v < 8; ++v)
            __builtin_amdgcn_global_load_lds((GAS const u32*)(src + (size_t)v * 8 * 16384), (LAS u32*)(tile + v * 1024), 16, 0, 0);
    }
}
__device__ __forceinline__ void pack_job(const Params& P, const Ctx& c, LAS unsigned char* tile, LAS unsigned char* tile2, int pj, int next, u32x4 (&in)[8]) {
    const int kq = pj >> 10, q = pj & 3, type = (pj >> 2) & 1, g = (pj & 1023) >> 3, kt = 4 * q + kq, lane = c.lane, s = c.rank;
    u32x4 d[8];
    if (type == 0) {
#pragma unroll
        for (int f = 0; f < 8; ++f) d[f] = in[f];
    } else {
        wait_vm0();
        const int lg = lane >> 4, lq = (lane >> 2) & 3, lp = lane & 3;
        LAS unsigned char* b = tile + (8 * lg + lq) * 128 + lp * 16;
#pragma unroll
        for (int f = 0; f < 8; ++f) {
            const int o = (f & 1) * 32 * 128 + (f >> 2) * 64 + ((f >> 1) & 1) * 8;
            const s16x4 lo4 = __builtin_amdgcn_ds_read_tr16_b64_v4i16((LAS s16x4*)(b + o));
            const s16x4 hi4 = __builtin_amdgcn_ds_read_tr16_b64_v4i16((LAS s16x4*)(b + o + 4 * 128));
            const u32x2 l2 = __builtin_bit_cast(u32x2, lo4), h2 = __builtin_bit_cast(u32x2, hi4);
            d[f] = (u32x4){l2[0], l2[1], h2[0], h2[1]};
        }
        asm volatile("s_waitcnt lgkmcnt(0)" ::: "memory");
    }
    if (next >= 0) pack_issue(P, lane, tile2, next, in);
    int rc[2], tl[2], tt[2];
    consumers_of(s, pj, rc, tl, tt);
    const int sub = g & 3;
    const u32 meta = pack_store(d, c.own, OFF_IN + c.par * IN_BYTES + (u32)(type * 2048 + kt * 128 + g) * SLOT, lane, tile);
    if (rc[0] == s) store_operand(c, d, type, tl[0], tt[0], sub);
    if (rc[1] == s) store_operand(c, d, type, tl[1], tt[1], sub);
    wait_vm0();
#pragma unroll
    for (int i = 0; i < 2; ++i) {
        if (rc[i] == s) count_operand(c, type, tl[i], tt[i]);
        else post_flag(P, c, rc[i], F_IN + (u32)(((type * 16 + tl[i]) * 64 + tt[i]) * 4 + sub) * 8u, meta);
    }
}
__device__ __forceinline__ int pack_pick(const Params& P, int grp, int u, int r, int& urg) {
    int off = 0;
#pragma unroll
    for (int i = 0; i < 6; ++i) if (i < r) off += 384 * __builtin_popcount((PACK_M >> (4 * i)) & 15u);
    const u32 m = (PACK_M >> (4 * r)) & 15u;
    int i = off + 384 * __builtin_popcount(m & ((1u << grp) - 1u)) + u;
    if (GR_TOPW > 0 && r == 0) {
        const int w = u / 48, g = u % 48;
        i = w < GR_TOPW ? w * 192 + g * 4 + grp : 192 * GR_TOPW + grp * 48 * (8 - GR_TOPW) + (w - GR_TOPW) * 48 + g;
    }
    if (!((m >> grp) & 1u) || i >= 4096) return -1;
    const u32 e = (P.order[(i & 1023) >> 1] >> (16 * (i & 1))) & 0xFFFFu;
    urg = i < 1024 ? (int)(e >> 10) : 63;
    return (i & ~1023) | (int)(e & 1023u);
}
__device__ __forceinline__ void pack_list(const Params& P, const Ctx& c, int j0, int j1, int j2, int j3, int j4, u32x4 (&in)[8]) {
    for (u32 buf = 0; j0 >= 0; buf ^= 8192u) { pack_job(P, c, c.kp + buf, c.kp + (buf ^ 8192u), j0, j1, in); j0 = j1; j1 = j2; j2 = j3; j3 = j4; j4 = -1; }
}

__device__ __forceinline__ void in_list(const Params& P, Ctx& c, int link, int widx, int wpl) {
    const int s = c.rank ^ link_d(link);
    const u32 tbl = P.inord[(c.lane >> 2) & 15] >> (8 * (c.lane & 3));
    auto gen = [&](u32 job, int) {
        const int j = (int)job, blk = __builtin_amdgcn_readlane((int)tbl, (j >> 4) & 63), kc = (blk >> 3) & 7, kind = blk & 7;
        const int type = kind >= 3 && kind < 7, tile = (kind == 7 ? 12 : type ? 4 * (kind - 3) : 4 * kind) + ((j >> 2) & 3), sub = j & 3;
        const int t = t_chunk(kc) + link, kt = 4 * link_q(link, kc & 1) + (kc >> 1);
        int g;
        if (type == 0) { int o, a; block_of_rowtile(c.rank, tile, o, a); g = o * 16 + a * 4 + sub; } else g = (c.rank >> 2) * 64 + tile * 4 + sub;
        Step st; st.foff = F_IN + (u32)(((type * 16 + tile) * 64 + t) * 4 + sub) * 8u;
        st.slot = OFF_IN + c.par * IN_BYTES + (u32)(type * 2048 + kt * 128 + g) * SLOT;
        st.sel = 0; st.meta = 0; st.p0 = type; st.p1 = tile; st.p2 = t; st.p3 = sub;
        return st;
    };
    auto act = [&](Step& st, const Pull& p, rsrc_t rs, bool ok) {
        u32x4 d[8];
        if (ok) unpack(d, p, rs, st.slot, st.meta, c.lane, c.kp); else fill_nan(d);
        store_operand(c, d, st.p0, st.p1, st.p2, st.p3);
    };
    auto pub = [&](const Step& st) { count_operand(c, st.p0, st.p1, st.p2); };
    const rsrc_t rs = mk_rsrc(P.ws[s & 7]);
    pull_list<1>(c, link, (u32)widx, (u32)wpl, 1024u, rs, rs, gen, act, pub);
}

__device__ __forceinline__ void acc_add(float (&acc)[64], const u32x4 (&d)[8]) {
#pragma unroll
    for (int q = 0; q < 32; ++q) { const u32 v = d[q >> 2][q & 3]; acc[2 * q] += __builtin_bit_cast(float, v << 16); acc[2 * q + 1] += __builtin_bit_cast(float, v & 0xFFFF0000u); }
}
__device__ __forceinline__ void write_out(const Params& P, const Ctx& c, const float (&acc)[64], int a, int cj, int jb) {
    const int w = jb >> 1, ai = jb & 1, lane = c.lane;
    GAS char* o = (GAS char*)P.out + (size_t)(a * 256 + ai * 128 + (w >> 2) * 64 + (lane & 15)) * 16384 + (cj * 256 + (w & 3) * 64 + 8 * (lane >> 4)) * 2;
#pragma unroll
    for (int f = 0; f < 8; ++f) {
        u32x4 v;
#pragma unroll
        for (int e = 0; e < 4; ++e) v[e] = cvt_pk_bf16(acc[f * 8 + 2 * e], acc[f * 8 + 2 * e + 1]);
        *(GAS u32x4*)(o + (size_t)(f >> 1) * 16 * 16384 + (f & 1) * 64) = v;
    }
}
__device__ __forceinline__ void copy_list(const Params& P, Ctx& c, int grp, int widx, int wpl) {
    const int r = c.rank;
    auto gen = [&](u32 job, int) {
        const int j = (int)job, jb = j & 15;
        Step st; st.meta = 0;
        if (grp == 0) {
            const int u = j >> 4, pt = (u & 3) * 16 + (u >> 2) * 2;
            st.foff = F_P + (u32)(pt * 16 + jb) * 8u; st.slot = OFF_OUT + c.par * OUT_BYTES + (u32)(pt * 16 + jb) * SLOT;
            st.p0 = (int)(OFF_OWN + (u32)((pt * 4 + 3) * 16 + jb) * SLOT); st.p1 = r; st.p2 = (int)(F_OWN + (u32)((pt * 4 + 3) * 16 + jb) * 8u);
        } else {
            const int second = j >= 256, v = (j >> 4) & 15, u = (v & 3) * 4 + (v >> 2), path = (grp - 1) ^ second, pt = 4 * u + 1 + 2 * path, rs = path * 16 + u;
            if (!second) {
                st.foff = F_R + (u32)(rs * 16 + jb) * 8u; st.slot = OFF_OUT + c.par * OUT_BYTES + (u32)(pt * 16 + jb) * SLOT;
                st.p0 = (int)(OFF_REL + c.par * REL_BYTES + (u32)(rs * 16 + jb) * SLOT); st.p1 = r ^ (path ? 1 : 3); st.p2 = (int)(F_P + (u32)(pt * 16 + jb) * 8u);
            } else {
                st.foff = F_P + (u32)(pt * 16 + jb) * 8u; st.slot = OFF_REL + c.par * REL_BYTES + (u32)(rs * 16 + jb) * SLOT;
                st.p0 = (int)(OFF_OWN + (u32)((pt * 4 + 3) * 16 + jb) * SLOT); st.p1 = r; st.p2 = (int)(F_OWN + (u32)((pt * 4 + 3) * 16 + jb) * 8u);
            }
        }
        st.p3 = 0; st.sel = 0;
        return st;
    };
    auto act = [&](Step& st, const Pull& p, rsrc_t rs, bool ok) {
        if (ok) copy_store(c, p, rs, st.slot, (u32)st.p0, st.meta);
        else { u32x4 d[8]; fill_nan(d); store_raw(d, c.own, (u32)st.p0, c.lane); st.meta = META_RAW; }
    };
    auto pub = [&](const Step& st) { post_flag(P, c, st.p1, (u32)st.p2, st.meta); };
    const rsrc_t rs = mk_rsrc(P.ws[(r ^ (grp == 0 ? 2 : grp == 1 ? 1 : 3)) & 7]);
    pull_list<1>(c, 6 + grp, (u32)widx, (u32)wpl, 512u, rs, rs, gen, act, pub);
}
__device__ __forceinline__ void reduce_list(const Params& P, Ctx& c, int h, int widx, int wpl) {
    const int r = c.rank;
    float acc[64];
    auto gen = [&](u32 ujob, int sub) {
        const int job = (int)ujob, sl = (sub ^ job) & 1, jb = job & 15, ot2 = job >> 4;
        const int a = 2 * h + (ot2 & 1), j = ot2 >> 1, ot = a * 16 + j, p = 2 * h + sl, ti = 4 + 2 * p + (a & 1);
        Step st; st.meta = 0;
        st.foff = F_D3 + (u32)((ot * 2 + sl) * 16 + jb) * 8u; st.sel = sl;
        st.slot = OFF_OUT + c.par * OUT_BYTES + (u32)((ti * 16 + j) * 16 + jb) * SLOT;
        st.p0 = sub; st.p1 = a; st.p2 = (1 - (r >> 2)) * 16 + j; st.p3 = jb;
        return st;
    };
    auto act = [&](Step& st, const Pull& p, rsrc_t rs, bool ok) {
        u32x4 d[8];
        if (ok) unpack(d, p, rs, st.slot, st.meta, c.lane, c.kp); else fill_nan(d);
        if (st.p0 == 0) {
#pragma unroll
            for (int q = 0; q < 64; ++q) acc[q] = 0.f;
        }
        acc_add(acc, d);
        if (st.p0 == 1) write_out(P, c, acc, st.p1, st.p2, st.p3);
    };
    auto pub = [&](const Step&) {};
    pull_list<2>(c, 9 + h, (u32)widx, (u32)wpl, 512u, mk_rsrc(P.ws[(r ^ d3_of_pair(2 * h)) & 7]), mk_rsrc(P.ws[(r ^ d3_of_pair(2 * h + 1)) & 7]), gen, act, pub);
}
__device__ __forceinline__ void own_job(const Params& P, const Ctx& c, int n) {
    const int pt = n >> 4, jb = n & 15;
    kp_clear(c.kp, c.lane);
    float acc[64];
#pragma unroll
    for (int i = 0; i < 64; ++i) acc[i] = 0.f;
    for (int part = 0; part < 4; ++part) {
        u32 meta; u32x4 d[8];
        if (wait_flag(c, F_OWN + (u32)((pt * 4 + part) * 16 + jb) * 8u, meta)) {
            Pull p; const u32 slot = OFF_OWN + (u32)((pt * 4 + part) * 16 + jb) * SLOT;
            pull_issue(p, c.own, slot, meta, c.lane); unpack(d, p, c.own, slot, meta, c.lane, c.kp);
        } else fill_nan(d);
        acc_add(acc, d);
    }
    write_out(P, c, acc, pt >> 4, (c.rank >> 2) * 16 + (pt & 15), jb);
}

struct GJob { int ti, tj, t0, nt; };

#define STAMP(i) do {} while (0)

__device__ __forceinline__ void gemm_role(const Params& P, const Ctx& c, LAS unsigned char* lds, int xcd, int idx) {
    const int tid = threadIdx.x, wid = (int)rfl(tid >> 6), lane = c.lane, wr = wid >> 2, wc = wid & 3;
    constexpr int HTB = 16384;
    const u32 voff = (u32)tid * 16u;
    const unsigned ldsw = (unsigned)wid * 1024u;
    const int aoff = wr * 8192 + lane * 16, boff = wc * 4096 + lane * 16;
    LAS volatile int* ctl = (LAS volatile int*)(lds + LDS_CTL);
    int avail = 0; bool bad = false;
#define G_SA(b, h) (((b) * 2 + (h)) * HTB)
#define G_SB(b, h) ((4 + (b) * 2 + (h)) * HTB)
#define G_STAGE(bufoff, gbase) do { _Pragma("unroll") for (int _i = 0; _i < 2; ++_i) \
        __builtin_amdgcn_global_load_lds((GAS const u32*)((GAS const char*)(gbase) + voff + _i * 8192), (LAS u32*)(lds + (bufoff) + ldsw + _i * 8192), 16, 0, AUX_DEV); } while (0)
#define G_LDA(dst, b, h) do { _Pragma("unroll") for (int m = 0; m < 4; ++m) _Pragma("unroll") for (int k = 0; k < 2; ++k) dst[m][k] = *(const LAS bf16x8*)(lds + G_SA(b, h) + aoff + m * 2048 + k * 1024); } while (0)
#define G_LDB(dst, b, h) do { _Pragma("unroll") for (int n = 0; n < 2; ++n) _Pragma("unroll") for (int k = 0; k < 2; ++k) dst[n][k] = *(const LAS bf16x8*)(lds + G_SB(b, h) + boff + n * 2048 + k * 1024); } while (0)
#define G_MMA(ai, bj, At, Bt) do { __builtin_amdgcn_s_setprio(1); _Pragma("unroll") for (int m = 0; m < 4; ++m) _Pragma("unroll") for (int n = 0; n < 2; ++n) _Pragma("unroll") for (int k = 0; k < 2; ++k) \
        acc[ai][bj][m][n] = __builtin_amdgcn_mfma_f32_16x16x32_bf16(Bt[n][k], At[m][k], acc[ai][bj][m][n], 0, 0, 0); __builtin_amdgcn_s_setprio(0); } while (0)
#define G_WAIT_V(n) asm volatile("s_waitcnt vmcnt(" #n ")" ::: "memory")
#define G_WAIT_L(n) asm volatile("s_waitcnt lgkmcnt(" #n ")" ::: "memory")
#define G_BAR __builtin_amdgcn_s_barrier()
#define G_SCHED __builtin_amdgcn_sched_barrier(0)
    auto ensure = [&](const GJob& j, int tneed) {
        if (avail >= tneed) return;
        if (wid == 0) {
            GAS const u32* ra = (GAS const u32*)(c.ws + OFF_RDYA) + j.ti * 64 + lane;
            GAS const u32* rb = (GAS const u32*)(c.ws + OFF_RDYB) + j.tj * 64 + lane;
            GAS u32* tmo = (GAS u32*)(c.ws + OFF_TMO);
            const u32 target = 4u * c.call;
            const u64 below = j.t0 ? ((1ull << j.t0) - 1ull) : 0ull;
            const u64 t0 = __builtin_amdgcn_s_memrealtime();
            int av, bd = 0;
            for (u32 it = 1;; ++it) {
                const u32 a = __hip_atomic_load(ra, __ATOMIC_RELAXED, __HIP_MEMORY_SCOPE_AGENT), b = __hip_atomic_load(rb, __ATOMIC_RELAXED, __HIP_MEMORY_SCOPE_AGENT);
                const u64 ok = __builtin_amdgcn_ballot_w64((int)(a - target) >= 0 && (int)(b - target) >= 0) | below;
                av = ok == ~0ull ? 64 : __builtin_ctzll(~ok);
                if (av >= tneed) break;
                if ((it & GR_TMOMASK) == 0 && (rfl(__hip_atomic_load(tmo, __ATOMIC_RELAXED, __HIP_MEMORY_SCOPE_AGENT)) == c.call || __builtin_amdgcn_s_memrealtime() - t0 > DEADLINE)) {
                    __hip_atomic_store(tmo, c.call, __ATOMIC_RELAXED, __HIP_MEMORY_SCOPE_AGENT); av = 64; bd = 1; break;
                }
                __builtin_amdgcn_s_sleep(GR_GSLEEP);
            }
            if (lane == 0) { ctl[0] = av; if (bd) ctl[1] = 1; }
            G_WAIT_L(0);
        }
        asm volatile("" ::: "memory"); G_BAR; asm volatile("" ::: "memory");
        avail = (int)rfl((u32)ctl[0]); bad = rfl((u32)ctl[1]) != 0;
    };
    GJob cur, nxt;
    cur.ti = 6 * (xcd & 1) + idx % 6; cur.tj = GR_XGRP ? 4 * (xcd >> 1) + idx / 6 : (xcd >> 1) + 4 * (idx / 6); cur.t0 = 0; cur.nt = 64;
    { const int col = xcd * 6 + (idx >> 2), third = col >> 4; nxt.ti = 12 + (idx & 3); nxt.tj = col & 15; nxt.t0 = 22 * third; nxt.nt = third == 2 ? 20 : 22; }
    f32x4 acc[2][2][4][2];
    bf16x8 At[4][2], B0[2][2], B1[2][2];
    constexpr size_t kstep = 32768, hstep = 16384;
    for (int ui = 0; ui < 2; ++ui) {
        const bool has_next = ui == 0;
#pragma unroll
        for (int a = 0; a < 2; ++a)
#pragma unroll
            for (int b = 0; b < 2; ++b)
#pragma unroll
                for (int m = 0; m < 4; ++m)
#pragma unroll
                    for (int n = 0; n < 2; ++n) acc[a][b][m][n] = (f32x4){0.f, 0.f, 0.f, 0.f};
        const char* cA = c.ws + OFF_A + (size_t)(cur.ti * 64 + cur.t0) * kstep;
        const char* cB = c.ws + OFF_B + (size_t)(cur.tj * 64 + cur.t0) * kstep;
        avail = 0;
        ensure(cur, cur.t0 + 2);
        G_STAGE(G_SB(0, 0), cB); G_STAGE(G_SA(0, 0), cA); G_STAGE(G_SB(0, 1), cB + hstep); G_STAGE(G_SA(0, 1), cA + hstep);
        if (wr == 1) G_BAR;
        G_WAIT_V(4); G_BAR;
        G_STAGE(G_SB(1, 0), cB + kstep); G_STAGE(G_SA(1, 0), cA + kstep); G_STAGE(G_SB(1, 1), cB + hstep + kstep);
        G_WAIT_V(6); G_BAR;
        if (ui == 0) STAMP(2);
        const int nt = cur.nt;
        for (int t = 0; t < nt; t += 2) {
            const bool last = (t == nt - 2);
            const char* a1 = cA + (size_t)(t + 1) * kstep;
            const char* a2 = cA + (size_t)(last ? t : t + 2) * kstep; const char* b2 = cB + (size_t)(last ? t : t + 2) * kstep;
            const char* a3 = a2 + kstep; const char* b3 = b2 + kstep;
            if (!last) ensure(cur, cur.t0 + t + 4);
            G_LDB(B0, 0, 0); G_SCHED; G_LDA(At, 0, 0); G_STAGE(G_SA(1, 1), a1 + hstep);
            G_WAIT_L(8); G_BAR; G_WAIT_L(0); G_MMA(0, 0, At, B0); G_BAR; G_SCHED;
            G_LDB(B1, 0, 1); G_STAGE(G_SB(0, 0), b2);
            G_BAR; G_WAIT_L(0); G_MMA(0, 1, At, B1); G_BAR;
            G_LDA(At, 0, 1); G_STAGE(G_SA(0, 0), a2);
            G_BAR; G_WAIT_L(0); G_MMA(1, 0, At, B0); G_BAR; G_SCHED;
            G_STAGE(G_SB(0, 1), b2 + hstep);
            G_WAIT_V(6); G_BAR; G_MMA(1, 1, At, B1); G_BAR;
            G_LDB(B0, 1, 0); G_SCHED; G_LDA(At, 1, 0); G_STAGE(G_SA(0, 1), a2 + hstep);
            G_WAIT_L(8); G_BAR; G_WAIT_L(0); G_MMA(0, 0, At, B0); G_BAR; G_SCHED;
            G_LDB(B1, 1, 1); G_STAGE(G_SB(1, 0), b3);
            G_BAR; G_WAIT_L(0); G_MMA(0, 1, At, B1); G_BAR;
            G_LDA(At, 1, 1); G_STAGE(G_SA(1, 0), a3);
            G_BAR; G_WAIT_L(0); G_MMA(1, 0, At, B0); G_BAR; G_SCHED;
            G_STAGE(G_SB(1, 1), b3 + hstep);
            G_WAIT_V(6); G_BAR; G_MMA(1, 1, At, B1); G_BAR;
        }
        if (wr == 0) G_BAR;
        G_WAIT_V(0); G_BAR;
        STAMP(3 + 2 * ui);
        {
            u32 meta[2], slot[2];
#pragma unroll
            for (int ai = 0; ai < 2; ++ai) {
                u32x4 d[8];
#pragma unroll
                for (int m = 0; m < 4; ++m)
#pragma unroll
                    for (int bj = 0; bj < 2; ++bj) {
                        const f32x4 v0 = acc[ai][bj][m][0], v1 = acc[ai][bj][m][1];
                        d[m * 2 + bj] = (u32x4){cvt_pk_bf16(v0[0], v0[1]), cvt_pk_bf16(v0[2], v0[3]), cvt_pk_bf16(v1[0], v1[1]), cvt_pk_bf16(v1[2], v1[3])};
                    }
                if (bad) fill_nan(d);
                const int jb = wid * 2 + ai;
                if (has_next) { slot[ai] = OFF_OUT + c.par * OUT_BYTES + (u32)((cur.ti * 16 + cur.tj) * 16 + jb) * SLOT; meta[ai] = pack_store(d, c.own, slot[ai], lane, c.kp); }
                else { slot[ai] = OFF_OWN + (u32)((((cur.ti - 12) * 16 + cur.tj) * 4 + (cur.t0 != 0) + (cur.t0 == 44)) * 16 + jb) * SLOT; store_raw(d, c.own, slot[ai], lane); meta[ai] = META_RAW; }
            }
            wait_vm0();
#pragma unroll
            for (int ai = 0; ai < 2; ++ai) {
                const int jb = wid * 2 + ai, r = c.rank;
                int dst; u32 foff;
                if (!has_next) { dst = r; foff = F_OWN + (u32)((((cur.ti - 12) * 16 + cur.tj) * 4 + (cur.t0 != 0) + (cur.t0 == 44)) * 16 + jb) * 8u; }
                else if (cur.ti < 4) {
                    const int pt = cur.ti * 16 + cur.tj;
                    if ((pt & 1) == 0) { dst = r ^ 2; foff = F_P + (u32)(pt * 16 + jb) * 8u; }
                    else { const int path = (pt >> 1) & 1; dst = r ^ (path ? 3 : 1); foff = F_R + (u32)((path * 16 + (pt >> 2)) * 16 + jb) * 8u; }
                } else {
                    const int p = (cur.ti - 4) >> 1, a = 2 * (p >> 1) + ((cur.ti - 4) & 1);
                    dst = r ^ d3_of_pair(p); foff = F_D3 + (u32)(((a * 16 + cur.tj) * 2 + (p & 1)) * 16 + jb) * 8u;
                }
                post_flag(P, c, dst, foff, meta[ai]);
            }
        }
        STAMP(4 + 2 * ui);
        cur = nxt;
    }
#undef G_SA
#undef G_SB
#undef G_STAGE
#undef G_LDA
#undef G_LDB
#undef G_MMA
#undef G_WAIT_V
#undef G_WAIT_L
#undef G_BAR
#undef G_SCHED
}

__global__ __launch_bounds__(NTH, 2) void gemm_rs_kernel(const Params P) {
    extern __shared__ __attribute__((aligned(16))) unsigned char shm[];
    LAS unsigned char* lds = (LAS unsigned char*)shm;
    const int wg = blockIdx.x, xcd = wg & 7, idx = wg >> 3, tid = threadIdx.x, wid = (int)rfl(tid >> 6);
    Ctx c;
    c.rank = P.rank & 7; c.lane = tid & 63;
    c.ws = (char*)P.ws[c.rank]; c.own = mk_rsrc(c.ws);
    c.kp = lds + wid * 16384;
    const bool gemm = idx < NGEMM / 8;
    const int gw = (xcd * (NGEMM / 8) + idx) * 8 + wid;
    const int cw = wid * NCOMM + (idx - NGEMM / 8) * 8 + xcd;
    int pj[5] = {-1, -1, -1, -1, -1}, pdelay = 0;
    u32x4 pin[8];
    const u64 tstart = __builtin_amdgcn_s_memrealtime();
    if (gemm) {
        const int grp = GR_XGRP ? xcd >> 1 : idx / 6, u = GR_TOPW < 0 ? (xcd * 6 + idx % 6) * 8 + wid : wid * 48 + (GR_XGRP ? idx * 2 + (xcd & 1) : (idx % 6) * 8 + xcd);
        int urg = 0, dummy;
        pj[0] = pack_pick(P, grp, u, 0, urg);
#pragma unroll
        for (int r = 1; r < 5; ++r) pj[r] = pack_pick(P, grp, u, r, dummy);
        pdelay = (urg * 4 - 16) * PACK_TAU / 16; pdelay = pdelay < 0 ? 0 : pdelay > PACK_DMAX ? PACK_DMAX : pdelay;
    }
    if (pj[0] >= 0 && pdelay == 0) pack_issue(P, c.lane, c.kp, pj[0], pin);
    LAS volatile u32* ctl = (LAS volatile u32*)(lds + LDS_CTL);
    if (tid == 0) {
        ctl[2] = __hip_atomic_fetch_add((GAS u32*)(c.ws + OFF_CALL + wg * 64), 1u, __ATOMIC_RELAXED, __HIP_MEMORY_SCOPE_AGENT) + 1u;
        ctl[0] = 0; ctl[1] = 0;
    }
    asm volatile("s_waitcnt lgkmcnt(0)" ::: "memory"); __builtin_amdgcn_s_barrier(); asm volatile("" ::: "memory");
    c.call = rfl(ctl[2]); c.par = c.call & 1u;
    STAMP(0);
    if (pdelay) {
        while (__builtin_amdgcn_s_memrealtime() - tstart < (u64)pdelay) __builtin_amdgcn_s_sleep(1);
        pack_issue(P, c.lane, c.kp, pj[0], pin);
    }
    pack_list(P, c, pj[0], pj[1], pj[2], pj[3], pj[4], pin);
    STAMP(1);
    if (gemm) {
        __syncthreads();
        gemm_role(P, c, lds, xcd, idx);
        if (gw < 1024) own_job(P, c, gw);
        STAMP(7);
    } else {
        const int ow = cw - O_FIRST;
        if (cw >= IN_WAVES && (ow < 0 || ow >= O_WAVES)) return;
        kp_clear(c.kp, c.lane);
#define GR_INMASK 63
        if (cw < IN_WAVES && ((GR_INMASK >> (cw % 6)) & 1)) in_list(P, c, cw % 6, cw / 6, IN_WPL);
        STAMP(3);
        if (ow >= 0 && ow < O_WAVES) {
            if (ow < 3 * C_WPL) copy_list(P, c, ow % 3, ow / 3, C_WPL);
            else reduce_list(P, c, (ow - 3 * C_WPL) & 1, (ow - 3 * C_WPL) >> 1, 2 * X_WPL);
        }
        STAMP(7);
    }
}
}

extern "C" void kernel_launch(int rank, int nranks, ncclComm_t comm, hipStream_t stream,
                              void* const* d_in, const int* in_sizes, int n_in,
                              void* d_out, int out_size,
                              const int* in_dtypes, int out_dtype,
                              void* const* d_out_all,
                              void* const* d_ws_all, size_t ws_bytes) {
    (void)comm; (void)in_sizes; (void)n_in; (void)out_size; (void)in_dtypes; (void)out_dtype; (void)d_out_all;
    if (nranks != gr::NR || rank < 0 || rank >= gr::NR || ws_bytes < gr::WS_NEED) { fprintf(stderr, "kernel_launch: unexpected configuration\n"); return; }
    static bool init[gr::NR] = {};
    if (!init[rank]) {
        if (hipFuncSetAttribute((const void*)gr::gemm_rs_kernel, hipFuncAttributeMaxDynamicSharedMemorySize, gr::LDS_BYTES) != hipSuccess) { fprintf(stderr, "hipFuncSetAttribute failed\n"); return; }
        init[rank] = true;
    }
    gr::Params P;
    P.x = d_in[0]; P.w = d_in[1]; P.out = d_out; P.rank = rank; P.pad = 0;
    for (int i = 0; i < gr::NR; ++i) P.ws[i] = d_ws_all[i];
    int blockpos[64];
    {
        int key[64]; unsigned char ord[64];
        for (int b = 0; b < 64; ++b) {
            const int kind = b & 7;
            key[b] = GR_CH * (b >> 3) + (kind == 4 ? GR_D1 : kind == 5 ? GR_D2 : kind == 6 ? GR_D3 : kind == 7 ? GR_DOWN : 0);
        }
        for (int b = 0; b < 64; ++b) {
            int pos = 0;
            for (int o = 0; o < 64; ++o) pos += key[o] < key[b] || (key[o] == key[b] && o < b);
            blockpos[b] = pos; ord[pos] = (unsigned char)b;
        }
        for (int i = 0; i < 16; ++i) P.inord[i] = ord[4 * i] | (ord[4 * i + 1] << 8) | (ord[4 * i + 2] << 16) | ((unsigned)ord[4 * i + 3] << 24);
    }
    {
        unsigned short ord[1024]; int need[1024], first[1026] = {};
        for (int j = 0; j < 1024; ++j) { need[j] = gr::need_of(rank, j, blockpos); ++first[need[j] + 1]; }
        for (int v = 0; v < 1025; ++v) first[v + 1] += first[v];
        for (int j = 0; j < 1024; ++j) ord[first[need[j]]++] = (unsigned short)(j | ((need[j] / 4 > 63 ? 63 : need[j] / 4) << 10));
        for (int i = 0; i < 512; ++i) P.order[i] = (unsigned)ord[2 * i] | ((unsigned)ord[2 * i + 1] << 16);
    }
    hipLaunchKernelGGL(gr::gemm_rs_kernel, dim3(gr::NWG), dim3(gr::NTH), gr::LDS_BYTES, stream, P);
    hipError_t e = hipGetLastError();
    if (e != hipSuccess) fprintf(stderr, "launch failed: %s\n", hipGetErrorString(e));
}
```

```cpp
#include <hip/hip_runtime.h>
#include <rccl/rccl.h>
#include <cstdio>
#include <cstddef>

namespace gr {
typedef unsigned u32;
typedef unsigned long long u64;
typedef unsigned short u16;
typedef u32 u32x4 __attribute__((ext_vector_type(4)));
typedef u32 u32x8 __attribute__((ext_vector_type(8)));
typedef u32 u32x2 __attribute__((ext_vector_type(2)));
typedef u16 u16x2 __attribute__((ext_vector_type(2)));
typedef short bf16x8 __attribute__((ext_vector_type(8)));
typedef short s16x4 __attribute__((ext_vector_type(4)));
typedef float f32x4 __attribute__((ext_vector_type(4)));
#define LAS __attribute__((address_space(3)))
#define GAS __attribute__((address_space(1)))
typedef __amdgpu_buffer_rsrc_t rsrc_t;

constexpr int NR = 8;
constexpr int NWG = 256, NTH = 512, NGEMM = 192, NCOMM = NWG - NGEMM;
constexpr int GEMM_WAVES = NGEMM * 8, COMM_WAVES = NCOMM * 8;
constexpr u32 SLOT = 8448;
constexpr u32 PAY_Q2 = 5120, PAY_X = 5632;
constexpr u32 XMAX = 128;
constexpr int KP_LANE = 80, KP_BYTES = 64 * KP_LANE;
constexpr u64 DEADLINE = 100000000ull;

constexpr u32 OFF_CALL = 0;
constexpr u32 OFF_TMO = 16384;
constexpr u32 OFF_RDYA = 20480;
constexpr u32 OFF_RDYB = 24576;
constexpr u32 OFF_TKT = 28672;
constexpr u32 OFF_FLG = 32768;
constexpr u32 F_IN = 0;
constexpr u32 F_D3 = F_IN + 2 * 16 * 64 * 4 * 8;
constexpr u32 F_P = F_D3 + 64 * 2 * 16 * 8;
constexpr u32 F_R = F_P + 64 * 16 * 8;
constexpr u32 F_OWN = F_R + 32 * 16 * 8;
constexpr u32 FLG_BYTES = F_OWN + 64 * 4 * 16 * 8;
constexpr u32 OFF_DBG = 512 * 1024;
constexpr u32 CTL_BYTES = 1024 * 1024;
static_assert(OFF_FLG + 2 * FLG_BYTES <= OFF_DBG, "control region");
constexpr u32 IN_BYTES = 4096 * SLOT;
constexpr u32 OUT_BYTES = 192 * 16 * SLOT;
constexpr u32 REL_BYTES = 32 * 16 * SLOT;
constexpr u32 OFF_IN = CTL_BYTES;
constexpr u32 OFF_OUT = OFF_IN + 2 * IN_BYTES;
constexpr u32 OFF_REL = OFF_OUT + 2 * OUT_BYTES;
#define GR_NPUSH 16
constexpr int NPUSH = GR_NPUSH;
static_assert(NPUSH % 16 == 0 && NPUSH <= 64, "whole blocks, at most four");
constexpr u32 PUSH_BYTES = 6 * 64 * SLOT;
constexpr u32 OFF_PUSH = OFF_REL + 2 * REL_BYTES;
constexpr u32 OFF_OWN = OFF_PUSH + 2 * PUSH_BYTES;
constexpr u32 OFF_A = OFF_OWN + 64 * 4 * 16 * SLOT;
constexpr u32 OFF_B = OFF_A + 16 * 64 * 32768;
constexpr u32 WS_NEED = OFF_B + 16 * 64 * 32768;
static_assert(WS_NEED <= 257u * 1024u * 1024u, "workspace");
static_assert(OFF_A % 256 == 0 && OFF_OUT % 128 == 0 && OFF_OWN % 128 == 0, "alignment");

constexpr u32 META_RAW = 1u << 16;
#define GR_IN_WPL 16
#define GR_C_WPL 14
#define GR_X_WPL 14
constexpr int IN_WPL = GR_IN_WPL, C_WPL = GR_C_WPL, X_WPL = GR_X_WPL;
constexpr int O_WAVES = 3 * C_WPL + 4 * X_WPL, IN_WAVES = 6 * IN_WPL;
#define GR_PL2 0
#define GR_SEPOUT 1
#define GR_PACK_TAU 60
#define GR_PACK_DMAX 500
#define GR_PACK_M 0xCCEF
#define GR_LOC0 64
#define GR_LOC1 14
#define GR_CH 78
#define GR_D1 0
#define GR_D2 0
#define GR_D3 0
#define GR_DOWN 10000
#define GR_TPERM 0
#define GR_SLEEP 2
#define GR_GSLEEP 4
#define GR_TMOMASK 15
#define GR_TOPW 1
#define GR_XGRP 0
constexpr int PACK_TAU = GR_PACK_TAU, PACK_DMAX = GR_PACK_DMAX;
constexpr u32 PACK_M = GR_PACK_M;
constexpr int O_FIRST = GR_SEPOUT ? IN_WAVES : 0;
static_assert(O_FIRST + O_WAVES <= COMM_WAVES && IN_WAVES <= COMM_WAVES, "waves");
#define GR_PUSHOWN 0
constexpr int PUSH_FIRST = GR_PUSHOWN ? IN_WAVES : IN_WAVES + O_WAVES;
static_assert(GR_SEPOUT && PUSH_FIRST + 6 * NPUSH <= COMM_WAVES, "waves");
#define GR_GAUX 16
constexpr int AUX_SYS = 17, AUX_DEV = GR_GAUX;
constexpr int LDS_STAGE = 131072, LDS_CTL = LDS_STAGE, LDS_BYTES = LDS_STAGE + 512;

#define GR_PF 0
#define GR_PFB 40960
#define GR_ND 32
constexpr int ND = GR_ND, NRP = (64 - ND) / 2;
static_assert(ND % 2 == 0 && ND >= 32 && ND <= 48, "relay split");
constexpr int PF_BYTES = GR_PFB;

struct Params {
    const void* x;
    const void* w;
    void* own;
    int rank;
    u32 ord0;
    void* out;
    int pad[6];
    void* ws[NR];
    u32 proute[16];
    u32 plist[3][16];
    u32 inord[16];
    u32 order[512];
};

__device__ __forceinline__ u32 rfl(u32 v) { return (u32)__builtin_amdgcn_readfirstlane((int)v); }
__device__ __forceinline__ u32 pk_add_s(u32 a, u32 s) { u32 r; asm("v_pk_add_u16 %0, %1, %2" : "=v"(r) : "v"(a), "s"(s)); return r; }
__device__ __forceinline__ u32 pk_sub_s(u32 a, u32 s) { u32 r; asm("v_pk_sub_u16 %0, %1, %2" : "=v"(r) : "v"(a), "s"(s)); return r; }
__device__ __forceinline__ u32 bfi(u32 m, u32 a, u32 b) { u32 r; asm("v_bfi_b32 %0, %1, %2, %3" : "=v"(r) : "s"(m), "v"(a), "v"(b)); return r; }
__device__ __forceinline__ u32 rotl(u32 x, int r) { return __builtin_amdgcn_alignbit(x, x, (u32)(32 - r)); }
__device__ __forceinline__ u32 rotr(u32 x, int r) { return __builtin_amdgcn_alignbit(x, x, (u32)r); }
__device__ __forceinline__ u32 ffbl(u32 x) { u32 r; asm("v_ffbl_b32 %0, %1" : "=v"(r) : "v"(x)); return r; }
__device__ __forceinline__ u32 wave_scan(u32 x) {
    x += (u32)__builtin_amdgcn_update_dpp(0, (int)x, 0x111, 0xF, 0xF, false);
    x += (u32)__builtin_amdgcn_update_dpp(0, (int)x, 0x112, 0xF, 0xF, false);
    x += (u32)__builtin_amdgcn_update_dpp(0, (int)x, 0x114, 0xF, 0xF, false);
    x += (u32)__builtin_amdgcn_update_dpp(0, (int)x, 0x118, 0xF, 0xF, false);
    x += (u32)__builtin_amdgcn_update_dpp(0, (int)x, 0x142, 0xA, 0xF, false);
    x += (u32)__builtin_amdgcn_update_dpp(0, (int)x, 0x143, 0xC, 0xF, false);
    return x;
}
__device__ __forceinline__ u32 cvt_pk_bf16(float lo, float hi) { u32 r; asm volatile("v_cvt_pk_bf16_f32 %0, %1, %2" : "=v"(r) : "v"(lo), "v"(hi)); return r; }
__device__ __forceinline__ rsrc_t mk_rsrc(void* p) { return __builtin_amdgcn_make_buffer_rsrc(p, (short)0, (int)WS_NEED, 0x00020000); }
__device__ __forceinline__ void wait_vm0() { asm volatile("s_waitcnt vmcnt(0)" ::: "memory"); }
__device__ __forceinline__ void wait_vm0_known() { __builtin_amdgcn_s_waitcnt(0x0F70); asm volatile("" ::: "memory"); }

struct Ctx {
    const void* x;
    const void* w;
    char* ws;
    rsrc_t own;
    u32 call, par;
    int rank, lane;
    u32 ord0;
    LAS unsigned char* kp;
};
#define JLOG(list, job, sub, k) do {} while (0)
#define DT0() do {} while (0)
#define DT(i) do {} while (0)
#define DDEP(d) do {} while (0)

__device__ __forceinline__ u64 flag_load(const Ctx& c, u32 foff) {
    return __hip_atomic_load((GAS const u64*)(c.ws + OFF_FLG + c.par * FLG_BYTES + foff), __ATOMIC_RELAXED, __HIP_MEMORY_SCOPE_SYSTEM);
}
__device__ __forceinline__ bool flag_test(const Ctx& c, u64 v, u32& meta) {
    meta = rfl((u32)v);
    return (int)(rfl((u32)(v >> 32)) - c.call) >= 0;
}
__device__ __forceinline__ bool wait_flag(const Ctx& c, u32 foff, u32& meta) {
    GAS u32* tmo = (GAS u32*)(c.ws + OFF_TMO);
    if (flag_test(c, flag_load(c, foff), meta)) return true;
    const u64 t0 = __builtin_amdgcn_s_memrealtime();
    for (u32 it = 1;; ++it) {
        __builtin_amdgcn_s_sleep(GR_SLEEP);
        if (flag_test(c, flag_load(c, foff), meta)) return true;
        if ((it & GR_TMOMASK) == 0 && (rfl(__hip_atomic_load(tmo, __ATOMIC_RELAXED, __HIP_MEMORY_SCOPE_AGENT)) == c.call || __builtin_amdgcn_s_memrealtime() - t0 > DEADLINE)) {
            __hip_atomic_store(tmo, c.call, __ATOMIC_RELAXED, __HIP_MEMORY_SCOPE_AGENT);
            meta = 0; return false;
        }
    }
}
__device__ __forceinline__ void post_flag(const Params& P, const Ctx& c, int dst, u32 foff, u32 meta) {
    if ((unsigned)dst >= (unsigned)NR) return;
    GAS u64* f = (GAS u64*)((char*)P.ws[dst] + OFF_FLG + c.par * FLG_BYTES + foff);
    if (c.lane == 0) __hip_atomic_store(f, ((u64)c.call << 32) | meta, __ATOMIC_RELAXED, __HIP_MEMORY_SCOPE_SYSTEM);
}

__device__ __forceinline__ void store_raw(const u32x4 (&d)[8], rsrc_t rs, u32 slot, int lane) {
#pragma unroll
    for (int v = 0; v < 8; ++v) __builtin_amdgcn_raw_buffer_store_b128(d[v], rs, (int)(slot + v * 1024 + lane * 16), 0, AUX_SYS);
}
__device__ __forceinline__ u32 kz_of(u32 L) { return ((0x8000u - L) & 0x7FFFu) >> 10; }
__device__ __forceinline__ u32 pack_store(const u32x4 (&d)[8], rsrc_t rs, u32 slot, int lane, LAS unsigned char* stage, bool two = false, rsrc_t rs2 = rsrc_t(), u32 slot2 = 0) {
    float m0 = 0.f, m1 = 0.f;
#pragma unroll
    for (int f = 0; f < 8; ++f) {
        asm("v_max3_f32 %0, |%1|, |%2|, %0" : "+v"(m0) : "v"(d[f][0]), "v"(d[f][1]));
        asm("v_max3_f32 %0, |%1|, |%2|, %0" : "+v"(m1) : "v"(d[f][2]), "v"(d[f][3]));
    }
    const u32 b0 = __builtin_bit_cast(u32, m0), b1 = __builtin_bit_cast(u32, m1);
    const u32 sum = (u32)__builtin_amdgcn_readlane((int)wave_scan(((b0 > b1 ? b0 : b1) >> 16) & 0x7FFFu), 63);
    const u32 top = ((sum >> 6) + 32u) >> 7, lo = top > 7u ? top - 7u : 0u, L = lo << 7, LL = L | (L << 16);
    u32 lb[16], hb[16], M[2] = {0u, 0u};
#pragma unroll
    for (int j = 0; j < 16; ++j) {
        const u32 w0 = pk_sub_s(d[j >> 1][(j & 1) * 2], LL), w1 = pk_sub_s(d[j >> 1][(j & 1) * 2 + 1], LL);
        lb[j] = __builtin_amdgcn_perm(w1, w0, 0x06040200u); hb[j] = __builtin_amdgcn_perm(w1, w0, 0x07050301u);
        M[j >> 3] = bfi(0x80808080u, (hb[j] & 0x7C7C7C7Cu) + 0x7C7C7C7Cu, M[j >> 3] >> 1);
    }
    const u32 c0 = (u32)__builtin_popcount(M[0]), c1 = (u32)__builtin_popcount(M[1]), incl = wave_scan(c0 + c1);
    const u32 cnt = (u32)__builtin_amdgcn_readlane((int)incl, 63);
    if (cnt > XMAX) { store_raw(d, rs, slot, lane); if (two) store_raw(d, rs2, slot2, lane); return META_RAW; }
    u32 Q[6];
#pragma unroll
    for (int g = 0; g < 2; ++g) {
        const u32* h = hb + 8 * g;
        Q[3 * g + 0] = bfi(0x60606060u, rotl(h[6], 6), bfi(0x1C1C1C1Cu, rotl(h[1], 3), h[0]));
        Q[3 * g + 1] = bfi(0x40404040u, rotl(h[7], 5), bfi(0x20202020u, rotl(h[6], 4), bfi(0x1C1C1C1Cu, rotl(h[3], 3), h[2])));
        Q[3 * g + 2] = bfi(0x60606060u, rotl(h[7], 6), bfi(0x1C1C1C1Cu, rotl(h[5], 3), h[4]));
    }
#pragma unroll
    for (int v = 0; v < 4; ++v)
        __builtin_amdgcn_raw_buffer_store_b128((u32x4){lb[4 * v], lb[4 * v + 1], lb[4 * v + 2], lb[4 * v + 3]}, rs, (int)(slot + v * 1024 + lane * 16), 0, AUX_SYS);
    __builtin_amdgcn_raw_buffer_store_b128((u32x4){Q[0], Q[1], Q[2], Q[3]}, rs, (int)(slot + 4096 + lane * 16), 0, AUX_SYS);
    __builtin_amdgcn_raw_buffer_store_b64((u32x2){Q[4], Q[5]}, rs, (int)(slot + PAY_Q2 + lane * 8), 0, AUX_SYS);
    if (two) {
#pragma unroll
        for (int v = 0; v < 4; ++v)
            __builtin_amdgcn_raw_buffer_store_b128((u32x4){lb[4 * v], lb[4 * v + 1], lb[4 * v + 2], lb[4 * v + 3]}, rs2, (int)(slot2 + v * 1024 + lane * 16), 0, AUX_SYS);
        __builtin_amdgcn_raw_buffer_store_b128((u32x4){Q[0], Q[1], Q[2], Q[3]}, rs2, (int)(slot2 + 4096 + lane * 16), 0, AUX_SYS);
        __builtin_amdgcn_raw_buffer_store_b64((u32x2){Q[4], Q[5]}, rs2, (int)(slot2 + PAY_Q2 + lane * 8), 0, AUX_SYS);
    }
    if (cnt) {
        LAS unsigned char* mine = stage + lane * KP_LANE;
        LAS u16* list = (LAS u16*)(stage + KP_BYTES);
#pragma unroll
        for (int i = 0; i < 4; ++i) *(LAS u32x4*)(mine + 16 * i) = (u32x4){hb[4 * i], hb[4 * i + 1], hb[4 * i + 2], hb[4 * i + 3]};
        const u32 kz = kz_of(L);
        u32 off[2] = {incl - c0 - c1, incl - c1};
        bool bad = false;
        do {
#pragma unroll
            for (int g = 0; g < 2; ++g) {
                const bool has = M[g] != 0u;
                const u32 p = ffbl(M[g]), pos = (u32)(g * 32) + ((p & 7u) << 2) + ((p >> 3) & 3u);
                M[g] &= M[g] - 1u;
                const u32 k5 = ((u32)mine[pos] >> 2) & 31u;
                const bool z = k5 == kz;
                bad |= has && !z && ((k5 + 7u) & 31u) >= 15u;
                list[has ? off[g] : XMAX] = (u16)((((u32)lane * 64u + pos) << 4) | (z ? 8u : k5 & 15u));
                off[g] += has ? 1u : 0u;
            }
        } while (__builtin_amdgcn_ballot_w64((M[0] | M[1]) != 0u));
        if (__builtin_amdgcn_ballot_w64(bad)) { wait_vm0_known(); store_raw(d, rs, slot, lane); if (two) store_raw(d, rs2, slot2, lane); return META_RAW; }
        const u32 e = ((LAS const u32*)list)[lane];
        if (2u * (u32)lane < cnt) {
            __builtin_amdgcn_raw_buffer_store_b32(e, rs, (int)(slot + PAY_X + lane * 4), 0, AUX_SYS);
            if (two) __builtin_amdgcn_raw_buffer_store_b32(e, rs2, (int)(slot2 + PAY_X + lane * 4), 0, AUX_SYS);
        }
    }
    return lo | (cnt << 8);
}
struct Pull { u32x4 V[5]; u32x2 q; u32 h; };
__device__ __forceinline__ void pull_issue(Pull& p, rsrc_t rs, u32 slot, u32 meta, int lane) {
#pragma unroll
    for (int v = 0; v < 5; ++v) p.V[v] = __builtin_amdgcn_raw_buffer_load_b128(rs, (int)(slot + v * 1024 + lane * 16), 0, AUX_SYS);
    p.q = __builtin_amdgcn_raw_buffer_load_b64(rs, (int)(slot + PAY_Q2 + lane * 8), 0, AUX_SYS);
    const u32 hoff = 2u * (u32)lane < ((meta >> 8) & 0xFFu) ? slot + PAY_X + (u32)lane * 4u : 0x80000000u;
    p.h = __builtin_amdgcn_raw_buffer_load_b32(rs, (int)hoff, 0, AUX_SYS);
}
__device__ __forceinline__ void kp_clear(LAS unsigned char* kp, int lane) {
#pragma unroll
    for (int i = 0; i < 5; ++i) *(LAS u32x4*)(kp + lane * KP_LANE + 16 * i) = (u32x4){0u, 0u, 0u, 0u};
}
__device__ __forceinline__ void unpack(u32x4 (&d)[8], const Pull& p, rsrc_t rs, u32 slot, u32 meta, int lane, LAS unsigned char* kp) {
    if (meta & META_RAW) {
#pragma unroll
        for (int v = 0; v < 5; ++v) d[v] = p.V[v];
#pragma unroll
        for (int v = 5; v < 8; ++v) d[v] = __builtin_amdgcn_raw_buffer_load_b128(rs, (int)(slot + v * 1024 + lane * 16), 0, AUX_SYS);
        wait_vm0_known();
        return;
    }
    const u32 cnt = (meta >> 8) & 0xFFu;
    const u32 L = (meta & 0xFFu) << 7, LL = L | (L << 16), kz = kz_of(L);
    u32 at[2];
#pragma unroll
    for (int i = 0; i < 2; ++i) {
        const u32 e = i ? p.h >> 16 : p.h & 0xFFFFu, pos = e >> 4, k4 = e & 15u;
        at[i] = 2u * (u32)lane + (u32)i < cnt ? pos + (pos >> 6) * 16u : (u32)KP_BYTES;
        kp[at[i]] = (unsigned char)((k4 == 8u ? kz : k4 | ((k4 & 8u) << 1)) << 2);
    }
    u32x4 K[4];
#pragma unroll
    for (int i = 0; i < 4; ++i) K[i] = *(LAS const u32x4*)(kp + lane * KP_LANE + 16 * i);
    kp[at[0]] = 0; kp[at[1]] = 0;
    const u32 Q[6] = {p.V[4][0], p.V[4][1], p.V[4][2], p.V[4][3], p.q[0], p.q[1]};
#pragma unroll
    for (int g = 0; g < 2; ++g) {
        const u32 q0 = Q[3 * g], q1 = Q[3 * g + 1], q2 = Q[3 * g + 2];
        const u32x4 k0 = K[2 * g], k1 = K[2 * g + 1];
        u32 h[8];
        h[0] = (q0 & 0x83838383u) | k0[0]; h[1] = (rotr(q0, 3) & 0x83838383u) | k0[1];
        h[2] = (q1 & 0x83838383u) | k0[2]; h[3] = (rotr(q1, 3) & 0x83838383u) | k0[3];
        h[4] = (q2 & 0x83838383u) | k1[0]; h[5] = (rotr(q2, 3) & 0x83838383u) | k1[1];
        h[6] = (rotr(q0, 6) & 0x81818181u) | ((rotr(q1, 4) & 0x02020202u) | k1[2]);
        h[7] = (rotr(q2, 6) & 0x81818181u) | ((rotr(q1, 5) & 0x02020202u) | k1[3]);
#pragma unroll
        for (int j = 0; j < 8; ++j) {
            const u32 B = p.V[2 * g + (j >> 2)][j & 3];
            d[4 * g + (j >> 1)][(j & 1) * 2] = pk_add_s(__builtin_amdgcn_perm(h[j], B, 0x05010400u), LL);
            d[4 * g + (j >> 1)][(j & 1) * 2 + 1] = pk_add_s(__builtin_amdgcn_perm(h[j], B, 0x07030602u), LL);
        }
    }
}
__device__ __forceinline__ void fill_nan(u32x4 (&d)[8]) {
#pragma unroll
    for (int v = 0; v < 8; ++v) d[v] = (u32x4){0x7FC07FC0u, 0x7FC07FC0u, 0x7FC07FC0u, 0x7FC07FC0u};
}
__device__ __forceinline__ void copy_store(const Ctx& c, const Pull& p, rsrc_t rs, u32 slot, u32 dst, u32 meta) {
    const int lane = c.lane;
    if (meta & META_RAW) {
        u32x4 a[3];
#pragma unroll
        for (int v = 0; v < 3; ++v) a[v] = __builtin_amdgcn_raw_buffer_load_b128(rs, (int)(slot + (5 + v) * 1024 + lane * 16), 0, AUX_SYS);
        wait_vm0_known();
#pragma unroll
        for (int v = 0; v < 3; ++v) __builtin_amdgcn_raw_buffer_store_b128(a[v], c.own, (int)(dst + (5 + v) * 1024 + lane * 16), 0, AUX_SYS);
    } else {
        __builtin_amdgcn_raw_buffer_store_b64(p.q, c.own, (int)(dst + PAY_Q2 + lane * 8), 0, AUX_SYS);
        if (2u * (u32)lane < ((meta >> 8) & 0xFFu)) __builtin_amdgcn_raw_buffer_store_b32(p.h, c.own, (int)(dst + PAY_X + lane * 4), 0, AUX_SYS);
    }
#pragma unroll
    for (int v = 0; v < 5; ++v) __builtin_amdgcn_raw_buffer_store_b128(p.V[v], c.own, (int)(dst + v * 1024 + lane * 16), 0, AUX_SYS);
}

__device__ __forceinline__ u32 ticket_issue(const Ctx& c, u32 off) {
    u32 r; u64 sv; const u32 one = 1u;
    asm volatile("s_mov_b64 %1, exec\n\ts_mov_b64 exec, 1\n\tglobal_atomic_add %0, %2, %3, %4 sc0\n\ts_mov_b64 exec, %1"
                 : "=&v"(r), "=&s"(sv) : "v"(off), "v"(one), "s"(c.ws) : "memory");
    return r;
}
__device__ __forceinline__ u32 ticket_get(u32 r) { asm volatile("" : "+v"(r) :: "memory"); return rfl(r); }

struct Step { u32 foff, slot, meta; int sel, p0, p1, p2, p3; };
template <int SPT, class Gen, class Act, class Pub>
__device__ __forceinline__ void pull_list(Ctx& c, int list, u32 first, u32 nwaves, u32 njobs, rsrc_t rs0, rsrc_t rs1, Gen gen, Act act, Pub pub) {
    DT0();
    const u32 tkoff = OFF_TKT + (u32)list * 128u, tkbase = nwaves - (c.call - 1u) * njobs;
    u32 jc = first, jn = 0, tk = 0;
    if (SPT == 1) tk = ticket_issue(c, tkoff);
    Step cur = gen(jc, 0), prev = cur;
    bool ok = wait_flag(c, cur.foff, cur.meta), have_prev = false, more = true;
    if (SPT == 1) { wait_vm0_known(); jn = ticket_get(tk) + tkbase; }
    DT(1);
    Pull pa, pb;
    JLOG(list, jc, 0, 0);
    pull_issue(pa, cur.sel ? rs1 : rs0, cur.slot, cur.meta, c.lane);
    auto half = [&](const Pull& p, Pull& pn, int sub) {
        const bool same = SPT == 2 && sub == 0;
        more = same || jn < njobs;
        Step nxt = gen(same || !more ? jc : jn, same ? 1 : 0);
        const bool draw = SPT == 1 ? more : same;
        if (draw) tk = ticket_issue(c, tkoff);
        const u64 fv = flag_load(c, nxt.foff);
        wait_vm0_known();
        DT(0);
        JLOG(list, jc, sub, 1);
        const u32 jd = draw ? ticket_get(tk) + tkbase : njobs;
        if (have_prev) pub(prev);
        const bool nready = more && flag_test(c, fv, nxt.meta);
        if (nready) { JLOG(list, same ? jc : jn, same ? 1 : 0, 0); pull_issue(pn, nxt.sel ? rs1 : rs0, nxt.slot, nxt.meta, c.lane); }
        DT(3);
        act(cur, p, cur.sel ? rs1 : rs0, ok, []() {});
        prev = cur; have_prev = true; ok = true;
        DT(2);
        if (more && !nready) {
            wait_vm0_known(); pub(prev); have_prev = false;
            ok = wait_flag(c, nxt.foff, nxt.meta);
            JLOG(list, same ? jc : jn, same ? 1 : 0, 0);
            pull_issue(pn, nxt.sel ? rs1 : rs0, nxt.slot, nxt.meta, c.lane);
            DT(1);
        }
        cur = nxt;
        if (SPT == 1) { jc = jn; jn = jd; }
        else if (same) jn = jd; else jc = jn;
    };
    for (;;) {
        half(pa, pb, 0); if (!more) break;
        half(pb, pa, 1); if (!more) break;
    }
    wait_vm0_known();
    if (have_prev) pub(prev);
}

template <int N> __device__ __forceinline__ void wait_vm_known() { __builtin_amdgcn_s_waitcnt(0x0F70 | (N & 15) | ((N >> 4) << 14)); asm volatile("" ::: "memory"); }
template <int SPT, int NS0, int NS1, class Gen, class Act, class Pub>
__device__ __forceinline__ void pull_list2(Ctx& c, int list, u32 first, u32 nwaves, u32 njobs, rsrc_t rs0, rsrc_t rs1, Gen gen, Act act, Pub pub) {
    DT0();
    const u32 tkoff = OFF_TKT + (u32)list * 128u, tkbase = nwaves - (c.call - 1u) * njobs;
    u32 tk = ticket_issue(c, tkoff);
    Step cur = gen(first, 0), prev = cur;
    bool ok = wait_flag(c, cur.foff, cur.meta), have_prev = false;
    wait_vm0_known();
    u32 jc = first, jb = ticket_get(tk) + tkbase;
    DT(1);
    Pull pa, pb;
    JLOG(list, jc, 0, 0);
    pull_issue(pa, cur.sel ? rs1 : rs0, cur.slot, cur.meta, c.lane);
    bool more1 = SPT == 2 || jb < njobs, drawn = SPT == 1 && more1;
    Step nxt = gen(SPT == 2 || !more1 ? jc : jb, SPT == 2 ? 1 : 0);
    if (drawn) tk = ticket_issue(c, tkoff);
    u64 fv = flag_load(c, nxt.foff);
    wait_vm0_known();
    auto half = [&](const Pull& p, Pull& pn, int sub) {
        DT(0);
        JLOG(list, jc, sub, 1);
        u32 j2; bool more2;
        if (SPT == 1) { j2 = drawn ? ticket_get(tk) + tkbase : njobs; more2 = j2 < njobs; }
        else { if (sub == 0 && drawn) jb = ticket_get(tk) + tkbase; j2 = jb; more2 = jb < njobs; }
        Step nn = gen(more2 ? j2 : jc, SPT == 2 ? sub : 0);
        const bool nready = more1 && flag_test(c, fv, nxt.meta);
        if (nready) { JLOG(list, SPT == 2 && sub == 0 ? jc : jb, SPT == 2 ? sub ^ 1 : 0, 0); pull_issue(pn, nxt.sel ? rs1 : rs0, nxt.slot, nxt.meta, c.lane); }
        drawn = more2 && (SPT == 1 || sub == 1);
        if (drawn) tk = ticket_issue(c, tkoff);
        fv = flag_load(c, nn.foff);
        DT(3);
        auto mid = [&]() {
            if (!have_prev) return;
            if (nready) { if (drawn) asm volatile("s_waitcnt vmcnt(9)" ::: "memory"); else asm volatile("s_waitcnt vmcnt(8)" ::: "memory"); }
            else asm volatile("s_waitcnt vmcnt(0)" ::: "memory");
            pub(prev);
        };
        act(cur, p, cur.sel ? rs1 : rs0, ok, mid);
        prev = cur; have_prev = true; ok = true;
        DT(2);
        if (more1 && !nready) {
            wait_vm0_known(); pub(prev); have_prev = false;
            ok = wait_flag(c, nxt.foff, nxt.meta);
            JLOG(list, SPT == 2 && sub == 0 ? jc : jb, SPT == 2 ? sub ^ 1 : 0, 0);
            pull_issue(pn, nxt.sel ? rs1 : rs0, nxt.slot, nxt.meta, c.lane);
            fv = flag_load(c, nn.foff);
            wait_vm0_known();
            DT(1);
        }
        if (SPT == 2 && sub == 0) wait_vm_known<NS0>(); else wait_vm_known<NS1>();
        const bool go = more1;
        cur = nxt; nxt = nn; more1 = more2;
        if (SPT == 1) { jc = jb; jb = j2; }
        else if (sub == 1) jc = jb;
        return go;
    };
    for (;;) {
        if (!half(pa, pb, 0)) break;
        if (!half(pb, pa, 1)) break;
    }
    wait_vm0_known();
    if (have_prev) pub(prev);
}

__host__ __device__ __forceinline__ int cq_of(int q, int r) {
    return ((r >> 1) ^ ((q & 1) ? (r >> 2) : 0) ^ ((q & 2) ? r : 0)) & 1;
}
__host__ __device__ __forceinline__ int link_d(int link) { return (0x765431 >> (4 * link)) & 7; }
__host__ __device__ __forceinline__ int link_q(int link, int i) { return ((i ? 0x233231 : 0x110020) >> (4 * link)) & 3; }
__host__ __device__ __forceinline__ int link_of_d(int d) { return d == 1 ? 0 : d - 2; }
__host__ __device__ __forceinline__ int t_chunk(int kc) { return GR_TPERM && kc < 2 ? 8 + 10 * kc : 16 + 6 * kc; }
__host__ __device__ __forceinline__ int t_own(int kq) { return GR_TPERM && kq >= 2 ? 10 * kq - 6 : 4 * kq; }
__host__ __device__ __forceinline__ int chunk_of(int d, int q, int kq) { return 2 * kq + (link_q(link_of_d(d), 0) == q ? 0 : 1); }
__host__ __device__ __forceinline__ int t_of(int d, int q, int kq) { return d == 0 ? t_own(kq) + q : t_chunk(chunk_of(d, q, kq)) + link_of_d(d); }
__host__ __device__ __forceinline__ int d3_pair(int d) { return (d & 1) * 2 + ((d >> 1) & 1); }
__host__ __device__ __forceinline__ int d3_of_pair(int p) { return 4 + (p >> 1) + (p & 1) * 2; }
__host__ __device__ __forceinline__ int rowtile_of(int rc, int o, int a) {
    const int d = rc ^ o;
    return d == 0 ? 12 + a : d == 2 ? a : 4 + 2 * d3_pair(d) + (a & 1);
}
__device__ __forceinline__ void block_of_rowtile(int rc, int i, int& o, int& a) {
    if (i >= 12) { o = rc; a = i - 12; }
    else if (i < 4) { o = rc ^ 2; a = i; }
    else { const int p = (i - 4) >> 1; o = rc ^ d3_of_pair(p); a = 2 * (p >> 1) + ((i - 4) & 1); }
}
__device__ __forceinline__ u32 raw_frag_off(int type, int sub, int f) {
    return type == 0 ? (u32)(sub * 8192 + f * 1024) : (u32)((f >> 2) * 16384 + sub * 4096 + (f & 3) * 1024);
}
__device__ __forceinline__ void store_operand(const Ctx& c, const u32x4 (&d)[8], int type, int tile, int t, int sub) {
    const u32 base = (type == 0 ? OFF_A : OFF_B) + (u32)(tile * 64 + t) * 32768u + (u32)c.lane * 16u;
#pragma unroll
    for (int f = 0; f < 8; ++f) __builtin_amdgcn_raw_buffer_store_b128(d[f], c.own, (int)(base + raw_frag_off(type, sub, f)), 0, AUX_SYS);
}
__device__ __forceinline__ void count_operand(const Ctx& c, int type, int tile, int t) {
    GAS u32* p = (GAS u32*)(c.ws + (type == 0 ? OFF_RDYA : OFF_RDYB)) + tile * 64 + t;
    if (c.lane == 0) __hip_atomic_fetch_add(p, 1u, __ATOMIC_RELAXED, __HIP_MEMORY_SCOPE_AGENT);
}

__host__ __device__ __forceinline__ void consumers_of(int s, int pj, int (&rc)[2], int (&tl)[2], int (&tt)[2]) {
    const int kq = pj >> 10, q = pj & 3, type = (pj >> 2) & 1, g = (pj & 1023) >> 3;
    const int cs = cq_of(q, s);
#pragma unroll
    for (int i = 0; i < 2; ++i) {
        int e1, gam;
        if (type == 0) { const int o = g >> 4, h = (g >> 3) & 1; gam = i; e1 = h == 0 ? (o & 1) : ((o ^ (o >> 2) ^ gam) & 1); }
        else { gam = g >> 6; e1 = i; }
        const int e2 = (cs ^ ((q & 2) ? e1 : 0) ^ ((q & 1) ? gam : 0)) & 1;
        rc[i] = e1 | (e2 << 1) | (gam << 2);
        tl[i] = type == 0 ? rowtile_of(rc[i], g >> 4, (g >> 2) & 3) : ((g >> 2) & 15);
        tt[i] = t_of(rc[i] ^ s, q, kq);
    }
}
__host__ inline int need_of(int s, int pj, const int (&blockpos)[64]) {
    int rc[2], tl[2], tt[2], need = 1023;
    consumers_of(s, pj, rc, tl, tt);
    const int kq = pj >> 10, q = pj & 3, type = (pj >> 2) & 1, sub = (pj >> 3) & 3;
    for (int i = 0; i < 2; ++i) {
        const int kind = type == 0 ? (tl[i] >= 12 ? 7 : tl[i] >> 2) : 3 + (tl[i] >> 2);
        const int pos = rc[i] != s ? blockpos[chunk_of(rc[i] ^ s, q, kq) * 8 + kind] * 16 + (tl[i] & 3) * 4 + sub : kind == 7 ? 896 : GR_LOC0 + GR_LOC1 * tt[i];
        if (pos < need) need = pos;
    }
    return need;
}
__device__ __forceinline__ void pack_issue(const Ctx& P, int lane, LAS unsigned char* tile, int pj, u32x4 (&d)[8]) {
    const int kq = pj >> 10, q = pj & 3, type = (pj >> 2) & 1, g = (pj & 1023) >> 3, kt = 4 * q + kq;
    if (type == 0) {
        GAS const char* src = (GAS const char*)P.x + (size_t)(64 * g + (lane & 15)) * 2048 + (64 * kt + 8 * (lane >> 4)) * 2;
#pragma unroll
        for (int f = 0; f < 8; ++f) d[f] = *(GAS const u32x4*)(src + (f >> 1) * 16 * 2048 + (f & 1) * 64);
    } else {
        GAS const char* src = (GAS const char*)P.w + (size_t)(64 * kt + (lane >> 3)) * 16384 + (64 * g) * 2 + (lane & 7) * 16;
#pragma unroll
        for (int v = 0; v < 8; ++v)
            __builtin_amdgcn_global_load_lds((GAS const u32*)(src + (size_t)v * 8 * 16384), (LAS u32*)(tile + v * 1024), 16, 0, 0);
    }
}
__device__ __forceinline__ void pack_job(const Params& P, const Ctx& c, LAS unsigned char* tile, LAS unsigned char* tile2, int pj, int next, u32x4 (&in)[8]) {
    const int kq = pj >> 10, q = pj & 3, type = (pj >> 2) & 1, g = (pj & 1023) >> 3, kt = 4 * q + kq, lane = c.lane, s = c.rank;
    u32x4 d[8];
    if (type == 0) {
#pragma unroll
        for (int f = 0; f < 8; ++f) d[f] = in[f];
    } else {
        wait_vm0();
        const int lg = lane >> 4, lq = (lane >> 2) & 3, lp = lane & 3;
        LAS unsigned char* b = tile + (8 * lg + lq) * 128 + lp * 16;
#pragma unroll
        for (int f = 0; f < 8; ++f) {
            const int o = (f & 1) * 32 * 128 + (f >> 2) * 64 + ((f >> 1) & 1) * 8;
            const s16x4 lo4 = __builtin_amdgcn_ds_read_tr16_b64_v4i16((LAS s16x4*)(b + o));
            const s16x4 hi4 = __builtin_amdgcn_ds_read_tr16_b64_v4i16((LAS s16x4*)(b + o + 4 * 128));
            const u32x2 l2 = __builtin_bit_cast(u32x2, lo4), h2 = __builtin_bit_cast(u32x2, hi4);
            d[f] = (u32x4){l2[0], l2[1], h2[0], h2[1]};
        }
        asm volatile("s_waitcnt lgkmcnt(0)" ::: "memory");
    }
    if (next >= 0) pack_issue(c, lane, tile2, next, in);
    int rc[2], tl[2], tt[2];
    consumers_of(s, pj, rc, tl, tt);
    const int sub = g & 3;
    int dr[2]; u32 ds[2];
#pragma unroll
    for (int i = 0; i < 2; ++i) {
        dr[i] = s; ds[i] = OFF_IN + c.par * IN_BYTES + (u32)(type * 2048 + kt * 128 + g) * SLOT;
        const int dd = rc[i] ^ s;
        if (dd != 0) {
            const u32 blk = (u32)(chunk_of(dd, q, kq) * 8 + (type == 0 ? (tl[i] >= 12 ? 7 : tl[i] >> 2) : 3 + (tl[i] >> 2)));
#pragma unroll
            for (int b = 0; b < NPUSH / 16; ++b)
                if (((c.ord0 >> (8 * b)) & 255u) == blk) {
                    dr[i] = rc[i]; ds[i] = OFF_PUSH + c.par * PUSH_BYTES + (u32)(link_of_d(dd) * 64 + b * 16 + (tl[i] & 3) * 4 + sub) * SLOT;
                }
        }
    }
    const int first = rc[0] == s ? 1 : 0;
    const bool two = rc[0] != s && rc[1] != s && (dr[0] != s || dr[1] != s);
    const u32 meta = pack_store(d, mk_rsrc(P.ws[dr[first] & 7]), ds[first], lane, tile, two, mk_rsrc(P.ws[dr[1] & 7]), ds[1]);
    if (rc[0] == s) store_operand(c, d, type, tl[0], tt[0], sub);
    if (rc[1] == s) store_operand(c, d, type, tl[1], tt[1], sub);
    wait_vm0();
#pragma unroll
    for (int i = 0; i < 2; ++i) {
        if (rc[i] == s) count_operand(c, type, tl[i], tt[i]);
        else post_flag(P, c, rc[i], F_IN + (u32)(((type * 16 + tl[i]) * 64 + tt[i]) * 4 + sub) * 8u, meta);
    }
}
__device__ __forceinline__ int pack_pos(int grp, int u, int r) {
    int off = 0;
#pragma unroll
    for (int i = 0; i < 6; ++i) if (i < r) off += 384 * __builtin_popcount((PACK_M >> (4 * i)) & 15u);
    const u32 m = (PACK_M >> (4 * r)) & 15u;
    int i = off + 384 * __builtin_popcount(m & ((1u << grp) - 1u)) + u;
    if (GR_TOPW > 0 && r == 0) {
        const int w = u / 48, g = u % 48;
        i = w < GR_TOPW ? w * 192 + g * 4 + grp : 192 * GR_TOPW + grp * 48 * (8 - GR_TOPW) + (w - GR_TOPW) * 48 + g;
    }
    return !((m >> grp) & 1u) || i >= 4096 ? -1 : i;
}
__device__ __forceinline__ int pack_pick(int i, u32 e, int& urg) {
    e = (e >> (16 * (i & 1))) & 0xFFFFu;
    urg = i < 1024 ? (int)(e >> 10) : 63;
    return i < 0 ? -1 : (i & ~1023) | (int)(e & 1023u);
}
__device__ __forceinline__ void pack_list(const Params& P, const Ctx& c, int j0, int j1, int j2, int j3, int j4, u32x4 (&in)[8]) {
    for (u32 buf = 0; j0 >= 0; buf ^= 8192u) { pack_job(P, c, c.kp + buf, c.kp + (buf ^ 8192u), j0, j1, in); j0 = j1; j1 = j2; j2 = j3; j3 = j4; j4 = -1; }
}

__device__ __forceinline__ Step in_step(const Ctx& c, u32 tbl, int link, int j) {
    const int blk = __builtin_amdgcn_readlane((int)tbl, (j >> 4) & 63), kc = (blk >> 3) & 7, kind = blk & 7;
    const int type = kind >= 3 && kind < 7, tile = (kind == 7 ? 12 : type ? 4 * (kind - 3) : 4 * kind) + ((j >> 2) & 3), sub = j & 3;
    const int t = t_chunk(kc) + link, kt = 4 * link_q(link, kc & 1) + (kc >> 1);
    int g;
    if (type == 0) { int o, a; block_of_rowtile(c.rank, tile, o, a); g = o * 16 + a * 4 + sub; } else g = (c.rank >> 2) * 64 + tile * 4 + sub;
    Step st; st.foff = F_IN + (u32)(((type * 16 + tile) * 64 + t) * 4 + sub) * 8u;
    st.slot = OFF_IN + c.par * IN_BYTES + (u32)(type * 2048 + kt * 128 + g) * SLOT;
    st.sel = 0; st.meta = 0; st.p0 = type; st.p1 = tile; st.p2 = t; st.p3 = sub;
    return st;
}
__device__ __forceinline__ u32 in_table(const Params& P, const Ctx& c) { return P.inord[(c.lane >> 2) & 15] >> (8 * (c.lane & 3)); }
__device__ __forceinline__ void in_list(const Params& P, Ctx& c, int link, int widx, int wpl) {
    const int s = c.rank ^ link_d(link);
    const u32 tbl = in_table(P, c);
    auto gen = [&](u32 job, int) { return in_step(c, tbl, link, (int)job + NPUSH); };
    auto act = [&](Step& st, const Pull& p, rsrc_t rs, bool ok, auto mid) {
        u32x4 d[8];
        if (ok) unpack(d, p, rs, st.slot, st.meta, c.lane, c.kp); else fill_nan(d);
        mid();
        store_operand(c, d, st.p0, st.p1, st.p2, st.p3);
    };
    auto pub = [&](const Step& st) { count_operand(c, st.p0, st.p1, st.p2); };
    const rsrc_t rs = mk_rsrc(P.ws[s & 7]);
    pull_list<1>(c, link, (u32)widx, (u32)wpl, 1024u - NPUSH, rs, rs, gen, act, pub);
}
__device__ __forceinline__ void push_job(const Params& P, Ctx& c, int link, int j) {
    Step st = in_step(c, in_table(P, c), link, j);
    st.slot = OFF_PUSH + c.par * PUSH_BYTES + (u32)(link * 64 + j) * SLOT;
    u32x4 d[8];
    if (wait_flag(c, st.foff, st.meta)) {
        JLOG(link, 1024 - NPUSH + j, 0, 0);
        Pull p; pull_issue(p, c.own, st.slot, st.meta, c.lane); unpack(d, p, c.own, st.slot, st.meta, c.lane, c.kp);
    } else fill_nan(d);
    store_operand(c, d, st.p0, st.p1, st.p2, st.p3);
    wait_vm0();
    count_operand(c, st.p0, st.p1, st.p2);
    JLOG(link, 1024 - NPUSH + j, 0, 1);
}

__device__ __forceinline__ void acc_add(float (&acc)[64], const u32x4 (&d)[8]) {
#pragma unroll
    for (int q = 0; q < 32; ++q) { const u32 v = d[q >> 2][q & 3]; acc[2 * q] += __builtin_bit_cast(float, v << 16); acc[2 * q + 1] += __builtin_bit_cast(float, v & 0xFFFF0000u); }
}
__device__ __forceinline__ void write_out(const Params& P, const Ctx& c, const float (&acc)[64], int a, int cj, int jb) {
    const int w = jb >> 1, ai = jb & 1, lane = c.lane;
    GAS char* o = (GAS char*)P.out + (size_t)(a * 256 + ai * 128 + (w >> 2) * 64 + (lane & 15)) * 16384 + (cj * 256 + (w & 3) * 64 + 8 * (lane >> 4)) * 2;
#pragma unroll
    for (int f = 0; f < 8; ++f) {
        u32x4 v;
#pragma unroll
        for (int e = 0; e < 4; ++e) v[e] = cvt_pk_bf16(acc[f * 8 + 2 * e], acc[f * 8 + 2 * e + 1]);
        *(GAS u32x4*)(o + (size_t)(f >> 1) * 16 * 16384 + (f & 1) * 64) = v;
    }
}
__device__ __forceinline__ void copy_list(const Params& P, Ctx& c, int grp, int widx, int wpl) {
    const int r = c.rank;
    const u32 tbl = (grp == 0 ? P.plist[0][(c.lane >> 2) & 15] : P.plist[1 + ((c.lane >> 4) & 1)][(c.lane >> 2) & 3]) >> (8 * (c.lane & 3));
    auto gen = [&](u32 job, int) {
        const int j = (int)job, jb = j & 15;
        Step st; st.meta = 0;
        if (grp == 0) {
            const int pt = __builtin_amdgcn_readlane((int)tbl, (j >> 4) & 63) & 63;
            st.foff = F_P + (u32)(pt * 16 + jb) * 8u; st.slot = OFF_OUT + c.par * OUT_BYTES + (u32)(pt * 16 + jb) * SLOT;
            st.p0 = (int)(OFF_OWN + (u32)((pt * 4 + 3) * 16 + jb) * SLOT); st.p1 = r; st.p2 = (int)(F_OWN + (u32)((pt * 4 + 3) * 16 + jb) * 8u);
        } else {
            const int second = j >= NRP * 16, v = ((j >> 4) - (second ? NRP : 0)) & 15, path = (grp - 1) ^ second, rs = path * 16 + v;
            const int pt = __builtin_amdgcn_readlane((int)tbl, rs) & 63;
            if (!second) {
                st.foff = F_R + (u32)(rs * 16 + jb) * 8u; st.slot = OFF_OUT + c.par * OUT_BYTES + (u32)(pt * 16 + jb) * SLOT;
                st.p0 = (int)(OFF_REL + c.par * REL_BYTES + (u32)(rs * 16 + jb) * SLOT); st.p1 = r ^ (path ? 1 : 3); st.p2 = (int)(F_P + (u32)(pt * 16 + jb) * 8u);
            } else {
                st.foff = F_P + (u32)(pt * 16 + jb) * 8u; st.slot = OFF_REL + c.par * REL_BYTES + (u32)(rs * 16 + jb) * SLOT;
                st.p0 = (int)(OFF_OWN + (u32)((pt * 4 + 3) * 16 + jb) * SLOT); st.p1 = r; st.p2 = (int)(F_OWN + (u32)((pt * 4 + 3) * 16 + jb) * 8u);
            }
        }
        st.p3 = 0; st.sel = 0;
        return st;
    };
    auto act = [&](Step& st, const Pull& p, rsrc_t rs, bool ok, auto mid) {
        mid();
        if (ok) copy_store(c, p, rs, st.slot, (u32)st.p0, st.meta);
        else { u32x4 d[8]; fill_nan(d); store_raw(d, c.own, (u32)st.p0, c.lane); st.meta = META_RAW; }
    };
    auto pub = [&](const Step& st) { post_flag(P, c, st.p1, (u32)st.p2, st.meta); };
    const rsrc_t rs = mk_rsrc(P.ws[(r ^ (grp == 0 ? 2 : grp == 1 ? 1 : 3)) & 7]);
    pull_list<1>(c, 6 + grp, (u32)widx, (u32)wpl, grp == 0 ? ND * 16u : NRP * 32u, rs, rs, gen, act, pub);
}
__device__ __forceinline__ void reduce_list(const Params& P, Ctx& c, int h, int widx, int wpl) {
    const int r = c.rank;
    float acc[64];
    auto gen = [&](u32 ujob, int sub) {
        const int job = (int)ujob, sl = (sub ^ job) & 1, jb = job & 15, ot2 = job >> 4;
        const int a = 2 * h + (ot2 & 1), j = ot2 >> 1, ot = a * 16 + j, p = 2 * h + sl, ti = 4 + 2 * p + (a & 1);
        Step st; st.meta = 0;
        st.foff = F_D3 + (u32)((ot * 2 + sl) * 16 + jb) * 8u; st.sel = sl;
        st.slot = OFF_OUT + c.par * OUT_BYTES + (u32)((ti * 16 + j) * 16 + jb) * SLOT;
        st.p0 = sub; st.p1 = a; st.p2 = (1 - (r >> 2)) * 16 + j; st.p3 = jb;
        return st;
    };
    auto act = [&](Step& st, const Pull& p, rsrc_t rs, bool ok, auto) {
        u32x4 d[8];
        if (ok) unpack(d, p, rs, st.slot, st.meta, c.lane, c.kp); else fill_nan(d);
        if (st.p0 == 0) {
#pragma unroll
            for (int q = 0; q < 64; ++q) acc[q] = 0.f;
        }
        acc_add(acc, d);
        if (st.p0 == 1) write_out(P, c, acc, st.p1, st.p2, st.p3);
    };
    auto pub = [&](const Step&) {};
    pull_list<2>(c, 9 + h, (u32)widx, (u32)wpl, 512u, mk_rsrc(P.ws[(r ^ d3_of_pair(2 * h)) & 7]), mk_rsrc(P.ws[(r ^ d3_of_pair(2 * h + 1)) & 7]), gen, act, pub);
}
__device__ __forceinline__ void own_job(const Params& P, const Ctx& c, int n) {
    const int pt = n >> 4, jb = n & 15;
    kp_clear(c.kp, c.lane);
    float acc[64];
#pragma unroll
    for (int i = 0; i < 64; ++i) acc[i] = 0.f;
    for (int part = 0; part < 4; ++part) {
        u32 meta; u32x4 d[8];
        if (wait_flag(c, F_OWN + (u32)((pt * 4 + part) * 16 + jb) * 8u, meta)) {
            Pull p; const u32 slot = OFF_OWN + (u32)((pt * 4 + part) * 16 + jb) * SLOT;
            pull_issue(p, c.own, slot, meta, c.lane); unpack(d, p, c.own, slot, meta, c.lane, c.kp);
        } else fill_nan(d);
        acc_add(acc, d);
    }
    write_out(P, c, acc, pt >> 4, (c.rank >> 2) * 16 + (pt & 15), jb);
}

struct GJob { int ti, tj, t0, nt; };

#define STAMP(i) do {} while (0)

__device__ __forceinline__ void gemm_role(const Params& P, const Ctx& c, LAS unsigned char* lds, int xcd, int idx) {
    const int tid = threadIdx.x, wid = (int)rfl(tid >> 6), lane = c.lane, wr = wid >> 2, wc = wid & 3;
    constexpr int HTB = 16384;
    const u32 voff = (u32)tid * 16u;
    const unsigned ldsw = (unsigned)wid * 1024u;
    const int aoff = wr * 8192 + lane * 16, boff = wc * 4096 + lane * 16;
    LAS volatile int* ctl = (LAS volatile int*)(lds + LDS_CTL);
    int avail = 0; bool bad = false;
#define G_SA(b, h) (((b) * 2 + (h)) * HTB)
#define G_SB(b, h) ((4 + (b) * 2 + (h)) * HTB)
#define G_STAGE(bufoff, gbase) do { _Pragma("unroll") for (int _i = 0; _i < 2; ++_i) \
        __builtin_amdgcn_global_load_lds((GAS const u32*)((GAS const char*)(gbase) + voff + _i * 8192), (LAS u32*)(lds + (bufoff) + ldsw + _i * 8192), 16, 0, AUX_DEV); } while (0)
#define G_LDA(dst, b, h) do { _Pragma("unroll") for (int m = 0; m < 4; ++m) _Pragma("unroll") for (int k = 0; k < 2; ++k) dst[m][k] = *(const LAS bf16x8*)(lds + G_SA(b, h) + aoff + m * 2048 + k * 1024); } while (0)
#define G_LDB(dst, b, h) do { _Pragma("unroll") for (int n = 0; n < 2; ++n) _Pragma("unroll") for (int k = 0; k < 2; ++k) dst[n][k] = *(const LAS bf16x8*)(lds + G_SB(b, h) + boff + n * 2048 + k * 1024); } while (0)
#define G_MMA(ai, bj, At, Bt) do { __builtin_amdgcn_s_setprio(1); _Pragma("unroll") for (int m = 0; m < 4; ++m) _Pragma("unroll") for (int n = 0; n < 2; ++n) _Pragma("unroll") for (int k = 0; k < 2; ++k) \
        acc[ai][bj][m][n] = __builtin_amdgcn_mfma_f32_16x16x32_bf16(Bt[n][k], At[m][k], acc[ai][bj][m][n], 0, 0, 0); __builtin_amdgcn_s_setprio(0); } while (0)
#define G_WAIT_V(n) asm volatile("s_waitcnt vmcnt(" #n ")" ::: "memory")
#define G_WAIT_L(n) asm volatile("s_waitcnt lgkmcnt(" #n ")" ::: "memory")
#define G_BAR __builtin_amdgcn_s_barrier()
#define G_SCHED __builtin_amdgcn_sched_barrier(0)
    auto ensure = [&](const GJob& j, int tneed) {
        if (avail >= tneed) return;
        if (wid == 0) {
            GAS const u32* ra = (GAS const u32*)(c.ws + OFF_RDYA) + j.ti * 64 + lane;
            GAS const u32* rb = (GAS const u32*)(c.ws + OFF_RDYB) + j.tj * 64 + lane;
            GAS u32* tmo = (GAS u32*)(c.ws + OFF_TMO);
            const u32 target = 4u * c.call;
            const u64 below = j.t0 ? ((1ull << j.t0) - 1ull) : 0ull;
            const u64 t0 = __builtin_amdgcn_s_memrealtime();
            int av, bd = 0;
            for (u32 it = 1;; ++it) {
                const u32 a = __hip_atomic_load(ra, __ATOMIC_RELAXED, __HIP_MEMORY_SCOPE_AGENT), b = __hip_atomic_load(rb, __ATOMIC_RELAXED, __HIP_MEMORY_SCOPE_AGENT);
                const u64 ok = __builtin_amdgcn_ballot_w64((int)(a - target) >= 0 && (int)(b - target) >= 0) | below;
                av = ok == ~0ull ? 64 : __builtin_ctzll(~ok);
                if (av >= tneed) break;
                if ((it & GR_TMOMASK) == 0 && (rfl(__hip_atomic_load(tmo, __ATOMIC_RELAXED, __HIP_MEMORY_SCOPE_AGENT)) == c.call || __builtin_amdgcn_s_memrealtime() - t0 > DEADLINE)) {
                    __hip_atomic_store(tmo, c.call, __ATOMIC_RELAXED, __HIP_MEMORY_SCOPE_AGENT); av = 64; bd = 1; break;
                }
                __builtin_amdgcn_s_sleep(GR_GSLEEP);
            }
            if (lane == 0) { ctl[0] = av; if (bd) ctl[1] = 1; }
            G_WAIT_L(0);
        }
        asm volatile("" ::: "memory"); G_BAR; asm volatile("" ::: "memory");
        avail = (int)rfl((u32)ctl[0]); bad = rfl((u32)ctl[1]) != 0;
    };
    GJob cur, nxt;
    cur.ti = 6 * (xcd & 1) + idx % 6; cur.tj = GR_XGRP ? 4 * (xcd >> 1) + idx / 6 : (xcd >> 1) + 4 * (idx / 6); cur.t0 = 0; cur.nt = 64;
    { const int col = xcd * 6 + (idx >> 2), third = col >> 4; nxt.ti = 12 + (idx & 3); nxt.tj = col & 15; nxt.t0 = 22 * third; nxt.nt = third == 2 ? 20 : 22; }
    f32x4 acc[2][2][4][2];
    bf16x8 At[4][2], B0[2][2], B1[2][2];
    constexpr size_t kstep = 32768, hstep = 16384;
    for (int ui = 0; ui < 2; ++ui) {
        const bool has_next = ui == 0;
#pragma unroll
        for (int a = 0; a < 2; ++a)
#pragma unroll
            for (int b = 0; b < 2; ++b)
#pragma unroll
                for (int m = 0; m < 4; ++m)
#pragma unroll
                    for (int n = 0; n < 2; ++n) acc[a][b][m][n] = (f32x4){0.f, 0.f, 0.f, 0.f};
        const char* cA = c.ws + OFF_A + (size_t)(cur.ti * 64 + cur.t0) * kstep;
        const char* cB = c.ws + OFF_B + (size_t)(cur.tj * 64 + cur.t0) * kstep;
        avail = 0;
        ensure(cur, cur.t0 + 2);
        G_STAGE(G_SB(0, 0), cB); G_STAGE(G_SA(0, 0), cA); G_STAGE(G_SB(0, 1), cB + hstep); G_STAGE(G_SA(0, 1), cA + hstep);
        if (wr == 1) G_BAR;
        G_WAIT_V(4); G_BAR;
        G_STAGE(G_SB(1, 0), cB + kstep); G_STAGE(G_SA(1, 0), cA + kstep); G_STAGE(G_SB(1, 1), cB + hstep + kstep);
        G_WAIT_V(6); G_BAR;
        if (ui == 0) STAMP(2);
        const int nt = cur.nt;
        for (int t = 0; t < nt; t += 2) {
            const bool last = (t == nt - 2);
            const char* a1 = cA + (size_t)(t + 1) * kstep;
            const char* a2 = cA + (size_t)(last ? t : t + 2) * kstep; const char* b2 = cB + (size_t)(last ? t : t + 2) * kstep;
            const char* a3 = a2 + kstep; const char* b3 = b2 + kstep;
            if (!last) ensure(cur, cur.t0 + t + 4);
            G_LDB(B0, 0, 0); G_SCHED; G_LDA(At, 0, 0); G_STAGE(G_SA(1, 1), a1 + hstep);
            G_WAIT_L(8); G_BAR; G_WAIT_L(0); G_MMA(0, 0, At, B0); G_BAR; G_SCHED;
            G_LDB(B1, 0, 1); G_STAGE(G_SB(0, 0), b2);
            G_BAR; G_WAIT_L(0); G_MMA(0, 1, At, B1); G_BAR;
            G_LDA(At, 0, 1); G_STAGE(G_SA(0, 0), a2);
            G_BAR; G_WAIT_L(0); G_MMA(1, 0, At, B0); G_BAR; G_SCHED;
            G_STAGE(G_SB(0, 1), b2 + hstep);
            G_WAIT_V(6); G_BAR; G_MMA(1, 1, At, B1); G_BAR;
            G_LDB(B0, 1, 0); G_SCHED; G_LDA(At, 1, 0); G_STAGE(G_SA(0, 1), a2 + hstep);
            G_WAIT_L(8); G_BAR; G_WAIT_L(0); G_MMA(0, 0, At, B0); G_BAR; G_SCHED;
            G_LDB(B1, 1, 1); G_STAGE(G_SB(1, 0), b3);
            G_BAR; G_WAIT_L(0); G_MMA(0, 1, At, B1); G_BAR;
            G_LDA(At, 1, 1); G_STAGE(G_SA(1, 0), a3);
            G_BAR; G_WAIT_L(0); G_MMA(1, 0, At, B0); G_BAR; G_SCHED;
            G_STAGE(G_SB(1, 1), b3 + hstep);
            G_WAIT_V(6); G_BAR; G_MMA(1, 1, At, B1); G_BAR;
        }
        if (wr == 0) G_BAR;
        G_WAIT_V(0); G_BAR;
        STAMP(3 + 2 * ui);
        {
            u32 meta[2], slot[2];
#pragma unroll
            for (int ai = 0; ai < 2; ++ai) {
                u32x4 d[8];
#pragma unroll
                for (int m = 0; m < 4; ++m)
#pragma unroll
                    for (int bj = 0; bj < 2; ++bj) {
                        const f32x4 v0 = acc[ai][bj][m][0], v1 = acc[ai][bj][m][1];
                        d[m * 2 + bj] = (u32x4){cvt_pk_bf16(v0[0], v0[1]), cvt_pk_bf16(v0[2], v0[3]), cvt_pk_bf16(v1[0], v1[1]), cvt_pk_bf16(v1[2], v1[3])};
                    }
                if (bad) fill_nan(d);
                const int jb = wid * 2 + ai;
                if (has_next) { slot[ai] = OFF_OUT + c.par * OUT_BYTES + (u32)((cur.ti * 16 + cur.tj) * 16 + jb) * SLOT; meta[ai] = pack_store(d, c.own, slot[ai], lane, c.kp); }
                else { slot[ai] = OFF_OWN + (u32)((((cur.ti - 12) * 16 + cur.tj) * 4 + (cur.t0 != 0) + (cur.t0 == 44)) * 16 + jb) * SLOT; store_raw(d, c.own, slot[ai], lane); meta[ai] = META_RAW; }
            }
            wait_vm0();
#pragma unroll
            for (int ai = 0; ai < 2; ++ai) {
                const int jb = wid * 2 + ai, r = c.rank;
                int dst; u32 foff;
                if (!has_next) { dst = r; foff = F_OWN + (u32)((((cur.ti - 12) * 16 + cur.tj) * 4 + (cur.t0 != 0) + (cur.t0 == 44)) * 16 + jb) * 8u; }
                else if (cur.ti < 4) {
                    const int pt = cur.ti * 16 + cur.tj;
                    const u32 route = (P.proute[pt >> 2] >> (8 * (pt & 3))) & 255u;
                    if (route == 255u) { dst = r ^ 2; foff = F_P + (u32)(pt * 16 + jb) * 8u; }
                    else { dst = r ^ ((route & 16u) ? 3 : 1); foff = F_R + ((route & 31u) * 16u + (u32)jb) * 8u; }
                } else {
                    const int p = (cur.ti - 4) >> 1, a = 2 * (p >> 1) + ((cur.ti - 4) & 1);
                    dst = r ^ d3_of_pair(p); foff = F_D3 + (u32)(((a * 16 + cur.tj) * 2 + (p & 1)) * 16 + jb) * 8u;
                }
                post_flag(P, c, dst, foff, meta[ai]);
            }
        }
        STAMP(4 + 2 * ui);
        cur = nxt;
    }
#undef G_SA
#undef G_SB
#undef G_STAGE
#undef G_LDA
#undef G_LDB
#undef G_MMA
#undef G_WAIT_V
#undef G_WAIT_L
#undef G_BAR
#undef G_SCHED
}

__global__ __launch_bounds__(NTH, 2) void gemm_rs_kernel(const Params P) {
    extern __shared__ __attribute__((aligned(16))) unsigned char shm[];
    LAS unsigned char* lds = (LAS unsigned char*)shm;
    const int wg = blockIdx.x, xcd = wg & 7, idx = wg >> 3, tid = threadIdx.x, wid = (int)rfl(tid >> 6);
    const bool gemm = idx < NGEMM / 8;
    const int gw = (xcd * (NGEMM / 8) + idx) * 8 + wid;
    const int cw = wid * NCOMM + (idx - NGEMM / 8) * 8 + xcd;
    int pj[5] = {-1, -1, -1, -1, -1}, pdelay = 0;
    u32x4 pin[8];
    if (gemm) {
        const int grp = GR_XGRP ? xcd >> 1 : idx / 6, u = GR_TOPW < 0 ? (xcd * 6 + idx % 6) * 8 + wid : wid * 48 + (GR_XGRP ? idx * 2 + (xcd & 1) : (idx % 6) * 8 + xcd);
#pragma unroll
        for (int r = 0; r < 5; ++r) pj[r] = pack_pos(grp, u, r);
    }
    u32x8 head; u32 oe[5]; u64 tstart;
    static_assert(offsetof(Params, order) == 0x1c0 && offsetof(Params, rank) == 24, "argument layout");
    asm volatile("s_load_dwordx8 %0, %7, 0x0\n\ts_load_dword %1, %7, %8 offset:0x1c0\n\ts_load_dword %2, %7, %9 offset:0x1c0\n\ts_load_dword %3, %7, %10 offset:0x1c0\n\t"
                 "s_load_dword %4, %7, %11 offset:0x1c0\n\ts_load_dword %5, %7, %12 offset:0x1c0\n\ts_memrealtime %6\n\ts_waitcnt lgkmcnt(0)"
                 : "=&s"(head), "=&s"(oe[0]), "=&s"(oe[1]), "=&s"(oe[2]), "=&s"(oe[3]), "=&s"(oe[4]), "=&s"(tstart)
                 : "s"(__builtin_amdgcn_kernarg_segment_ptr()), "s"((pj[0] & 1023) >> 1 << 2), "s"((pj[1] & 1023) >> 1 << 2), "s"((pj[2] & 1023) >> 1 << 2),
                   "s"((pj[3] & 1023) >> 1 << 2), "s"((pj[4] & 1023) >> 1 << 2));
    Ctx c;
    c.x = (const void*)(((u64)head[1] << 32) | head[0]); c.w = (const void*)(((u64)head[3] << 32) | head[2]);
    c.rank = (int)(head[6] & 7u); c.lane = tid & 63; c.ord0 = head[7];
    c.ws = (char*)(((u64)head[5] << 32) | head[4]); c.own = mk_rsrc(c.ws);
    c.kp = lds + wid * 16384;
    if (gemm) {
        int urg = 0, dummy;
        pj[0] = pack_pick(pj[0], oe[0], urg);
#pragma unroll
        for (int r = 1; r < 5; ++r) pj[r] = pack_pick(pj[r], oe[r], dummy);
        pdelay = (urg * 4 - 16) * PACK_TAU / 16; pdelay = pdelay < 0 ? 0 : pdelay > PACK_DMAX ? PACK_DMAX : pdelay;
    }
    if (pj[0] >= 0 && pdelay == 0) pack_issue(c, c.lane, c.kp, pj[0], pin);
    LAS volatile u32* ctl = (LAS volatile u32*)(lds + LDS_CTL);
    if (tid == 0) {
        ctl[2] = __hip_atomic_fetch_add((GAS u32*)(c.ws + OFF_CALL + wg * 64), 1u, __ATOMIC_RELAXED, __HIP_MEMORY_SCOPE_AGENT) + 1u;
        ctl[0] = 0; ctl[1] = 0;
    }
    asm volatile("s_waitcnt lgkmcnt(0)" ::: "memory"); __builtin_amdgcn_s_barrier(); asm volatile("" ::: "memory");
    c.call = rfl(ctl[2]); c.par = c.call & 1u;
    STAMP(0);
    if (pdelay) {
        while (__builtin_amdgcn_s_memrealtime() - tstart < (u64)pdelay) __builtin_amdgcn_s_sleep(1);
        pack_issue(c, c.lane, c.kp, pj[0], pin);
    }
    pack_list(P, c, pj[0], pj[1], pj[2], pj[3], pj[4], pin);
    STAMP(1);
    if (gemm) {
        __syncthreads();
        gemm_role(P, c, lds, xcd, idx);
        if (gw < 1024) own_job(P, c, gw);
        STAMP(7);
    } else {
        const int ow = cw - O_FIRST;
        const int pw = cw - PUSH_FIRST;
        if (cw >= IN_WAVES && (ow < 0 || ow >= O_WAVES) && (pw < 0 || pw >= 6 * NPUSH)) return;
        kp_clear(c.kp, c.lane);
        if (pw >= 0 && pw < 6 * NPUSH) push_job(P, c, pw % 6, pw / 6);
#define GR_INMASK 63
        if (cw < IN_WAVES && ((GR_INMASK >> (cw % 6)) & 1)) in_list(P, c, cw % 6, cw / 6, IN_WPL);
        STAMP(3);
        if (ow >= 0 && ow < O_WAVES) {
            if (ow < 3 * C_WPL) copy_list(P, c, ow % 3, ow / 3, C_WPL);
            else reduce_list(P, c, (ow - 3 * C_WPL) & 1, (ow - 3 * C_WPL) >> 1, 2 * X_WPL);
        }
        STAMP(7);
    }
}
}

extern "C" void kernel_launch(int rank, int nranks, ncclComm_t comm, hipStream_t stream,
                              void* const* d_in, const int* in_sizes, int n_in,
                              void* d_out, int out_size,
                              const int* in_dtypes, int out_dtype,
                              void* const* d_out_all,
                              void* const* d_ws_all, size_t ws_bytes) {
    (void)comm; (void)in_sizes; (void)n_in; (void)out_size; (void)in_dtypes; (void)out_dtype; (void)d_out_all;
    if (nranks != gr::NR || rank < 0 || rank >= gr::NR || ws_bytes < gr::WS_NEED) { fprintf(stderr, "kernel_launch: unexpected configuration\n"); return; }
    static bool init[gr::NR] = {};
    if (!init[rank]) {
        if (hipFuncSetAttribute((const void*)gr::gemm_rs_kernel, hipFuncAttributeMaxDynamicSharedMemorySize, gr::LDS_BYTES) != hipSuccess) { fprintf(stderr, "hipFuncSetAttribute failed\n"); return; }
        init[rank] = true;
    }
    gr::Params P;
    P.x = d_in[0]; P.w = d_in[1]; P.out = d_out; P.own = d_ws_all[rank]; P.rank = rank;
        for (int i = 0; i < 6; ++i) P.pad[i] = 0;
    for (int i = 0; i < gr::NR; ++i) P.ws[i] = d_ws_all[i];
    int blockpos[64];
    {
        int key[64]; unsigned char ord[64];
        for (int b = 0; b < 64; ++b) {
            const int kind = b & 7;
            key[b] = GR_CH * (b >> 3) + (kind == 4 ? GR_D1 : kind == 5 ? GR_D2 : kind == 6 ? GR_D3 : kind == 7 ? GR_DOWN : 0);
        }
        for (int b = 0; b < 64; ++b) {
            int pos = 0;
            for (int o = 0; o < 64; ++o) pos += key[o] < key[b] || (key[o] == key[b] && o < b);
            blockpos[b] = pos; ord[pos] = (unsigned char)b;
        }
        for (int i = 0; i < 16; ++i) P.inord[i] = ord[4 * i] | (ord[4 * i + 1] << 8) | (ord[4 * i + 2] << 16) | ((unsigned)ord[4 * i + 3] << 24);
        P.ord0 = P.inord[0];
    }
    {
        unsigned char route[64], list[3][64] = {};
        int n[3] = {0, 0, 0};
        for (int k = 0; k < 64; ++k) {
            const int tj = k >> 2, ti = k & 3, pt = ti * 16 + tj;
            int way = (tj & 1) == 0 ? 0 : 1 + ((tj >> 1) & 1);
            if (way && n[way] >= gr::NRP) way = 0;
            route[pt] = way == 0 ? 255 : (unsigned char)((way - 1) * 16 + n[way]);
            list[way][n[way]++] = (unsigned char)pt;
        }
        auto word = [](const unsigned char* b) { return b[0] | (b[1] << 8) | (b[2] << 16) | ((unsigned)b[3] << 24); };
        for (int i = 0; i < 16; ++i) { P.proute[i] = word(route + 4 * i); for (int w = 0; w < 3; ++w) P.plist[w][i] = word(list[w] + 4 * i); }
    }
    {
        unsigned short ord[1024]; int need[1024], first[1026] = {};
        for (int j = 0; j < 1024; ++j) { need[j] = gr::need_of(rank, j, blockpos); ++first[need[j] + 1]; }
        for (int v = 0; v < 1025; ++v) first[v + 1] += first[v];
        for (int j = 0; j < 1024; ++j) ord[first[need[j]]++] = (unsigned short)(j | ((need[j] / 4 > 63 ? 63 : need[j] / 4) << 10));
        for (int i = 0; i < 512; ++i) P.order[i] = (unsigned)ord[2 * i] | ((unsigned)ord[2 * i + 1] << 16);
    }
    hipLaunchKernelGGL(gr::gemm_rs_kernel, dim3(gr::NWG), dim3(gr::NTH), gr::LDS_BYTES, stream, P);
    hipError_t e = hipGetLastError();
    if (e != hipSuccess) fprintf(stderr, "launch failed: %s\n", hipGetErrorString(e));
}
```
